```python
import numpy as np

kernel_name = "gpu_ag_gemm_m4096_k4096_n4096_bf16_silu"


ABSMAX_RTOL = 0.008

COMPUTE_DTYPE = "bfloat16"
EPILOGUE = "silu"
IN_QUANT = None
OUT_QUANT = None
IN_DTYPES = {"x": "bfloat16", "w_mat": "bfloat16"}
WORLD_SIZE = 8

def _to_bf16(x):
    u = np.ascontiguousarray(x, dtype=np.float32).view(np.uint32)
    return ((u + 0x7FFF + ((u >> 16) & 1)) & 0xFFFF0000).view(np.float32)

def setup_inputs():

    x = _to_bf16(np.random.randn(4096, 4096).astype(np.float32))
    w_mat = _to_bf16((np.random.randn(4096, 4096) / np.sqrt(4096)).astype(np.float32))
    return {"x": x, "w_mat": w_mat}

def _epilogue(y):
    return (y / (1.0 + np.exp(-np.clip(y, -60.0, 60.0)))).astype(np.float32)

def reference(x, w_mat):
    y = (x @ w_mat).astype(np.float32)
    return _epilogue(y)
```

```cpp
#include <hip/hip_runtime.h>
#include <rccl/rccl.h>
#include <cstdio>
#include <cstdint>

#define HIP_CHECK(expr) do { hipError_t _e = (expr); if (_e != hipSuccess) { \
    fprintf(stderr, "HIP error %s at %s:%d\n", hipGetErrorString(_e), __FILE__, __LINE__); \
    return; } } while (0)

typedef unsigned u32;
typedef unsigned long long u64;
typedef u32 u32x4 __attribute__((ext_vector_type(4)));
typedef u32 u32x2 __attribute__((ext_vector_type(2)));
typedef float f32x4 __attribute__((ext_vector_type(4)));
typedef short s16x4 __attribute__((ext_vector_type(4)));
typedef unsigned short u16x2 __attribute__((ext_vector_type(2)));
typedef __bf16 bf16x8 __attribute__((ext_vector_type(8)));
typedef __bf16 bf16x2 __attribute__((ext_vector_type(2)));
typedef float f32x2 __attribute__((ext_vector_type(2)));
#define GAS __attribute__((address_space(1)))
#define LAS __attribute__((address_space(3)))
typedef __amdgpu_buffer_rsrc_t rsrc_t;

#define PUSH_WAVES_PER_WG 1
#define PUSH_WGS 256
#define LAG_STEPS 12

constexpr int NRANKS = 8;
constexpr int STEPS = 32;
constexpr int N_COMM = 256, N_GEMM = 256;
constexpr int N_IN = 4096;
constexpr int N_PART = 2048;
constexpr int N_BLK = 2 * N_IN + N_PART;
constexpr int N_SRC = 2048;
constexpr int PUSH_W = PUSH_WGS * PUSH_WAVES_PER_WG;
constexpr int LAG = LAG_STEPS;

constexpr u32 OFF_CNT = 0;
constexpr u32 OFF_TMO = 16384;
constexpr u32 OFF_FLAG = 32768;
constexpr u32 OFF_DBG = 262144;
constexpr u32 OFF_DATA = 1u << 20;
constexpr u32 OFF_EXT = OFF_DATA + N_BLK * 3072u;
constexpr u32 WS_NEED = OFF_EXT + N_BLK * 1024u;
constexpr u64 DEADLINE_TICKS = 40000000ull;
constexpr int SYS = 17;

struct Params {
    const char* x;
    const char* w;
    char* out;
    char* ws[NRANKS];
    u32 rank;
    u32 pad;
};

__device__ __forceinline__ rsrc_t make_rsrc(const void* p, u32 bytes) {
    return __builtin_amdgcn_make_buffer_rsrc((void*)p, 0, bytes, 0x00020000);
}
__device__ __forceinline__ u32 uni(u32 v) { return __builtin_amdgcn_readfirstlane(v); }
__device__ __forceinline__ u32 rotr(u32 v, u32 s) { return __builtin_amdgcn_alignbit(v, v, s); }

#define STAMP_AT(slot) do {} while (0)

struct Packed { u32x4 l0, l1, n, e; };

__device__ __forceinline__ u32 wave_max_u32(u32 v) {
    v = max(v, (u32)__builtin_amdgcn_update_dpp(0, (int)v, 0x111, 0xf, 0xf, true));
    v = max(v, (u32)__builtin_amdgcn_update_dpp(0, (int)v, 0x112, 0xf, 0xf, true));
    v = max(v, (u32)__builtin_amdgcn_update_dpp(0, (int)v, 0x114, 0xf, 0xf, true));
    v = max(v, (u32)__builtin_amdgcn_update_dpp(0, (int)v, 0x118, 0xf, 0xf, true));
    u32 a = __builtin_amdgcn_readlane(v, 15), b = __builtin_amdgcn_readlane(v, 31);
    u32 c = __builtin_amdgcn_readlane(v, 47), d = __builtin_amdgcn_readlane(v, 63);
    return max(max(a, b), max(c, d));
}

__device__ __forceinline__ u32 pack_block(const u32x4 (&v)[4], Packed& o) {
    u16x2 m = {0, 0};
#pragma unroll
    for (int j = 0; j < 4; ++j)
#pragma unroll
        for (int d = 0; d < 4; ++d)
            m = __builtin_elementwise_max(m, __builtin_bit_cast(u16x2, v[j][d] & 0x7FFF7FFFu));
    u32 mx = wave_max_u32(max((u32)m[0], (u32)m[1]) >> 8);
    u32 base = mx > 7u ? mx - 7u : 0u;
    u32 base4 = base * 0x01010101u;
    u32 lo[8], any = 0;
#pragma unroll
    for (int j = 0; j < 4; ++j) {
        lo[2 * j] = __builtin_amdgcn_perm(v[j][1], v[j][0], 0x06040200u);
        lo[2 * j + 1] = __builtin_amdgcn_perm(v[j][3], v[j][2], 0x06040200u);
        u32 h0 = __builtin_amdgcn_perm(v[j][1], v[j][0], 0x07050301u);
        u32 h1 = __builtin_amdgcn_perm(v[j][3], v[j][2], 0x07050301u);
        u32 n0 = (((h0 | 0x80808080u) - base4) & 0x07070707u) | (h0 & 0x80808080u);
        u32 n1 = (((h1 | 0x80808080u) - base4) & 0x07070707u) | (h1 & 0x80808080u);
        u32 p0 = (n0 + base4) ^ h0, p1 = (n1 + base4) ^ h1;
        o.n[j] = n0 | rotr(n1, 28);
        u32 e = (p0 >> 3) | (p1 << 1);
        o.e[j] = e;
        any |= e;
    }
    o.l0 = u32x4{lo[0], lo[1], lo[2], lo[3]};
    o.l1 = u32x4{lo[4], lo[5], lo[6], lo[7]};
    u64 bal = __ballot(any != 0);
    u32 b32 = (u32)bal | (u32)(bal >> 32);
    u32 mask = (b32 | (b32 >> 16)) & 0xFFFFu;
    return base | (mask << 8);
}

__device__ __forceinline__ u32 ext_off(u32 hdr, u32 lane) {
    u32 in = (hdr >> (8u + (lane & 15u))) & 1u;
    return in ? (((lane & 15u) * 4u + (lane >> 4)) * 16u) : 0x80000000u;
}

__device__ __forceinline__ void store_block(rsrc_t dst, u32 id, const Packed& p, u32 hdr, u32 lane) {
    u32 so = OFF_DATA + id * 3072u;
    __builtin_amdgcn_raw_buffer_store_b128(p.l0, dst, lane * 16u, so, SYS);
    __builtin_amdgcn_raw_buffer_store_b128(p.l1, dst, lane * 16u + 1024u, so, SYS);
    __builtin_amdgcn_raw_buffer_store_b128(p.n, dst, lane * 16u + 2048u, so, SYS);
    if (hdr >> 8)
        __builtin_amdgcn_raw_buffer_store_b128(p.e, dst, ext_off(hdr, lane), OFF_EXT + id * 1024u, SYS);
}

__device__ __forceinline__ void load_block(rsrc_t src, u32 id, Packed& p, u32 hdr, u32 lane) {
    u32 so = OFF_DATA + id * 3072u;
    p.l0 = __builtin_amdgcn_raw_buffer_load_b128(src, lane * 16u, so, SYS);
    p.l1 = __builtin_amdgcn_raw_buffer_load_b128(src, lane * 16u + 1024u, so, SYS);
    p.n = __builtin_amdgcn_raw_buffer_load_b128(src, lane * 16u + 2048u, so, SYS);
    p.e = __builtin_amdgcn_raw_buffer_load_b128(src, ext_off(hdr, lane), OFF_EXT + id * 1024u, SYS);
}

__device__ __forceinline__ void unpack_block(const Packed& p, u32 hdr, u32x4 (&v)[4]) {
    u32 base4 = (hdr & 0xFFu) * 0x01010101u;
    u32 h0[4], h1[4];
#pragma unroll
    for (int j = 0; j < 4; ++j) {
        h0[j] = (p.n[j] & 0x87878787u) + base4;
        h1[j] = (rotr(p.n[j], 4) & 0x87878787u) + base4;
    }
    if (hdr >> 8) {
#pragma unroll
        for (int j = 0; j < 4; ++j) {
            h0[j] ^= (p.e[j] & 0x0F0F0F0Fu) << 3;
            h1[j] ^= (p.e[j] & 0xF0F0F0F0u) >> 1;
        }
    }
#pragma unroll
    for (int j = 0; j < 4; ++j) {
        u32 a = j < 2 ? p.l0[2 * (j & 1)] : p.l1[2 * (j & 1)];
        u32 b = j < 2 ? p.l0[2 * (j & 1) + 1] : p.l1[2 * (j & 1) + 1];
        v[j][0] = __builtin_amdgcn_perm(h0[j], a, 0x05010400u);
        v[j][1] = __builtin_amdgcn_perm(h0[j], a, 0x07030602u);
        v[j][2] = __builtin_amdgcn_perm(h1[j], b, 0x05010400u);
        v[j][3] = __builtin_amdgcn_perm(h1[j], b, 0x07030602u);
    }
}

__device__ __forceinline__ void publish(char* dst_ws, u32 id, u32 call, u32 hdr) {
    __hip_atomic_store((GAS u64*)(dst_ws + OFF_FLAG + id * 8u), ((u64)call << 32) | hdr,
                       __ATOMIC_RELAXED, __HIP_MEMORY_SCOPE_SYSTEM);
}

__device__ __forceinline__ bool give_up(char* myws, u32 call, u64 t0) {
    GAS u32* tmo = (GAS u32*)(myws + OFF_TMO);
    u32 t = uni(__hip_atomic_load(tmo, __ATOMIC_RELAXED, __HIP_MEMORY_SCOPE_AGENT));
    if (t == call || __builtin_amdgcn_s_memrealtime() - t0 > DEADLINE_TICKS) {
        __hip_atomic_store(tmo, call, __ATOMIC_RELAXED, __HIP_MEMORY_SCOPE_AGENT);
        asm volatile("s_waitcnt vmcnt(0)" ::: "memory");
        return true;
    }
    return false;
}

__device__ __forceinline__ float silu(float y) {
    float c = __builtin_amdgcn_fmed3f(y, -60.0f, 60.0f);
    float e = __builtin_amdgcn_exp2f(c * -1.4426950408889634f);
    return y * __builtin_amdgcn_rcpf(1.0f + e);
}
__device__ __forceinline__ u32 cvt2(float a, float b) {
    f32x2 f = {a, b};
    return __builtin_bit_cast(u32, __builtin_convertvector(f, bf16x2));
}

struct Src { u32 off, stride, id, d0, d1, isx; };

__device__ __forceinline__ Src decode_src(u32 e, u32 rank, u32 par) {
    u32 gi = e >> 4, sub = e & 15u, t, type;
    constexpr u32 G0 = 4u * (STEPS - LAG), G1 = G0 + 3u * LAG;
    if (gi < G0) { t = gi >> 2; type = gi & 3u; }
    else if (gi < G1) { u32 q = gi - G0; t = (STEPS - LAG) + q / 3u; type = q % 3u; }
    else { t = (STEPS - LAG) + (gi - G1); type = 3u; }
    u32 kh = sub & 1u;
    Src s;
    if (type < 2u) {
        u32 cg = (sub >> 1) & 3u, slot = type * 2u + (sub >> 3);
        s.off = (2048u * kh + 64u * t + 16u * slot) * 1024u + cg * 256u;
        s.stride = 4096u;
        s.id = par * N_IN + ((8u + rank) * 8u + (cg & 1u) * 4u + slot) * 32u + t;
        s.d0 = ((cg >> 1) << 1) | (kh << 2) | (cg & 1u);
        s.d1 = s.d0 ^ 1u;
        s.isx = 0;
    } else {
        u32 i = (sub >> 1) & 1u, rg = (type - 2u) * 4u + (sub >> 2);
        s.off = (256u * i + 32u * rg) * 8192u + (2048u * kh + 64u * t) * 2u;
        s.stride = 65536u;
        s.id = par * N_IN + (rank * 8u + rg) * 32u + t;
        s.d0 = i | (kh << 2) | ((rg & 1u) << 1);
        s.d1 = s.d0 ^ 2u;
        s.isx = 1;
    }
    return s;
}

__device__ __forceinline__ void load_src(const Params& P, const Src& s, u32 lane, u32x4 (&v)[4]) {
    rsrc_t r = make_rsrc(s.isx ? P.x : P.w, 4u << 20);
    u32 lo = s.isx ? (lane >> 3) * 8192u + (lane & 7u) * 16u : (lane >> 4) * 1024u + (lane & 15u) * 16u;
#pragma unroll
    for (int j = 0; j < 4; ++j)
        v[j] = __builtin_amdgcn_raw_buffer_load_b128(r, lo, s.off + j * s.stride, 0);
}

__global__ void __launch_bounds__(256, 2) ag_gemm_silu(Params P) {
    __shared__ __attribute__((aligned(16))) char smem[65536];
    const u32 lane = threadIdx.x & 63u;
    const u32 wave = uni(threadIdx.x >> 6);
    const u32 gwave = blockIdx.x * 4u + wave;
    const u32 rank = P.rank & 7u;
    char* myws = P.ws[rank];
    const rsrc_t mine = make_rsrc(myws, WS_NEED);
    STAMP_AT(0);

    GAS u32* cnt = (GAS u32*)(myws + OFF_CNT + gwave * 4u);

    if (blockIdx.x < N_COMM) {
        u32x4 v[4];
        const bool pusher = blockIdx.x < PUSH_WGS && wave < PUSH_WAVES_PER_WG;
        u32 e = blockIdx.x * PUSH_WAVES_PER_WG + wave;
        if (pusher) load_src(P, decode_src(e, rank, 0), lane, v);
        const u32 call = uni(__hip_atomic_load(cnt, __ATOMIC_RELAXED, __HIP_MEMORY_SCOPE_AGENT)) + 1u;
        if (lane == 0) __hip_atomic_store(cnt, call, __ATOMIC_RELAXED, __HIP_MEMORY_SCOPE_AGENT);
        const u32 par = call & 1u;
        if (pusher) {
            for (; e < (u32)N_SRC; e += PUSH_W) {
                Src s = decode_src(e, rank, par);
                Packed p;
                u32 hdr = pack_block(v, p);
                if (e + PUSH_W < (u32)N_SRC) load_src(P, decode_src(e + PUSH_W, rank, par), lane, v);
                char* w0 = P.ws[s.d0 & 7u];
                char* w1 = P.ws[s.d1 & 7u];
                store_block(make_rsrc(w0, WS_NEED), s.id, p, hdr, lane);
                store_block(make_rsrc(w1, WS_NEED), s.id, p, hdr, lane);
                asm volatile("s_waitcnt vmcnt(0)" ::: "memory");
                if (lane < 2) publish(lane ? w1 : w0, s.id, call, hdr);
            }
        }
        STAMP_AT(1);
        const u32 unit = gwave;
        const u32 id0 = 2 * N_IN + unit * 2u;
        GAS u64* fl = (GAS u64*)(myws + OFF_FLAG + (id0 + (lane & 1u)) * 8u);
        u64 t0 = __builtin_amdgcn_s_memrealtime();
        u32 hv;
        for (;;) {
            u64 f = __hip_atomic_load(fl, __ATOMIC_RELAXED, __HIP_MEMORY_SCOPE_SYSTEM);
            hv = (u32)f;
            if (__all((u32)(f >> 32) >= call)) break;
            if (give_up(myws, call, t0)) break;
            __builtin_amdgcn_s_sleep(4);
        }
        asm volatile("" ::: "memory");
        STAMP_AT(2);
        const u32 hA = __builtin_amdgcn_readlane(hv, 0), hB = __builtin_amdgcn_readlane(hv, 1);
        Packed pa, pb;
        load_block(mine, id0, pa, hA, lane);
        load_block(mine, id0 + 1u, pb, hB, lane);
        const u32 bad = uni(__hip_atomic_load((GAS u32*)(myws + OFF_TMO), __ATOMIC_RELAXED,
                                              __HIP_MEMORY_SCOPE_AGENT)) == call;
        u32x4 a[4], b[4];
        unpack_block(pa, hA, a);
        unpack_block(pb, hB, b);
        const rsrc_t ro = make_rsrc(P.out, 4u << 20);
        const u32 obase = (unit >> 3) * 32u * 1024u + (unit & 7u) * 128u;
        const u32 olane = (lane & 15u) * 1024u + (lane >> 4) * 16u;
#pragma unroll
        for (int j = 0; j < 4; ++j) {
            u32x4 o;
#pragma unroll
            for (int d = 0; d < 4; ++d) {
                float y0 = __builtin_bit_cast(float, a[j][d] << 16) + __builtin_bit_cast(float, b[j][d] << 16);
                float y1 = __builtin_bit_cast(float, a[j][d] & 0xFFFF0000u) +
                           __builtin_bit_cast(float, b[j][d] & 0xFFFF0000u);
                o[d] = bad ? 0x7FC07FC0u : cvt2(silu(y0), silu(y1));
            }
            __builtin_amdgcn_raw_buffer_store_b128(o, ro, olane, obase + (j >> 1) * 16384u + (j & 1) * 64u, 0);
        }
        STAMP_AT(3);
        return;
    }

    const u32 call = uni(__hip_atomic_load(cnt, __ATOMIC_RELAXED, __HIP_MEMORY_SCOPE_AGENT)) + 1u;
    if (lane == 0) __hip_atomic_store(cnt, call, __ATOMIC_RELAXED, __HIP_MEMORY_SCOPE_AGENT);
    const u32 par = call & 1u;
    const u32 gb = blockIdx.x - N_COMM, xcd = gb & 7u, idx = gb >> 3;
    const u32 tm = (xcd >> 1) * 4u + (idx & 3u), tn = (xcd & 1u) * 8u + (idx >> 2);
    const u32 wm = wave >> 1, wn = wave & 1u;
    const u32 idA = par * N_IN + ((tm >> 1) * 8u + (tm & 1u) * 4u + wave) * 32u;
    const u32 idB = par * N_IN + ((8u + (tn >> 1)) * 8u + (tn & 1u) * 4u + wave) * 32u;
    GAS u64* fl = (GAS u64*)(myws + OFF_FLAG + ((lane < 32u ? idA : idB) + (lane & 31u)) * 8u);
    const u64 t0 = __builtin_amdgcn_s_memrealtime();
    u32 rdy = 0, hv = 0;

    LAS char* lds = (LAS char*)smem;
    const u32 wrA = (32u * wave + (lane >> 3)) * 128u + (((lane & 7u) ^ (lane >> 4)) * 16u);
    const u32 wrB = 16384u + (16u * wave + (lane >> 4)) * 256u + (((lane & 15u) ^ ((lane >> 4) << 2)) * 16u);
    const u32 rdA = (64u * wm + (lane & 15u)) * 128u + (((lane >> 4) ^ ((lane & 15u) >> 1)) * 16u);
    const u32 rdB = 16384u + (8u * (lane >> 4) + ((lane >> 2) & 3u)) * 256u +
                    (((wn * 8u + (lane & 3u)) ^ (((lane >> 2) & 3u) << 2)) * 16u) + ((lane >> 4) & 1u) * 8u;

    f32x4 acc[4][4];
#pragma unroll
    for (int a = 0; a < 4; ++a)
#pragma unroll
        for (int b = 0; b < 4; ++b) acc[a][b] = f32x4{0.f, 0.f, 0.f, 0.f};

    Packed sa[2], sb[2];
    u32 ha[2], hb[2];

    auto fetch = [&](u32 s, Packed& pa, Packed& pb, u32& hA, u32& hB) {
        while (rdy <= s) {
            u64 f = __hip_atomic_load(fl, __ATOMIC_RELAXED, __HIP_MEMORY_SCOPE_SYSTEM);
            hv = (u32)f;
            u64 ok = __ballot((u32)(f >> 32) >= call);
            u32 both = (u32)ok & (u32)(ok >> 32);
            rdy = both == 0xFFFFFFFFu ? 32u : (u32)__builtin_ctz(~both);
            if (rdy > s) break;
            if (give_up(myws, call, t0)) { rdy = 32u; break; }
            __builtin_amdgcn_s_sleep(2);
        }
        asm volatile("" ::: "memory");
        hA = __builtin_amdgcn_readlane(hv, s);
        hB = __builtin_amdgcn_readlane(hv, 32u + s);
        load_block(mine, idA + s, pa, hA, lane);
        load_block(mine, idB + s, pb, hB, lane);
    };
    auto stage = [&](const Packed& pa, const Packed& pb, u32 hA, u32 hB, u32 buf) {
        u32x4 v[4];
        unpack_block(pa, hA, v);
#pragma unroll
        for (int j = 0; j < 4; ++j)
            *(LAS u32x4*)(lds + buf + ((wrA + j * 1024u) ^ ((j & 1) * 64u))) = v[j];
        unpack_block(pb, hB, v);
#pragma unroll
        for (int j = 0; j < 4; ++j) {
            u32x4 t = j < 2 ? v[j] : u32x4{v[j][2], v[j][3], v[j][0], v[j][1]};
            *(LAS u32x4*)(lds + buf + wrB + j * 1024u) = t;
        }
    };
    auto compute = [&](u32 buf) {
#pragma unroll
        for (int kk = 0; kk < 2; ++kk) {
            bf16x8 xf[4], wf[4];
#pragma unroll
            for (int mi = 0; mi < 4; ++mi)
                xf[mi] = *(LAS bf16x8*)(lds + buf + ((rdA ^ (kk * 64u)) + mi * 2048u));
#pragma unroll
            for (int ni = 0; ni < 4; ++ni) {
                u32 a = buf + (rdB ^ ((ni >> 1) * 64u + (ni & 1) * 8u)) + kk * 8192u;
                s16x4 lo = __builtin_amdgcn_ds_read_tr16_b64_v4i16((LAS s16x4*)(lds + a));
                s16x4 hi = __builtin_amdgcn_ds_read_tr16_b64_v4i16((LAS s16x4*)(lds + a + 1024u));
                wf[ni] = __builtin_bit_cast(bf16x8, __builtin_shufflevector(lo, hi, 0, 1, 2, 3, 4, 5, 6, 7));
            }
#pragma unroll
            for (int mi = 0; mi < 4; ++mi)
#pragma unroll
                for (int ni = 0; ni < 4; ++ni)
                    acc[mi][ni] = __builtin_amdgcn_mfma_f32_16x16x32_bf16(wf[ni], xf[mi], acc[mi][ni], 0, 0, 0);
        }
    };

    fetch(0, sa[0], sb[0], ha[0], hb[0]);
    STAMP_AT(1);
    fetch(1, sa[1], sb[1], ha[1], hb[1]);
    stage(sa[0], sb[0], ha[0], hb[0], 0);
    fetch(2, sa[0], sb[0], ha[0], hb[0]);
    for (u32 s = 0; s < (u32)STEPS; s += 2) {
        __syncthreads();
        stage(sa[1], sb[1], ha[1], hb[1], 32768u);
        compute(0);
        if (s + 3 < (u32)STEPS) fetch(s + 3, sa[1], sb[1], ha[1], hb[1]);
        __syncthreads();
        if (s + 2 < (u32)STEPS) stage(sa[0], sb[0], ha[0], hb[0], 0);
        compute(32768u);
        if (s + 4 < (u32)STEPS) fetch(s + 4, sa[0], sb[0], ha[0], hb[0]);
    }
    STAMP_AT(2);

    {
        const u32 owner = (tn >> 1) & 7u;
        char* ows = P.ws[owner];
        const rsrc_t dst = make_rsrc(ows, WS_NEED);
        const u32 row = 512u * (tm >> 1) + 256u * (rank & 1u) + 128u * (tm & 1u) + 64u * wm;
        const u32 col = 256u * ((rank >> 1) & 1u) + 128u * (tn & 1u) + 64u * wn;
        const u32 id = 2 * N_IN + ((row >> 5) * 8u + (col >> 6)) * 2u + (rank >> 2);
        const u32 bad = __hip_atomic_load((GAS u32*)(myws + OFF_TMO), __ATOMIC_RELAXED, __HIP_MEMORY_SCOPE_AGENT);
        u32 hdr[2];
#pragma unroll
        for (int h = 0; h < 2; ++h) {
            u32x4 v[4];
#pragma unroll
            for (int j = 0; j < 4; ++j) {
                const f32x4 c0 = acc[2 * h + (j >> 1)][2 * (j & 1)], c1 = acc[2 * h + (j >> 1)][2 * (j & 1) + 1];
                v[j] = u32x4{cvt2(c0[0], c0[1]), cvt2(c0[2], c0[3]), cvt2(c1[0], c1[1]), cvt2(c1[2], c1[3])};
            }
            Packed p;
            hdr[h] = pack_block(v, p);
            store_block(dst, id + 16u * h, p, hdr[h], lane);
        }
        asm volatile("s_waitcnt vmcnt(0)" ::: "memory");
        if (lane < 2 && bad != call) publish(ows, id + 16u * lane, call, lane ? hdr[1] : hdr[0]);
    }
    STAMP_AT(3);
}

extern "C" void kernel_launch(int rank, int nranks, ncclComm_t comm, hipStream_t stream,
                              void* const* d_in, const int* in_sizes, int n_in,
                              void* d_out, int out_size, const int* in_dtypes, int out_dtype,
                              void* const* d_out_all, void* const* d_ws_all, size_t ws_bytes) {
    if (nranks != NRANKS || (unsigned)rank >= (unsigned)NRANKS || ws_bytes < (size_t)WS_NEED || n_in != 2 ||
        in_sizes[0] != 512 * 4096 || in_sizes[1] != 4096 * 512 || out_size != 4096 * 512) {
        fprintf(stderr, "kernel_launch: unexpected problem shape\n");
        return;
    }
    Params p;
    p.x = (const char*)d_in[0];
    p.w = (const char*)d_in[1];
    p.out = (char*)d_out;
    for (int r = 0; r < NRANKS; ++r) p.ws[r] = (char*)d_ws_all[r];
    p.rank = (u32)rank;
    p.pad = 0;
    hipLaunchKernelGGL(ag_gemm_silu, dim3(N_COMM + N_GEMM), dim3(256), 0, stream, p);
    HIP_CHECK(hipGetLastError());
}
```

```cpp
#include <hip/hip_runtime.h>
#include <rccl/rccl.h>
#include <cstdio>
#include <cstdint>

#define HIP_CHECK(expr) do { hipError_t _e = (expr); if (_e != hipSuccess) { \
    fprintf(stderr, "HIP error %s at %s:%d\n", hipGetErrorString(_e), __FILE__, __LINE__); \
    return; } } while (0)

typedef unsigned u32;
typedef unsigned long long u64;
typedef u32 u32x4 __attribute__((ext_vector_type(4)));
typedef u32 u32x2 __attribute__((ext_vector_type(2)));
typedef float f32x16 __attribute__((ext_vector_type(16)));
typedef short s16x4 __attribute__((ext_vector_type(4)));
typedef unsigned short u16x2 __attribute__((ext_vector_type(2)));
typedef __bf16 bf16x8 __attribute__((ext_vector_type(8)));
typedef __bf16 bf16x2 __attribute__((ext_vector_type(2)));
typedef float f32x2 __attribute__((ext_vector_type(2)));
#define GAS __attribute__((address_space(1)))
#define LAS __attribute__((address_space(3)))
typedef __amdgpu_buffer_rsrc_t rsrc_t;

#define POLL_SLEEP 1
#define CREDITS 32
#define LAG_R 8
#define LAG_B 12

constexpr int NRANKS = 8;
constexpr int STEPS = 32;
constexpr int N_WG = 256;
constexpr int N_IN = 4096;
constexpr int N_PART = 2048;
constexpr int N_BLK = 2 * N_IN + N_PART;
constexpr int N_SRC = 2048;
constexpr int PUSH_W = 128;

constexpr u32 OFF_CNT = 0;
constexpr u32 OFF_TKT = 16512;
constexpr u32 OFF_PTK = 20480;
constexpr u32 OFF_CRD = 24576;
constexpr u32 OFF_TMO = 16384;
constexpr u32 OFF_FLAG = 32768;
constexpr u32 OFF_DBG = 262144;
constexpr u32 OFF_DATA = 1u << 20;
constexpr u32 OFF_EXT = OFF_DATA + N_BLK * 3072u;
constexpr u32 WS_NEED = OFF_EXT + N_BLK * 1024u;
constexpr u64 DEADLINE_TICKS = 40000000ull;
constexpr int SYS = 17;

struct Params {
    const char* x;
    const char* w;
    char* out;
    char* ws[NRANKS];
    u32 rank;
    u32 pad;
    u32 order[32];
};

__device__ __forceinline__ rsrc_t make_rsrc(const void* p, u32 bytes) {
    return __builtin_amdgcn_make_buffer_rsrc((void*)p, 0, bytes, 0x00020000);
}
__device__ __forceinline__ u32 uni(u32 v) { return __builtin_amdgcn_readfirstlane(v); }
__device__ __forceinline__ u32 rotr(u32 v, u32 s) { return __builtin_amdgcn_alignbit(v, v, s); }

#define STAMP_AT(slot) do {} while (0)

#define SB() __builtin_amdgcn_sched_barrier(0)
#define PROF_T(v)
#define PROF_ADD(acc, a, b)
#define PROF_OUT(slot, v)
struct Packed { u32x4 l0, l1, n, e; };

__device__ __forceinline__ u32 wave_max_u32(u32 v) {
    v = max(v, (u32)__builtin_amdgcn_update_dpp(0, (int)v, 0x111, 0xf, 0xf, true));
    v = max(v, (u32)__builtin_amdgcn_update_dpp(0, (int)v, 0x112, 0xf, 0xf, true));
    v = max(v, (u32)__builtin_amdgcn_update_dpp(0, (int)v, 0x114, 0xf, 0xf, true));
    v = max(v, (u32)__builtin_amdgcn_update_dpp(0, (int)v, 0x118, 0xf, 0xf, true));
    u32 a = __builtin_amdgcn_readlane(v, 15), b = __builtin_amdgcn_readlane(v, 31);
    u32 c = __builtin_amdgcn_readlane(v, 47), d = __builtin_amdgcn_readlane(v, 63);
    return max(max(a, b), max(c, d));
}

__device__ __forceinline__ u32 pack_block(const u32x4 (&v)[4], Packed& o) {
    u16x2 m = {0, 0};
#pragma unroll
    for (int j = 0; j < 4; ++j)
#pragma unroll
        for (int d = 0; d < 4; ++d)
            m = __builtin_elementwise_max(m, __builtin_bit_cast(u16x2, v[j][d] & 0x7FFF7FFFu));
    u32 mx = wave_max_u32(max((u32)m[0], (u32)m[1]) >> 8);
    u32 base = mx > 7u ? mx - 7u : 0u;
    u32 base4 = base * 0x01010101u;
    u32 lo[8], any = 0;
#pragma unroll
    for (int j = 0; j < 4; ++j) {
        lo[2 * j] = __builtin_amdgcn_perm(v[j][1], v[j][0], 0x06040200u);
        lo[2 * j + 1] = __builtin_amdgcn_perm(v[j][3], v[j][2], 0x06040200u);
        u32 h0 = __builtin_amdgcn_perm(v[j][1], v[j][0], 0x07050301u);
        u32 h1 = __builtin_amdgcn_perm(v[j][3], v[j][2], 0x07050301u);
        u32 n0 = (((h0 | 0x80808080u) - base4) & 0x07070707u) | (h0 & 0x80808080u);
        u32 n1 = (((h1 | 0x80808080u) - base4) & 0x07070707u) | (h1 & 0x80808080u);
        u32 p0 = (n0 + base4) ^ h0, p1 = (n1 + base4) ^ h1;
        o.n[j] = n0 | rotr(n1, 28);
        u32 e = (p0 >> 3) | (p1 << 1);
        o.e[j] = e;
        any |= e;
    }
    o.l0 = u32x4{lo[0], lo[1], lo[2], lo[3]};
    o.l1 = u32x4{lo[4], lo[5], lo[6], lo[7]};
    u64 bal = __ballot(any != 0);
    u32 b32 = (u32)bal | (u32)(bal >> 32);
    u32 mask = (b32 | (b32 >> 16)) & 0xFFFFu;
    return base | (mask << 8);
}

__device__ __forceinline__ u32 ext_off(u32 hdr, u32 lane) {
    u32 in = (hdr >> (8u + (lane & 15u))) & 1u;
    return in ? (((lane & 15u) * 4u + (lane >> 4)) * 16u) : 0x80000000u;
}

__device__ __forceinline__ void store_planes(rsrc_t dst, u32 id, const Packed& p, u32 lane) {
    u32 vo = OFF_DATA + id * 3072u + lane * 16u;
    __builtin_amdgcn_raw_buffer_store_b128(p.l0, dst, vo, 0, SYS);
    __builtin_amdgcn_raw_buffer_store_b128(p.l1, dst, vo + 1024u, 0, SYS);
    __builtin_amdgcn_raw_buffer_store_b128(p.n, dst, vo + 2048u, 0, SYS);
}
__device__ __forceinline__ void store_ext(rsrc_t dst, u32 id, const Packed& p, u32 hdr, u32 lane) {
    __builtin_amdgcn_raw_buffer_store_b128(p.e, dst, ext_off(hdr, lane) + OFF_EXT + id * 1024u, 0, SYS);
}
__device__ __forceinline__ void store_block(rsrc_t dst, u32 id, const Packed& p, u32 hdr, u32 lane) {
    store_planes(dst, id, p, lane);
    if (hdr >> 8) store_ext(dst, id, p, hdr, lane);
}

#define LDAUX SYS
__device__ __forceinline__ void load_block(rsrc_t src, u32 id, Packed& p, u32 hdr, u32 lane) {
    u32 so = OFF_DATA + id * 3072u;
    p.n = __builtin_amdgcn_raw_buffer_load_b128(src, lane * 16u + 2048u, so, LDAUX);
    p.e = __builtin_amdgcn_raw_buffer_load_b128(src, ext_off(hdr, lane), OFF_EXT + id * 1024u, LDAUX);
    p.l0 = __builtin_amdgcn_raw_buffer_load_b128(src, lane * 16u, so, LDAUX);
    p.l1 = __builtin_amdgcn_raw_buffer_load_b128(src, lane * 16u + 1024u, so, LDAUX);
}

__device__ __forceinline__ void unpack_hi(const Packed& p, u32 hdr, u32 (&h0)[4], u32 (&h1)[4]) {
    u32 base4 = (hdr & 0x7Fu) * 0x01010101u;
#pragma unroll
    for (int j = 0; j < 4; ++j) {
        h0[j] = (p.n[j] & 0x87878787u) + base4;
        h1[j] = (rotr(p.n[j], 4) & 0x87878787u) + base4;
    }
    if (hdr >> 8) {
        const u32x4 e = p.e;
#pragma unroll
        for (int j = 0; j < 4; ++j) {
            h0[j] ^= (e[j] & 0x0F0F0F0Fu) << 3;
            h1[j] ^= (e[j] & 0xF0F0F0F0u) >> 1;
        }
    }
}
__device__ __forceinline__ u32x4 unpack_piece(const Packed& p, const u32 (&h0)[4], const u32 (&h1)[4], int j) {
    u32 a = j < 2 ? p.l0[2 * (j & 1)] : p.l1[2 * (j & 1)];
    u32 b = j < 2 ? p.l0[2 * (j & 1) + 1] : p.l1[2 * (j & 1) + 1];
    return u32x4{__builtin_amdgcn_perm(h0[j], a, 0x05010400u), __builtin_amdgcn_perm(h0[j], a, 0x07030602u),
                 __builtin_amdgcn_perm(h1[j], b, 0x05010400u), __builtin_amdgcn_perm(h1[j], b, 0x07030602u)};
}
__device__ __forceinline__ void unpack_block(const Packed& p, u32 hdr, u32x4 (&v)[4]) {
    u32 h0[4], h1[4];
    unpack_hi(p, hdr, h0, h1);
#pragma unroll
    for (int j = 0; j < 4; ++j) v[j] = unpack_piece(p, h0, h1, j);
}

__device__ __forceinline__ void publish(char* dst_ws, u32 id, u32 call, u32 hdr) {
    __hip_atomic_store((GAS u64*)(dst_ws + OFF_FLAG + id * 8u), ((u64)call << 32) | hdr,
                       __ATOMIC_RELAXED, __HIP_MEMORY_SCOPE_SYSTEM);
}

__device__ __forceinline__ bool give_up(char* myws, u32 call, u64 t0) {
    u64 dt = __builtin_amdgcn_s_memrealtime() - t0;
    if (dt < 100000ull) return false;
    GAS u32* tmo = (GAS u32*)(myws + OFF_TMO);
    u32 t = uni(__hip_atomic_load(tmo, __ATOMIC_RELAXED, __HIP_MEMORY_SCOPE_AGENT));
    if (t == call || dt > DEADLINE_TICKS) {
        __hip_atomic_store(tmo, call, __ATOMIC_RELAXED, __HIP_MEMORY_SCOPE_AGENT);
        asm volatile("s_waitcnt vmcnt(0)" ::: "memory");
        return true;
    }
    return false;
}

__device__ __forceinline__ float silu(float y) {
    float c = __builtin_amdgcn_fmed3f(y, -60.0f, 60.0f);
    float e = __builtin_amdgcn_exp2f(c * -1.4426950408889634f);
    return y * __builtin_amdgcn_rcpf(1.0f + e);
}
__device__ __forceinline__ u32 cvt2(float a, float b) {
    f32x2 f = {a, b};
    return __builtin_bit_cast(u32, __builtin_convertvector(f, bf16x2));
}

struct Src { u32 off, stride, id, d0, d1, isx; };

__device__ __forceinline__ Src decode_src(const Params& P, u32 e, u32 rank, u32 par) {
    u32 gi = (e >> 4) & 127u, sub = e & 15u;
    u32 code = (P.order[gi >> 2] >> ((gi & 3u) * 8u)) & 0xFFu;
    u32 t = code & 31u, type = (code >> 5) & 3u;
    u32 kh = sub & 1u, half = (sub >> 1) & 1u, q = sub >> 2;
    Src s;
    if (type < 2u) {
        s.off = (2048u * kh + 64u * t + 16u * q) * 1024u + (half * 2u + type) * 256u;
        s.stride = 4096u;
        s.id = par * N_IN + ((8u + rank) * 8u + type * 4u + q) * 32u + t;
        s.d0 = (half << 1) | (kh << 2) | (q & 1u);
        s.d1 = s.d0 ^ 1u;
        s.isx = 0;
    } else {
        u32 rg = (type - 2u) * 4u + q;
        s.off = (256u * half + 32u * rg) * 8192u + (2048u * kh + 64u * t) * 2u;
        s.stride = 65536u;
        s.id = par * N_IN + (rank * 8u + rg) * 32u + t;
        s.d0 = half | (kh << 2) | ((q & 1u) << 1);
        s.d1 = s.d0 ^ 2u;
        s.isx = 1;
    }
    return s;
}

__device__ __forceinline__ void load_src(const Params& P, const Src& s, u32 lane, u32x4 (&v)[4]) {
    rsrc_t r = make_rsrc(s.isx ? P.x : P.w, 4u << 20);
    u32 lo = s.isx ? (lane >> 3) * 8192u + (lane & 7u) * 16u : (lane >> 4) * 1024u + (lane & 15u) * 16u;
#pragma unroll
    for (int j = 0; j < 4; ++j)
        v[j] = __builtin_amdgcn_raw_buffer_load_b128(r, lo, s.off + j * s.stride, 0);
}

constexpr u32 SLOT = 32768;
constexpr u32 LDS_CTL = 4 * SLOT;
constexpr u32 LDS_TAB = LDS_CTL + 256;
typedef LAS u32* lds_u32p;

__device__ __forceinline__ u32 prog_read(lds_u32p p) {
    return __hip_atomic_load(p, __ATOMIC_RELAXED, __HIP_MEMORY_SCOPE_WORKGROUP);
}
__device__ __forceinline__ void prog_wait(lds_u32p p, u32 seen, u32 need) {
    if (__builtin_expect(!__all(seen >= need), 0)) {
        while (!__all(prog_read(p) >= need)) __builtin_amdgcn_s_sleep(1);
    }
    asm volatile("" ::: "memory");
}
__device__ __forceinline__ void prog_set(lds_u32p p, u32 v) {
    asm volatile("" ::: "memory");
    __hip_atomic_store(p, v, __ATOMIC_RELAXED, __HIP_MEMORY_SCOPE_WORKGROUP);
    asm volatile("" ::: "memory");
}

__global__ void __launch_bounds__(1024, 4) ag_gemm_silu(Params P) {
    __shared__ __attribute__((aligned(16))) char smem[LDS_TAB + 1024];
    LAS char* lds = (LAS char*)smem;
    const u32 lane = threadIdx.x & 63u;
    const u32 wave = uni(threadIdx.x >> 6);
    const u32 gwave = blockIdx.x * 16u + wave;
    const u32 rank = P.rank & 7u;
    char* myws = P.ws[rank];
    const rsrc_t mine = make_rsrc(myws, WS_NEED);
    STAMP_AT(0);
    lds_u32p prog = (lds_u32p)(lds + LDS_CTL);
    lds_u32p tab = (lds_u32p)(lds + LDS_TAB);
    const u32 xcd = blockIdx.x & 7u, idx = blockIdx.x >> 3;
    const u32 tm = (xcd >> 1) * 4u + (idx & 3u), tn = (xcd & 1u) * 8u + (idx >> 2);

    GAS u32* cnt = (GAS u32*)(myws + OFF_CNT + gwave * 4u);

    if (wave < 4u) {
        u32x4 v[4];
        const bool pusher = wave == 0u && (idx & 1u);
        u32 e = (idx >> 1) * 8u + xcd;
        const u32 tkoff = lane ? 0x80000000u : OFF_TKT;
        u32 tk;
        auto draw = [&](bool on) {
            tk = 1u;
            asm volatile("buffer_atomic_add %0, %1, %2, 0 offen sc0" : "+v"(tk)
                         : "v"(on ? tkoff : 0x80000000u), "s"(mine) : "memory");
        };
        if (pusher) load_src(P, decode_src(P, e, rank, 0), lane, v);
        const u32 cv = __hip_atomic_load(cnt, __ATOMIC_RELAXED, __HIP_MEMORY_SCOPE_AGENT);
        __builtin_amdgcn_s_barrier();
        const u32 call = uni(cv) + 1u;
        if (pusher) draw(true);
        if (lane == 0) __hip_atomic_store(cnt, call, __ATOMIC_RELAXED, __HIP_MEMORY_SCOPE_AGENT);
        const u32 par = call & 1u;
        if (pusher) {
            const u32 tbase = (call - 1u) * (u32)N_SRC - (u32)PUSH_W;
            while (e < (u32)N_SRC) {
                Src s = decode_src(P, e, rank, par);
                Packed p;
                u32 hdr = pack_block(v, p);
                char* w0 = P.ws[s.d0 & 7u];
                char* w1 = P.ws[s.d1 & 7u];
                const rsrc_t r0 = make_rsrc(w0, WS_NEED), r1 = make_rsrc(w1, WS_NEED);
                if (hdr >> 8) { store_ext(r0, s.id, p, hdr, lane); store_ext(r1, s.id, p, hdr, lane); }
                store_planes(r0, s.id, p, lane);
                store_planes(r1, s.id, p, lane);
                asm volatile("s_waitcnt vmcnt(7)" : "+v"(tk) :: "memory");
                e = uni(tk) - tbase;
                if (e < (u32)N_SRC) load_src(P, decode_src(P, e, rank, par), lane, v);
                asm volatile("s_waitcnt vmcnt(0)" : "+v"(v[0]), "+v"(v[1]), "+v"(v[2]), "+v"(v[3]) :: "memory");
                draw(e < (u32)N_SRC);
                if (lane < 2) publish(lane ? w1 : w0, s.id, call, hdr);
            }
            if (lane < 8u)
                __hip_atomic_fetch_add((GAS u32*)(myws + OFF_CRD + lane * 128u), (u32)CREDITS, __ATOMIC_RELAXED,
                                       __HIP_MEMORY_SCOPE_AGENT);
            asm volatile("s_waitcnt vmcnt(2)" : "+v"(tk) :: "memory");
        }
        if (wave == 1u) {
            GAS u64* fa = (GAS u64*)(myws + OFF_FLAG + (par * N_IN + tm * 128u + lane) * 8u);
            GAS u64* fb = (GAS u64*)(myws + OFF_FLAG + (par * N_IN + (16u + tn) * 128u + lane) * 8u);
            const u64 t0 = __builtin_amdgcn_s_memrealtime();
            for (;;) {
                u64 f[4];
                f[0] = __hip_atomic_load(fa, __ATOMIC_RELAXED, __HIP_MEMORY_SCOPE_SYSTEM);
                f[1] = __hip_atomic_load(fa + 64, __ATOMIC_RELAXED, __HIP_MEMORY_SCOPE_SYSTEM);
                f[2] = __hip_atomic_load(fb, __ATOMIC_RELAXED, __HIP_MEMORY_SCOPE_SYSTEM);
                f[3] = __hip_atomic_load(fb + 64, __ATOMIC_RELAXED, __HIP_MEMORY_SCOPE_SYSTEM);
                bool all = true;
#pragma unroll
                for (int i = 0; i < 4; ++i) {
                    const bool ok = (u32)(f[i] >> 32) >= call;
                    all = all && ok;
                    __hip_atomic_store(tab + i * 64 + lane, ok ? ((u32)f[i] | 0x80u) : 0u, __ATOMIC_RELEASE,
                                       __HIP_MEMORY_SCOPE_WORKGROUP);
                }
                if (__all(all)) break;
                if (give_up(myws, call, t0)) {
#pragma unroll
                    for (int i = 0; i < 4; ++i)
                        __hip_atomic_store(tab + i * 64 + lane, 0x80u, __ATOMIC_RELEASE, __HIP_MEMORY_SCOPE_WORKGROUP);
                    break;
                }
                __builtin_amdgcn_s_sleep(POLL_SLEEP);
            }
        }
        STAMP_AT(1);
        const u32 unit = blockIdx.x * 4u + wave;
        const u32 id0 = 2 * N_IN + unit * 2u;
        GAS u64* fl = (GAS u64*)(myws + OFF_FLAG + (id0 + (lane & 1u)) * 8u);
        u64 t0 = __builtin_amdgcn_s_memrealtime();
        u32 hv;
        for (;;) {
            u64 f = __hip_atomic_load(fl, __ATOMIC_RELAXED, __HIP_MEMORY_SCOPE_SYSTEM);
            hv = (u32)f;
            if (__all((u32)(f >> 32) >= call)) break;
            if (give_up(myws, call, t0)) break;
            __builtin_amdgcn_s_sleep(4);
        }
        asm volatile("" ::: "memory");
        STAMP_AT(2);
        const u32 hA = __builtin_amdgcn_readlane(hv, 0), hB = __builtin_amdgcn_readlane(hv, 1);
        Packed pa, pb;
        load_block(mine, id0, pa, hA, lane);
        load_block(mine, id0 + 1u, pb, hB, lane);
        const u32 bad = uni(__hip_atomic_load((GAS u32*)(myws + OFF_TMO), __ATOMIC_RELAXED,
                                              __HIP_MEMORY_SCOPE_AGENT)) == call;
        u32x4 a[4], b[4];
        unpack_block(pa, hA, a);
        unpack_block(pb, hB, b);
        const rsrc_t ro = make_rsrc(P.out, 4u << 20);
        const u32 obase = (unit >> 3) * 32u * 1024u + (unit & 7u) * 128u;
        const u32 olane = (lane & 31u) * 1024u + (lane >> 5) * 16u;
#pragma unroll
        for (int j = 0; j < 4; ++j) {
            u32x4 o;
#pragma unroll
            for (int d = 0; d < 4; ++d) {
                float y0 = __builtin_bit_cast(float, a[j][d] << 16) + __builtin_bit_cast(float, b[j][d] << 16);
                float y1 = __builtin_bit_cast(float, a[j][d] & 0xFFFF0000u) +
                           __builtin_bit_cast(float, b[j][d] & 0xFFFF0000u);
                o[d] = bad ? 0x7FC07FC0u : cvt2(silu(y0), silu(y1));
            }
            __builtin_amdgcn_raw_buffer_store_b128(o, ro, olane + obase + j * 32u, 0, 0);
        }
        STAMP_AT(3);
        return;
    }

    if (wave == 4u) prog[lane] = 0u;
    if (wave >= 8u && wave < 12u) tab[(wave - 8u) * 64u + lane] = 0u;
    const u32 cv = __hip_atomic_load(cnt, __ATOMIC_RELAXED, __HIP_MEMORY_SCOPE_AGENT);
    asm volatile("s_waitcnt lgkmcnt(0)" ::: "memory");
    __builtin_amdgcn_s_barrier();
    const u32 call = uni(cv) + 1u;
    if (lane == 0) __hip_atomic_store(cnt, call, __ATOMIC_RELAXED, __HIP_MEMORY_SCOPE_AGENT);
    const u32 par = call & 1u;

    if (wave >= 8u) {
        const u32 lw = wave & 3u, isB = (wave >> 2) & 1u;
        const u32 t = isB ? 16u + tn : tm;
        const u32 id0 = par * N_IN + (t * 4u + lw) * 32u;
        const u32 wr0 = isB ? 16384u + (16u * lw + (lane >> 4)) * 256u + (((lane & 15u) ^ ((lane >> 4) << 2)) * 16u)
                            : (32u * lw + (lane >> 3)) * 128u + (((lane & 7u) ^ (lane >> 4)) * 16u);
        const u32 wr1 = isB ? wr0 : wr0 ^ 64u;
        const u32 eoff = ((lane & 15u) * 4u + (lane >> 4)) * 16u + OFF_EXT + id0 * 1024u;
        const u32 ebit = 256u << (lane & 15u);
        const u32 doff = lane * 16u + OFF_DATA + id0 * 3072u;
        lds_u32p row = tab + (wave - 8u) * 32u + (lane & 31u);
        lds_u32p mprog = prog + 4u + (lane & 3u);
        Packed R[4];
        u32 H[4] = {0, 0, 0, 0};
#pragma unroll
        for (int q = 0; q < 4; ++q)
            asm volatile("" : "=v"(R[q].l0), "=v"(R[q].l1), "=v"(R[q].n), "=v"(R[q].e));
        u32 rel = 0, hv = 0, mseen = 0;
        auto fetch = [&](int d, u32 s, u32 avail, Packed& q, u32& h) {
            u32 tv, ts, ts2;
            asm volatile("s_cmp_lg_u32 %[rel], %[d]\n\ts_cbranch_scc1 .Lskip%=\n\t"
                         "s_cmp_le_u32 %[av], %[d]\n\ts_cbranch_scc1 .Lskip%=\n\t"
                         "s_add_u32 %[ts], %[s], %[d]\n\t"
                         "v_readlane_b32 %[h], %[hv], %[ts]\n\t"
                         "s_add_u32 %[rel], %[rel], 1\n\t"
                         "s_mul_i32 %[ts2], %[ts], 0xc00\n\t"
                         "s_cmp_lt_u32 %[h], 0x100\n\ts_cbranch_scc1 .Lnoext%=\n\t"
                         "s_lshl_b32 %[ts], %[ts], 10\n\t"
                         "v_and_b32 %[tv], %[h], %[ebit]\n\tv_cmp_ne_u32 vcc, 0, %[tv]\n\t"
                         "v_bfrev_b32 %[tv], 1\n\ts_nop 0\n\t"
                         "v_cndmask_b32 %[tv], %[tv], %[eoff], vcc\n\t"
                         "buffer_load_dwordx4 %[e], %[tv], %[rs], %[ts] offen sc0 sc1\n"
                         ".Lnoext%=:\n\t"
                         "buffer_load_dwordx4 %[n], %[doff], %[rs], %[ts2] offen offset:2048 sc0 sc1\n\t"
                         "buffer_load_dwordx4 %[l0], %[doff], %[rs], %[ts2] offen sc0 sc1\n\t"
                         "buffer_load_dwordx4 %[l1], %[doff], %[rs], %[ts2] offen offset:1024 sc0 sc1\n"
                         ".Lskip%=:"
                         : [e] "+v"(q.e), [n] "+v"(q.n), [l0] "+v"(q.l0), [l1] "+v"(q.l1), [h] "+s"(h),
                           [rel] "+s"(rel), [tv] "=&v"(tv), [ts] "=&s"(ts), [ts2] "=&s"(ts2)
                         : [d] "i"(d), [av] "s"(avail), [s] "s"(s), [hv] "v"(hv), [ebit] "v"(ebit),
                           [eoff] "v"(eoff), [doff] "v"(doff), [rs] "s"(mine)
                         : "vcc", "scc", "memory");
        };
        auto landed = [&](Packed& q) {
            asm volatile("s_cmp_lt_u32 %[rel], 4\n\ts_cbranch_scc1 .La%=\n\ts_waitcnt vmcnt(9)\n\ts_branch .Lz%=\n"
                         ".La%=:\n\ts_cmp_lt_u32 %[rel], 3\n\ts_cbranch_scc1 .Lb%=\n\ts_waitcnt vmcnt(6)\n\ts_branch .Lz%=\n"
                         ".Lb%=:\n\ts_cmp_lt_u32 %[rel], 2\n\ts_cbranch_scc1 .Lc%=\n\ts_waitcnt vmcnt(3)\n\ts_branch .Lz%=\n"
                         ".Lc%=:\n\ts_waitcnt vmcnt(0)\n.Lz%=:"
                         : "+v"(q.e), "+v"(q.n), "+v"(q.l0), "+v"(q.l1) : [rel] "s"(rel) : "scc", "memory");
        };
        auto arrived = [&]() {
            const u32 ok = (u32)__ballot(hv != 0u);
            return ok == 0xFFFFFFFFu ? 32u : (u32)__builtin_ctz(~ok);
        };
        for (u32 s0 = 0; s0 < (u32)STEPS; s0 += 4) {
#pragma unroll
            for (int q = 0; q < 4; ++q) {
                const u32 s = s0 + q;
                u32 rdy = arrived();
                if (__builtin_expect(rdy <= s, 0)) {
                    for (;;) {
                        hv = __hip_atomic_load(row, __ATOMIC_ACQUIRE, __HIP_MEMORY_SCOPE_WORKGROUP);
                        rdy = arrived();
                        if (rdy > s) break;
                        __builtin_amdgcn_s_sleep(1);
                    }
                }
                const u32 avail = rdy - s;
#pragma unroll
                for (int d = 0; d < 4; ++d) fetch(d, s, avail, R[(q + d) & 3], H[(q + d) & 3]);
                Packed& c = R[q];
                LAS char* dst = lds + q * SLOT;
                landed(c);
                rel -= 1u;
                u32 h0[4], h1[4];
                unpack_hi(c, H[q], h0, h1);
                prog_wait(mprog, mseen, s > 3u ? s - 3u : 0u);
                hv = prog_read(row);
                mseen = prog_read(mprog);
                *(LAS u32x4*)(dst + wr0) = unpack_piece(c, h0, h1, 0); SB();
                *(LAS u32x4*)(dst + wr1 + 1024u) = unpack_piece(c, h0, h1, 1); SB();
                *(LAS u32x4*)(dst + wr0 + 2048u) = unpack_piece(c, h0, h1, 2); SB();
                *(LAS u32x4*)(dst + wr1 + 3072u) = unpack_piece(c, h0, h1, 3);
                prog_set(prog + wave, s + 1u);
                SB();
            }
        }
        STAMP_AT(2);
        PROF_OUT(4, pa_); PROF_OUT(5, pb_); PROF_OUT(6, pc_);
        return;
    }

    const u32 wm = (wave >> 1) & 1u, wn = wave & 1u;
    const u32 rdA = (64u * wm + (lane & 31u)) * 128u + (((lane >> 5) ^ ((lane >> 1) & 7u)) * 16u);
    const u32 rdB = 16384u + (8u * (lane >> 5) + ((lane >> 2) & 3u)) * 256u +
                    (((wn * 8u + ((lane >> 4) & 1u) * 2u + (lane & 1u)) ^ (((lane >> 2) & 3u) << 2)) * 16u) +
                    ((lane >> 1) & 1u) * 8u;
    f32x16 acc[2][2];
#pragma unroll
    for (int a = 0; a < 2; ++a)
#pragma unroll
        for (int b = 0; b < 2; ++b)
#pragma unroll
            for (int r = 0; r < 16; ++r) acc[a][b][r] = 0.f;

    lds_u32p lprog = prog + 8u + (lane & 7u);
    u32 lseen = 0;
    bf16x8 X[2][2], W[2][2];
    auto rd_x = [&](u32 buf, int ks, int mi) {
        return *(LAS bf16x8*)(lds + buf + ((rdA ^ (ks * 32u)) + mi * 4096u));
    };
    auto rd_w = [&](u32 buf, int ks, int ni) {
        u32 a = buf + (rdB ^ (ni * 64u)) + ks * 4096u;
        s16x4 lo = __builtin_amdgcn_ds_read_tr16_b64_v4i16((LAS s16x4*)(lds + a));
        s16x4 hi = __builtin_amdgcn_ds_read_tr16_b64_v4i16((LAS s16x4*)(lds + a + 1024u));
        return __builtin_bit_cast(bf16x8, __builtin_shufflevector(lo, hi, 0, 1, 2, 3, 4, 5, 6, 7));
    };
#define MMA(mi, ni, st) acc[mi][ni] = __builtin_amdgcn_mfma_f32_32x32x16_bf16(W[st][ni], X[st][mi], acc[mi][ni], 0, 0, 0)
    auto step = [&](u32 s, u32 buf, u32 nxt, bool more) {
#pragma unroll
        for (int ks = 0; ks < 4; ++ks) {
            const int st = ks & 1, k2 = (ks + 2) & 3;
            const u32 src = ks < 2 ? buf : nxt;
            const bool ld = ks < 2 || more;
            MMA(0, 0, st); SB();
            if (ks == 1 && more) lseen = prog_read(lprog);
            if (ks == 2 && more) prog_set(prog + wave, s + 1u);
            SB();
            MMA(0, 1, st); SB();
            if (ks == 2 && more) {
                PROF_T(q0);
                prog_wait(lprog, lseen, s + 2u);
                PROF_T(q1);
                PROF_ADD(pa_, q0, q1);
            }
            if (ld) X[st][0] = rd_x(src, k2, 0);
            SB();
            MMA(1, 1, st); SB();
            if (ld) W[st][1] = rd_w(src, k2, 1);
            SB();
            MMA(1, 0, st); SB();
            if (ld) { W[st][0] = rd_w(src, k2, 0); X[st][1] = rd_x(src, k2, 1); }
            SB();
        }
    };
    prog_wait(lprog, lseen, 1u);
#pragma unroll
    for (int st = 0; st < 2; ++st) {
        W[st][0] = rd_w(0, st, 0); X[st][0] = rd_x(0, st, 0);
        W[st][1] = rd_w(0, st, 1); X[st][1] = rd_x(0, st, 1);
    }
    for (u32 s = 0; s < (u32)STEPS - 4u; s += 4) {
        step(s, 0, SLOT, true);
        step(s + 1u, SLOT, 2 * SLOT, true);
        step(s + 2u, 2 * SLOT, 3 * SLOT, true);
        step(s + 3u, 3 * SLOT, 0, true);
    }
    step(28u, 0, SLOT, true);
    step(29u, SLOT, 2 * SLOT, true);
    step(30u, 2 * SLOT, 3 * SLOT, true);
    step(31u, 3 * SLOT, 0, false);
    STAMP_AT(2);
    PROF_OUT(4, pa_);

    {
        const u32 owner = (tn >> 1) & 7u;
        char* ows = P.ws[owner];
        const rsrc_t dst = make_rsrc(ows, WS_NEED);
        const u32 row = 512u * (tm >> 1) + 256u * (rank & 1u) + 128u * (tm & 1u) + 64u * wm;
        const u32 col = 256u * ((rank >> 1) & 1u) + 128u * (tn & 1u) + 64u * wn;
        const u32 id = 2 * N_IN + ((row >> 5) * 8u + (col >> 6)) * 2u + (rank >> 2);
        const u32 ptk = (u32)__builtin_amdgcn_raw_ptr_buffer_atomic_add_i32(
            1, mine, lane ? 0x80000000u : OFF_PTK + owner * 128u, 0, 0);
        const u32 bad = __hip_atomic_load((GAS u32*)(myws + OFF_TMO), __ATOMIC_RELAXED, __HIP_MEMORY_SCOPE_AGENT);
        u32 hdr[2];
        Packed p[2];
#pragma unroll
        for (int h = 0; h < 2; ++h) {
            u32x4 v[4];
#pragma unroll
            for (int j = 0; j < 4; ++j) {
                const f32x16 c = acc[h][j >> 1];
                const int o = 8 * (j & 1);
                v[j] = u32x4{cvt2(c[o], c[o + 1]), cvt2(c[o + 2], c[o + 3]), cvt2(c[o + 4], c[o + 5]), cvt2(c[o + 6], c[o + 7])};
            }
            hdr[h] = pack_block(v, p[h]);
        }
        STAMP_AT(4);
        GAS u32* crd = (GAS u32*)(myws + OFF_CRD + owner * 128u);
        if (owner != rank) {
            const u32 need = (call - 1u) * (u32)(PUSH_W * CREDITS) + (uni(ptk) + 1u) * (u32)PUSH_W;
            const u64 t0 = __builtin_amdgcn_s_memrealtime();
            while ((int)(uni(__hip_atomic_load(crd, __ATOMIC_RELAXED, __HIP_MEMORY_SCOPE_AGENT)) - need) < 0) {
                if (give_up(myws, call, t0)) break;
                __builtin_amdgcn_s_sleep(8);
            }
        }
        STAMP_AT(5);
        store_block(dst, id, p[0], hdr[0], lane);
        store_block(dst, id + 16u, p[1], hdr[1], lane);
        asm volatile("s_waitcnt vmcnt(0)" ::: "memory");
        if (lane == 2u) __hip_atomic_fetch_add(crd, (u32)PUSH_W, __ATOMIC_RELAXED, __HIP_MEMORY_SCOPE_AGENT);
        if (lane < 2 && bad != call) publish(ows, id + 16u * lane, call, lane ? hdr[1] : hdr[0]);
    }
    STAMP_AT(3);
}

extern "C" void kernel_launch(int rank, int nranks, ncclComm_t comm, hipStream_t stream,
                              void* const* d_in, const int* in_sizes, int n_in,
                              void* d_out, int out_size, const int* in_dtypes, int out_dtype,
                              void* const* d_out_all, void* const* d_ws_all, size_t ws_bytes) {
    if (nranks != NRANKS || (unsigned)rank >= (unsigned)NRANKS || ws_bytes < (size_t)WS_NEED || n_in != 2 ||
        in_sizes[0] != 512 * 4096 || in_sizes[1] != 4096 * 512 || out_size != 4096 * 512) {
        fprintf(stderr, "kernel_launch: unexpected problem shape\n");
        return;
    }
    Params p;
    p.x = (const char*)d_in[0];
    p.w = (const char*)d_in[1];
    p.out = (char*)d_out;
    for (int r = 0; r < NRANKS; ++r) p.ws[r] = (char*)d_ws_all[r];
    p.rank = (u32)rank;
    p.pad = 0;
    {
        unsigned char ord[128];
        int n = 0, nx[4] = {0, 0, 0, 0};
        const int lag[4] = {0, LAG_R, 0, LAG_B};
        for (int t = 1; t <= STEPS; ++t)
            for (int st = 0; st < 4; ++st)
                while (nx[st] < (t * (STEPS - lag[st]) + STEPS - 1) / STEPS) { ord[n++] = (unsigned char)(st * 32 + nx[st]); ++nx[st]; }
        while (n < 128)
            for (int st = 1; st < 4; st += 2)
                if (nx[st] < STEPS) { ord[n++] = (unsigned char)(st * 32 + nx[st]); ++nx[st]; }
        for (int i = 0; i < 32; ++i)
            p.order[i] = ord[4 * i] | (ord[4 * i + 1] << 8) | (ord[4 * i + 2] << 16) | ((u32)ord[4 * i + 3] << 24);
    }
    hipLaunchKernelGGL(ag_gemm_silu, dim3(N_WG), dim3(1024), 0, stream, p);
    HIP_CHECK(hipGetLastError());
}
```

```cpp
#include <hip/hip_runtime.h>
#include <rccl/rccl.h>
#include <cstdio>
#include <cstdint>

#define HIP_CHECK(expr) do { hipError_t _e = (expr); if (_e != hipSuccess) { \
    fprintf(stderr, "HIP error %s at %s:%d\n", hipGetErrorString(_e), __FILE__, __LINE__); \
    return; } } while (0)

typedef unsigned u32;
typedef unsigned long long u64;
typedef unsigned short u16;
typedef u32 u32x4 __attribute__((ext_vector_type(4)));
typedef float f32x16 __attribute__((ext_vector_type(16)));
typedef short s16x4 __attribute__((ext_vector_type(4)));
typedef unsigned short u16x2 __attribute__((ext_vector_type(2)));
typedef __bf16 bf16x8 __attribute__((ext_vector_type(8)));
typedef __bf16 bf16x2 __attribute__((ext_vector_type(2)));
typedef float f32x2 __attribute__((ext_vector_type(2)));
#define GAS __attribute__((address_space(1)))
#define LAS __attribute__((address_space(3)))
typedef __amdgpu_buffer_rsrc_t rsrc_t;
typedef LAS u32* lds_u32p;

#define CHAIN0 2
#define CHAIN1 2
#define CHAIN01 CHAIN0
#define POLL_SLEEP 24
#define POLL_FAST 28
#define RED_SLEEP 4
#define SPIN 1
#define PUSHERS 116
#define LAG_R 6
#define LAG_B 14
#define PUSH_PER_CU 1
#define PUSH_PRIO 3
#define GATE_AT 1920

constexpr int NRANKS = 8;
constexpr int STEPS = 32;
constexpr int N_WG = 256;
constexpr u32 W_WATCH = PUSH_PER_CU, W_MFMA = W_WATCH + 1, W_LOAD = W_MFMA + 4;
constexpr int N_WAVES = W_LOAD + 8;
constexpr int N_IN = 4096;
constexpr int N_PART = 2048;
constexpr int N_SRC = 2048;
constexpr int ID_PART = 2 * N_IN;
constexpr int ID_OWN = ID_PART + N_PART;
constexpr int N_BLK = ID_OWN + N_SRC;
constexpr int PUSH_W = PUSHERS;

constexpr u32 OFF_CNT = 0;
constexpr u32 OFF_TMO = 32768;
constexpr u32 OFF_TKT = 33024;
constexpr u32 OFF_GATE = 33280;
constexpr u32 OFF_FLAG = 65536;
constexpr u32 OFF_DBG = 262144;
#define BSTRIDE 4352
constexpr u32 OFF_DATA = 1u << 20;
constexpr u32 BSTR = BSTRIDE;
constexpr u32 WS_NEED = OFF_DATA + N_BLK * BSTR;
constexpr u64 DEADLINE_TICKS = 40000000ull;
constexpr int SYS = 17;
constexpr u32 OOB = 0x80000000u;

constexpr u32 H_VALID = 0x100, H_LONG = 0x200, H_RAW = 0x400;

constexpr u32 SLOT = 32768;
constexpr u32 LDS_CTL = 4 * SLOT;
constexpr u32 LDS_TAB = LDS_CTL + 256;
constexpr u32 LDS_PACK = LDS_TAB + 1024;
constexpr u32 LDS_RED = LDS_PACK + 8192;
constexpr u32 LDS_SIZE = LDS_RED + 8 * 2048;

struct Params {
    const char* in[2];
    char* out;
    char* ws[NRANKS];
    u32 rank;
    u32 pad;
    u32 order[32];
};

__device__ __forceinline__ rsrc_t make_rsrc(const void* p, u32 bytes) {
    return __builtin_amdgcn_make_buffer_rsrc((void*)p, 0, bytes, 0x00020000);
}
__device__ __forceinline__ u32 uni(u32 v) { return __builtin_amdgcn_readfirstlane(v); }
__device__ __forceinline__ u32 rotr(u32 v, u32 s) { return __builtin_amdgcn_alignbit(v, v, s); }
#define SB() __builtin_amdgcn_sched_barrier(0)
#define FENCE() asm volatile("" ::: "memory")

#define STAMP_AT(slot) do {} while (0)

struct Wire { u32x4 a, b, c, d; u32 hdr; };

__device__ __forceinline__ u32 wave_sum_u32(u32 v) {
    v += (u32)__builtin_amdgcn_update_dpp(0, (int)v, 0x111, 0xf, 0xf, true);
    v += (u32)__builtin_amdgcn_update_dpp(0, (int)v, 0x112, 0xf, 0xf, true);
    v += (u32)__builtin_amdgcn_update_dpp(0, (int)v, 0x114, 0xf, 0xf, true);
    v += (u32)__builtin_amdgcn_update_dpp(0, (int)v, 0x118, 0xf, 0xf, true);
    return __builtin_amdgcn_readlane(v, 15) + __builtin_amdgcn_readlane(v, 31) +
           __builtin_amdgcn_readlane(v, 47) + __builtin_amdgcn_readlane(v, 63);
}

template <bool PLANAR>
__device__ __forceinline__ void pack_block(const u32x4 (&v)[4], Wire& o, LAS char* scr, u32 lane) {
    *(LAS u16*)(scr + 768u + lane * 2u) = 0;
    u32 hv[8], sum = 0;
#pragma unroll
    for (int k = 0; k < 8; ++k) {
        hv[k] = __builtin_amdgcn_perm(v[k >> 1][2 * (k & 1) + 1], v[k >> 1][2 * (k & 1)], 0x07050301u);
        sum = __builtin_amdgcn_sad_u8(hv[k] & 0x7F7F7F7Fu, 0u, sum);
    }
    const int bi = ((int)wave_sum_u32(sum) - 3790) >> 10;
    const u32 base = bi < 0 ? 0u : min((u32)bi, 248u);
    const u16x2 b7 = __builtin_bit_cast(u16x2, base * 0x00800080u);
    u32 lo[8], f[8], px[8];
#pragma unroll
    for (int k = 0; k < 8; ++k) {
        const u32 v0 = v[k >> 1][2 * (k & 1)], v1 = v[k >> 1][2 * (k & 1) + 1];
        const u32 d0 = __builtin_bit_cast(u32, __builtin_bit_cast(u16x2, v0) - b7);
        const u32 d1 = __builtin_bit_cast(u32, __builtin_bit_cast(u16x2, v1) - b7);
        lo[k] = __builtin_amdgcn_perm(d1, d0, 0x06040200u);
        const u32 hd = __builtin_amdgcn_perm(d1, d0, 0x07050301u);
        f[k] = (hd & 0x03030303u) | (hv[k] & 0x80808080u);
        px[k] = (((hv[k] | 0x80808080u) - (hd & 0x7C7C7C7Cu)) ^ hv[k]) & 0x7C7C7C7Cu;
    }
    lds_u32p fp = (lds_u32p)(scr + lane * 12u);
    fp[0] = f[0] | rotr(f[1], 29) | ((f[6] << 5) & 0x60606060u);
    fp[1] = f[2] | rotr(f[3], 29) | ((f[7] << 5) & 0x60606060u);
    fp[2] = f[4] | rotr(f[5], 29) | ((f[6] >> 1) & 0x40404040u) | ((f[7] >> 2) & 0x20202020u);
    u32 todo = 0;
#pragma unroll
    for (int k = 0; k < 8; ++k) todo |= ((px[k] + 0x7C7C7C7Cu) >> (7 - k)) & (0x01010101u << k);
    u32 cnt = 0;
    u64 m = __ballot(todo != 0u);
    while (m != 0ull) {
        const u32 i = (u32)__builtin_ctz(todo | 0x80000000u);
        const u32 m0 = (u32)__builtin_amdgcn_sbfe((int)i, 0, 1), m1 = (u32)__builtin_amdgcn_sbfe((int)i, 1, 1),
                  m2 = (u32)__builtin_amdgcn_sbfe((int)i, 2, 1);
        auto sel = [](u32 c, u32 a, u32 b) { return (c & a) | (~c & b); };
        const u32 pk = sel(m2, sel(m1, sel(m0, px[7], px[6]), sel(m0, px[5], px[4])),
                           sel(m1, sel(m0, px[3], px[2]), sel(m0, px[1], px[0])));
        const u32 pos = PLANAR ? ((i << 8) & 0x700u) | ((i << 3) & 0x80u) | (lane * 2u) | ((i >> 3) & 1u)
                               : ((i << 8) & 0x600u) | (lane * 8u) | (m0 & 4u) | (i >> 3);
        const u32 ent = (((pk >> (i & 24u)) << 9) & 0xF800u) | pos;
        u32 slot = __builtin_amdgcn_mbcnt_hi((u32)(m >> 32), __builtin_amdgcn_mbcnt_lo((u32)m, cnt));
        slot = todo != 0u ? min(slot, 64u) : 64u;
        *(LAS u16*)(scr + 768u + slot * 2u) = (u16)ent;
        todo &= todo - 1u;
        cnt += (u32)__builtin_popcountll(m);
        m = cnt > 64u ? 0ull : __ballot(todo != 0u);
    }
    FENCE();
    o.a = u32x4{lo[0], lo[1], lo[2], lo[3]};
    o.b = u32x4{lo[4], lo[5], lo[6], lo[7]};
    o.c = *(LAS u32x4*)(scr + lane * 16u);
    o.d = v[3];
    FENCE();
    o.hdr = base | H_VALID | (cnt > 32u ? H_LONG : 0u);
    if (__builtin_expect(cnt > 64u, 0)) {
        o.a = v[0]; o.b = v[1]; o.c = v[2];
        o.hdr = H_VALID | H_RAW;
    }
}

__device__ __forceinline__ void store_main(rsrc_t dst, u32 id, const Wire& p, u32 lane) {
    const u32 vo = OFF_DATA + id * BSTR + lane * 16u;
    const u32 nc = (p.hdr & H_RAW) ? 64u : (p.hdr & H_LONG) ? 56u : 52u;
    __builtin_amdgcn_raw_buffer_store_b128(p.a, dst, vo, 0, SYS);
    __builtin_amdgcn_raw_buffer_store_b128(p.b, dst, vo + 1024u, 0, SYS);
    __builtin_amdgcn_raw_buffer_store_b128(p.c, dst, lane < nc ? vo + 2048u : OOB, 0, SYS);
}
__device__ __forceinline__ void store_tail(rsrc_t dst, u32 id, const Wire& p, u32 lane) {
    __builtin_amdgcn_raw_buffer_store_b128(p.d, dst, OFF_DATA + id * BSTR + lane * 16u + 3072u, 0, SYS);
}

__device__ __forceinline__ u32 field(const u32x4& n, int k, u32 m80) {
    if (k < 6) return ((k & 1) ? rotr(n[k >> 1], 3) : n[k >> 1]) & 0x83838383u;
    u32 sg = n[2];
    asm("v_add_u32 %0, %1, %1" : "=v"(sg) : "v"(sg));
    if (k == 7) asm("v_add_u32 %0, %1, %1" : "=v"(sg) : "v"(sg));
    return __builtin_amdgcn_bitop3_b32(m80, sg, n[k - 6] >> 5, 0xCA) & 0x83838383u;
}
__device__ __forceinline__ u32x4 unpack_piece(const u32x4& l, const u32x4& n, u32 b7, u32 m80, int j) {
    const u32 h0 = field(n, 2 * j, m80), h1 = field(n, 2 * j + 1, m80);
    const u32 a = l[2 * (j & 1)], b = l[2 * (j & 1) + 1];
    return u32x4{__builtin_amdgcn_perm(h0, a, 0x05010400u) + b7, __builtin_amdgcn_perm(h0, a, 0x07030602u) + b7,
                 __builtin_amdgcn_perm(h1, b, 0x05010400u) + b7, __builtin_amdgcn_perm(h1, b, 0x07030602u) + b7};
}
__device__ __forceinline__ u32 in_vgpr(u32 x) { asm("" : "+v"(x)); return x; }

template <int OFF>
__device__ __forceinline__ void put_rows(u32 m0v, const u32x4& v) {
    asm volatile("s_mov_b32 m0, %4\n\ts_nop 0\n\t"
                 "ds_write_addtid_b32 %0 offset:%5\n\tds_write_addtid_b32 %1 offset:%6\n\t"
                 "ds_write_addtid_b32 %2 offset:%7\n\tds_write_addtid_b32 %3 offset:%8"
                 :: "v"(v[0]), "v"(v[1]), "v"(v[2]), "v"(v[3]), "s"(m0v), "i"(OFF), "i"(OFF + 256), "i"(OFF + 512),
                    "i"(OFF + 768) : "memory");
}

__device__ __forceinline__ void publish(char* dst_ws, u32 id, u32 call, u32 hdr) {
    __hip_atomic_store((GAS u64*)(dst_ws + OFF_FLAG + id * 8u), ((u64)call << 32) | hdr,
                       __ATOMIC_RELAXED, __HIP_MEMORY_SCOPE_SYSTEM);
}

__device__ __forceinline__ bool give_up(char* myws, u32 call, u64 t0) {
    u64 dt = __builtin_amdgcn_s_memrealtime() - t0;
    if (dt < 100000ull) return false;
    GAS u32* tmo = (GAS u32*)(myws + OFF_TMO);
    u32 t = uni(__hip_atomic_load(tmo, __ATOMIC_RELAXED, __HIP_MEMORY_SCOPE_AGENT));
    if (t == call || dt > DEADLINE_TICKS) {
        __hip_atomic_store(tmo, call, __ATOMIC_RELAXED, __HIP_MEMORY_SCOPE_AGENT);
        asm volatile("s_waitcnt vmcnt(0)" ::: "memory");
        return true;
    }
    return false;
}

__device__ __forceinline__ float silu(float y) {
    float c = __builtin_amdgcn_fmed3f(y, -60.0f, 60.0f);
    float e = __builtin_amdgcn_exp2f(c * -1.4426950408889634f);
    return y * __builtin_amdgcn_rcpf(1.0f + e);
}
__device__ __forceinline__ u32 cvt2(float a, float b) {
    f32x2 f = {a, b};
    return __builtin_bit_cast(u32, __builtin_convertvector(f, bf16x2));
}

struct Src { u32 off, stride, id, own, d0, d1, isx; char *w0, *w1; const char* in; };

__device__ __forceinline__ Src decode_src(const Params& P, u32 e, u32 rank) {
    u32 gi = (e >> 4) & 127u, sub = e & 15u;
    u32 code = (P.order[gi >> 2] >> ((gi & 3u) * 8u)) & 0xFFu;
    u32 t = code & 31u, type = (code >> 5) & 3u;
    u32 kh = sub & 1u, half = (sub >> 1) & 1u, q = sub >> 2;
    Src s;
    if (type < 2u) {
        s.off = (2048u * kh + 64u * t + 16u * q) * 1024u + (half * 2u + type) * 256u;
        s.stride = 1024u;
        s.id = ((8u + rank) * 8u + type * 4u + q) * 32u + t;
        s.own = ID_OWN + 1024u + ((half | (kh << 1)) * 8u + type * 4u + q) * 32u + t;
        s.d0 = (half << 1) | (kh << 2) | (q & 1u);
        s.d1 = s.d0 ^ 1u;
        s.isx = 0;
    } else {
        u32 rg = (type - 2u) * 4u + q;
        s.off = (256u * half + 32u * rg) * 8192u + (2048u * kh + 64u * t) * 2u;
        s.stride = 16384u;
        s.id = (rank * 8u + rg) * 32u + t;
        s.own = ID_OWN + ((half | (kh << 1)) * 8u + rg) * 32u + t;
        s.d0 = half | (kh << 2) | ((q & 1u) << 1);
        s.d1 = s.d0 ^ 2u;
        s.isx = 1;
    }
    if (s.d0 == rank) { s.d0 = s.d1; s.d1 = rank; }
    s.w0 = P.ws[s.d0 & 7u];
    s.w1 = P.ws[s.d1 & 7u];
    s.in = P.in[s.isx & 1u];
    return s;
}

__device__ __forceinline__ void load_src(const Src& s, bool on, u32 lane, u32x4 (&v)[4]) {
    rsrc_t rs = make_rsrc(s.in, on ? 4u << 20 : 0u);
    const u32 lo = s.isx ? (lane >> 5) * 8192u + (lane & 31u) * 4u : lane * 4u;
#pragma unroll
    for (int r = 0; r < 16; ++r)
        v[r >> 2][r & 3] = __builtin_amdgcn_raw_buffer_load_b32(rs, lo ^ (s.isx ? (r & 7) * 16u : (r & 3) * 64u),
                                                                s.off + r * s.stride, 0);
}

__device__ __forceinline__ u32 prog_read(lds_u32p p) {
    return __hip_atomic_load(p, __ATOMIC_RELAXED, __HIP_MEMORY_SCOPE_WORKGROUP);
}
__device__ __forceinline__ void prog_wait(lds_u32p p, u32 seen, u32 need) {
    if (__builtin_expect(!__all(seen >= need), 0)) {
        while (!__all(prog_read(p) >= need)) __builtin_amdgcn_s_sleep(SPIN);
    }
    FENCE();
}
__device__ __forceinline__ void prog_set(lds_u32p p, u32 v) {
    FENCE();
    __hip_atomic_store(p, v, __ATOMIC_RELAXED, __HIP_MEMORY_SCOPE_WORKGROUP);
    FENCE();
}

struct Set { u32x4 l0, l1, n; u32 x; };
template <int N> struct IC { static constexpr int value = N; };

__global__ void __launch_bounds__(N_WAVES * 64, 4) ag_gemm_silu(Params P) {
    __shared__ __attribute__((aligned(16))) char smem[LDS_SIZE];
    LAS char* lds = (LAS char*)smem;
    const u32 lane = threadIdx.x & 63u;
    const u32 wave = uni(threadIdx.x >> 6);
    const u32 gwave = blockIdx.x * 16u + wave;
    const u32 rank = P.rank & 7u;
    char* myws = P.ws[rank];
    const rsrc_t mine = make_rsrc(myws, WS_NEED);
    STAMP_AT(0);
    lds_u32p prog = (lds_u32p)(lds + LDS_CTL);
    lds_u32p tab = (lds_u32p)(lds + LDS_TAB);
    const u32 xcd = blockIdx.x & 7u, idx = blockIdx.x >> 3;
    const u32 tm = (xcd >> 1) * 4u + (idx & 3u), tn = (idx >> 2) * 2u + (((idx >> 2) + xcd) & 1u);

    const bool diag = (tm >> 1) == (tn >> 1);
    const u32 cell = diag ? tm >> 1 : rank, srcx = diag ? rank : tm >> 1, owner = diag ? rank : tn >> 1;
    auto first_id = [&](u32 par, u32 isB) {
        const u32 own = ID_OWN + isB * 1024u +
                        ((((cell >> isB) & 1u) | ((cell >> 2) << 1)) * 2u + ((isB ? tn : tm) & 1u)) * 128u;
        return (isB ? owner : srcx) == rank ? own : par * N_IN + (isB ? 16u + tn : tm) * 128u;
    };

    GAS u32* cnt = (GAS u32*)(myws + OFF_CNT + gwave * 8u);

    if (wave < W_WATCH) {
        const u32 px = tm >> 1, po = tn >> 1;
        u32 e;
        if (owner == rank) {
            e = (diag ? px : 8u + px - (px > rank ? 1u : 0u)) * 4u + (tm & 1u) * 2u + (tn & 1u);
        } else {
            const u32 lo = min(px, rank), hi = max(px, rank);
            const u32 o = po - (po > lo ? 1u : 0u) - (po > hi && hi != lo ? 1u : 0u);
            e = (tm & 1u) && o < 6u ? 60u + (o * 8u + px) * 2u + (tn & 1u) : 4096u;
        }
        if (e >= (u32)PUSH_W) { __builtin_amdgcn_s_barrier(); return; }
        __builtin_amdgcn_s_setprio(PUSH_PRIO);
        u32x4 v[4], u[4];
        Src sa = decode_src(P, e, rank);
        load_src(sa, true, lane, u);
        const u32 cv = __hip_atomic_load(cnt, __ATOMIC_RELAXED, __HIP_MEMORY_SCOPE_AGENT);
        Src sb = decode_src(P, e + PUSH_W, rank);
        load_src(sb, true, lane, v);
        __builtin_amdgcn_s_barrier();
        const u32 call = uni(cv) + 1u;
        const u32 idp = (call & 1u) * N_IN;
        const u32 tkoff = lane ? OOB : OFF_TKT;
        const u32 tbase = (call - 1u) * (u32)(N_SRC - 2 * PUSH_W) - 3u * (u32)PUSH_W;
        u32 tk = e + 2u * (u32)PUSH_W + tbase;
        Wire p;
        pack_block<true>(u, p, lds + LDS_PACK + wave * 1024u, lane);
        asm volatile("" : "+v"(v[0]), "+v"(v[1]), "+v"(v[2]), "+v"(v[3]));
        STAMP_AT(4);
        char* fptr = myws + OFF_CNT + gwave * 8u - OFF_FLAG;
        u32 fn = 1u, fh = call;
        bool hasb = true, go;
#define TQ(x)
        do {
            TQ(tq0);
            if (lane < fn) publish(fptr, 0u, call, fh);
            FENCE();
            const rsrc_t r0 = make_rsrc(sa.w0, WS_NEED), r1 = make_rsrc(sa.w1, sa.d1 != rank ? WS_NEED : 0u);
            const u32 id = sa.id + idp;
            store_main(r0, id, p, lane);
            store_main(r1, id, p, lane);
            store_main(mine, sa.own, p, lane);
            if (__builtin_expect(p.hdr & H_RAW, 0)) {
                store_tail(r0, id, p, lane);
                store_tail(r1, id, p, lane);
                store_tail(mine, sa.own, p, lane);
            }
            fptr = (lane == 0u ? sa.w0 : lane == 1u ? myws : sa.w1) + (lane == 1u ? sa.own : id) * 8u;
            fn = sa.d1 != rank ? 3u : 2u;
            fh = p.hdr;
            e = uni(tk) - tbase;
            const bool hasc = hasb && e < (u32)N_SRC;
            const Src sc = decode_src(P, e, rank);
            load_src(sc, hasc, lane, u);
            tk = (u32)__builtin_amdgcn_raw_ptr_buffer_atomic_add_i32(1, mine, hasc ? tkoff : OOB, 0, 0);
            if (hasb && e == (u32)GATE_AT)
                __builtin_amdgcn_raw_buffer_store_b32(call, mine, lane ? OOB : OFF_GATE, 0, 16);
            TQ(tq1);
            if (hasb) pack_block<true>(v, p, lds + LDS_PACK + wave * 1024u, lane);
            TQ(tq2);
            asm volatile("s_waitcnt vmcnt(0)"
                         : "+v"(fptr), "+v"(tk), "+v"(u[0]), "+v"(u[1]), "+v"(u[2]), "+v"(u[3]) :: "memory");
            TQ(tq3);
#pragma unroll
            for (int j = 0; j < 4; ++j) v[j] = u[j];
            sa = sb; sb = sc;
            go = hasb; hasb = hasc;
        } while (go);
        if (lane < fn) publish(fptr, 0u, call, fh);
        STAMP_AT(1);
        return;
    }

    if (wave == W_MFMA) prog[lane] = 0u;
    if (wave >= W_LOAD && wave < W_LOAD + 4u) tab[(wave - W_LOAD) * 64u + lane] = 0u;
    const u32 cv = __hip_atomic_load(cnt, __ATOMIC_RELAXED, __HIP_MEMORY_SCOPE_AGENT);
    asm volatile("s_waitcnt lgkmcnt(0)" ::: "memory");
    __builtin_amdgcn_s_barrier();
    const u32 call = uni(cv) + 1u;
    if (lane == 0) __hip_atomic_store(cnt, call, __ATOMIC_RELAXED, __HIP_MEMORY_SCOPE_AGENT);
    const u32 par = call & 1u;

    if (wave == W_WATCH) {
        GAS u64* fa = (GAS u64*)(myws + OFF_FLAG + (first_id(par, 0u) + lane) * 8u);
        GAS u64* fb = (GAS u64*)(myws + OFF_FLAG + (first_id(par, 1u) + lane) * 8u);
        const u64 t0 = __builtin_amdgcn_s_memrealtime();
        for (;;) {
            u64 f[4];
            f[0] = __hip_atomic_load(fa, __ATOMIC_RELAXED, __HIP_MEMORY_SCOPE_SYSTEM);
            f[1] = __hip_atomic_load(fa + 64, __ATOMIC_RELAXED, __HIP_MEMORY_SCOPE_SYSTEM);
            f[2] = __hip_atomic_load(fb, __ATOMIC_RELAXED, __HIP_MEMORY_SCOPE_SYSTEM);
            f[3] = __hip_atomic_load(fb + 64, __ATOMIC_RELAXED, __HIP_MEMORY_SCOPE_SYSTEM);
            bool all = true;
#pragma unroll
            for (int i = 0; i < 4; ++i) {
                const bool ok = (u32)(f[i] >> 32) >= call;
                all = all && ok;
                __hip_atomic_store(tab + i * 64 + lane, ok ? (u32)f[i] : 0u, __ATOMIC_RELEASE,
                                   __HIP_MEMORY_SCOPE_WORKGROUP);
            }
            if (__all(all)) break;
            if (give_up(myws, call, t0)) {
#pragma unroll
                for (int i = 0; i < 4; ++i)
                    __hip_atomic_store(tab + i * 64 + lane, H_VALID, __ATOMIC_RELEASE, __HIP_MEMORY_SCOPE_WORKGROUP);
                break;
            }
            if (__any(!all && (lane & 31u) < (u32)POLL_FAST)) __builtin_amdgcn_s_sleep(POLL_SLEEP);
            else __builtin_amdgcn_s_sleep(1);
        }
        STAMP_AT(1);
        while (uni(__hip_atomic_load((GAS u32*)(myws + OFF_GATE), __ATOMIC_RELAXED, __HIP_MEMORY_SCOPE_AGENT)) < call) {
            if (give_up(myws, call, t0)) break;
            __builtin_amdgcn_s_sleep(4);
        }
        prog_set(prog + W_WATCH, 64u);
        STAMP_AT(2);
        GAS u64* fp = (GAS u64*)(myws + OFF_FLAG + (ID_PART + blockIdx.x * 8u + (lane & 7u)) * 8u);
        for (;;) {
            const u64 f = __hip_atomic_load(fp, __ATOMIC_RELAXED, __HIP_MEMORY_SCOPE_SYSTEM);
            const bool ok = (u32)(f >> 32) >= call;
            __hip_atomic_store(prog + 32u + (lane & 7u), ok ? (u32)f : 0u, __ATOMIC_RELEASE, __HIP_MEMORY_SCOPE_WORKGROUP);
            if (__all(ok)) break;
            if (give_up(myws, call, t0)) {
                __hip_atomic_store(prog + 32u + (lane & 7u), H_VALID, __ATOMIC_RELEASE, __HIP_MEMORY_SCOPE_WORKGROUP);
                break;
            }
            __builtin_amdgcn_s_sleep(RED_SLEEP);
        }
        return;
    }

    if (wave >= W_LOAD) {
        const u32 lw8 = wave - W_LOAD, lw = lw8 & 3u, isB = lw8 >> 2;
        const u32 id0 = first_id(par, isB) + lw * 32u;
        const u32 blk = lw8 * 4096u;
        const u32 m80 = in_vgpr(0x80808080u);
        const u32 doff = OFF_DATA + id0 * BSTR + lane * 16u;
        const u32 noff = OFF_DATA + id0 * BSTR + lane * 12u;
        const u32 xlong = OFF_DATA + id0 * BSTR + lane * 2u, xshort = lane < 32u ? xlong : OOB;
        lds_u32p row = tab + lw8 * 32u + (lane & 31u);
        lds_u32p mprog = prog + W_MFMA + (lane & 3u);
        Set R[4];
        u32 H[4] = {0, 0, 0, 0};
#pragma unroll
        for (int q = 0; q < 4; ++q)
            asm volatile("" : "=v"(R[q].l0), "=v"(R[q].l1), "=v"(R[q].n), "=v"(R[q].x));
        u32 rel = 0, hv = 0;
        auto fetch = [&](int d, u32 s, u32 avail, Set& q, u32& h) {
            u32 tv, tw, ts;
            asm volatile("s_cmp_lg_u32 %[rel], %[d]\n\ts_cbranch_scc1 .Lskip%=\n\t"
                         "s_cmp_le_u32 %[av], %[d]\n\ts_cbranch_scc1 .Lskip%=\n\t"
                         "s_add_u32 %[ts], %[s], %[d]\n\t"
                         "v_readlane_b32 %[h], %[hv], %[ts]\n\t"
                         "s_add_u32 %[rel], %[rel], 1\n\t"
                         "s_mul_i32 %[ts], %[ts], %[bs]\n\t"
                         "s_bitcmp1_b32 %[h], 10\n\ts_cselect_b64 vcc, -1, 0\n\ts_nop 1\n\t"
                         "v_cndmask_b32 %[tv], %[noff], %[doff], vcc\n\t"
                         "s_bitcmp1_b32 %[h], 9\n\ts_cselect_b64 vcc, -1, 0\n\ts_nop 1\n\t"
                         "v_cndmask_b32 %[tw], %[xs], %[xl], vcc\n\t"
                         "buffer_load_dwordx4 %[n], %[tv], %[rs], %[ts] offen offset:2048 sc0 sc1\n\t"
                         "buffer_load_dwordx4 %[l0], %[doff], %[rs], %[ts] offen sc0 sc1\n\t"
                         "buffer_load_dwordx4 %[l1], %[doff], %[rs], %[ts] offen offset:1024 sc0 sc1\n\t"
                         "buffer_load_ushort %[x], %[tw], %[rs], %[ts] offen offset:2816 sc0 sc1\n"
                         ".Lskip%=:"
                         : [x] "+v"(q.x), [n] "+v"(q.n), [l0] "+v"(q.l0), [l1] "+v"(q.l1), [h] "+s"(h),
                           [rel] "+s"(rel), [tv] "=&v"(tv), [tw] "=&v"(tw), [ts] "=&s"(ts)
                         : [d] "i"(d), [av] "s"(avail), [s] "s"(s), [hv] "v"(hv), [noff] "v"(noff),
                           [doff] "v"(doff), [xs] "v"(xshort), [xl] "v"(xlong), [rs] "s"(mine), [bs] "i"(BSTR)
                         : "vcc", "scc", "memory");
        };
        auto landed = [&](Set& q) {
            asm volatile("s_cmp_lt_u32 %[rel], 4\n\ts_cbranch_scc1 .La%=\n\ts_waitcnt vmcnt(12)\n\ts_branch .Lz%=\n"
                         ".La%=:\n\ts_cmp_lt_u32 %[rel], 3\n\ts_cbranch_scc1 .Lb%=\n\ts_waitcnt vmcnt(8)\n\ts_branch .Lz%=\n"
                         ".Lb%=:\n\ts_cmp_lt_u32 %[rel], 2\n\ts_cbranch_scc1 .Lc%=\n\ts_waitcnt vmcnt(4)\n\ts_branch .Lz%=\n"
                         ".Lc%=:\n\ts_waitcnt vmcnt(0)\n.Lz%=:"
                         : "+v"(q.x), "+v"(q.n), "+v"(q.l0), "+v"(q.l1) : [rel] "s"(rel) : "scc", "memory");
        };
        auto arrived = [&]() {
            const u32 ok = (u32)__ballot(hv != 0u);
            return ok == 0xFFFFFFFFu ? 32u : (u32)__builtin_ctz(~ok);
        };
        u32 lim = 0;
        for (u32 s0 = 0; s0 < (u32)STEPS; s0 += 4) {
#pragma unroll
            for (int q = 0; q < 4; ++q) {
                const u32 s = s0 + q;
                Set& c = R[q];
                LAS char* dst = lds + q * SLOT;
                const u32 hn = prog_read(row), mseen = prog_read(mprog);
                const u32 lim0 = lim;
                if (__builtin_expect(s + 3u >= lim0, 0)) {
                    u32 rdy = arrived();
                    while (rdy <= s) {
                        __builtin_amdgcn_s_sleep(SPIN);
                        hv = __hip_atomic_load(row, __ATOMIC_ACQUIRE, __HIP_MEMORY_SCOPE_WORKGROUP);
                        rdy = arrived();
                    }
                    const u32 avail = rdy - s;
#pragma unroll
                    for (int d = 0; d < 4; ++d) fetch(d, s, avail, R[(q + d) & 3], H[(q + d) & 3]);
                    landed(c);
                    rel -= 1u;
                    const u64 xm = (__ballot(hv >= H_LONG) | (1ull << 32)) >> (s + 4u);
                    lim = rel == 3u ? min(rdy, s + 4u + (u32)__builtin_ctzll(xm | (1ull << 40))) : 0u;
                }
                {
                    Set& nx = R[(q + 3) & 3];
                    u32 ts;
                    asm volatile("s_cmp_ge_u32 %[s3], %[lim]\n\ts_cbranch_scc1 .Lslow%=\n\t"
                                 "s_mul_i32 %[ts], %[s3], %[bs]\n\t"
                                 "v_readlane_b32 %[h], %[hv], %[s3]\n\t"
                                 "buffer_load_dwordx4 %[n], %[noff], %[rs], %[ts] offen offset:2048 sc0 sc1\n\t"
                                 "buffer_load_dwordx4 %[l0], %[doff], %[rs], %[ts] offen sc0 sc1\n\t"
                                 "buffer_load_dwordx4 %[l1], %[doff], %[rs], %[ts] offen offset:1024 sc0 sc1\n\t"
                                 "buffer_load_ushort %[x], %[xs], %[rs], %[ts] offen offset:2816 sc0 sc1\n\t"
                                 "s_waitcnt vmcnt(12)\n.Lslow%=:"
                                 : [n] "+v"(nx.n), [l0] "+v"(nx.l0), [l1] "+v"(nx.l1), [x] "+v"(nx.x), "+v"(c.x),
                                   "+v"(c.n), "+v"(c.l0), "+v"(c.l1), [h] "+s"(H[(q + 3) & 3]), [ts] "=&s"(ts)
                                 : [s3] "s"(s + 3u), [lim] "s"(lim0), [hv] "v"(hv), [doff] "v"(doff), [noff] "v"(noff),
                                   [xs] "v"(xshort), [rs] "s"(mine), [bs] "i"(BSTR)
                                 : "scc", "memory");
                }
                const u32 b7 = in_vgpr((H[q] & 0xFFu) * 0x00800080u);
                auto put = [&](auto jc, const u32x4& t) {
                    constexpr int j = decltype(jc)::value;
                    if (q == 0) put_rows<j * 1024>(blk, t);
                    if (q == 1) put_rows<32768 + j * 1024>(blk, t);
                    if (q == 2) put_rows<28800 + j * 1024>(blk + 36736u, t);
                    if (q == 3) put_rows<61568 + j * 1024>(blk + 36736u, t);
                };
                u32x4 t0 = unpack_piece(c.l0, c.n, b7, m80, 0);
                asm volatile("" : "+v"(t0));
                hv |= hn;
                prog_wait(mprog, mseen, s > 3u ? s - 3u : 0u);
                put(IC<0>{}, t0);
                put(IC<1>{}, unpack_piece(c.l0, c.n, b7, m80, 1));
                put(IC<2>{}, unpack_piece(c.l1, c.n, b7, m80, 2));
                put(IC<3>{}, unpack_piece(c.l1, c.n, b7, m80, 3));
                if (c.x != 0u)
                    __hip_atomic_fetch_xor((lds_u32p)(dst + blk + ((c.x & 0x7FEu) << 1)),
                                           ((c.x >> 1) & 0x7C00u) << ((c.x & 1u) << 4), __ATOMIC_RELAXED,
                                           __HIP_MEMORY_SCOPE_WORKGROUP);
                FENCE();
                if (__builtin_expect(H[q] >= H_RAW, 0)) {
                    put(IC<0>{}, c.l0);
                    put(IC<1>{}, c.l1);
                    put(IC<2>{}, c.n);
                    put(IC<3>{}, __builtin_amdgcn_raw_buffer_load_b128(mine, doff + 3072u, s * BSTR, SYS));
                }
                prog_set(prog + wave, s + 1u);
                SB();
            }
        }
        STAMP_AT(2);
        const u32 unit = blockIdx.x * 4u + (lw8 >> 1), hf = lw8 & 1u;
        const u32 idp = ID_PART + unit * 2u;
        LAS char* pl = lds + LDS_RED + lw8 * 2048u;
        const u32x4 zero = {0u, 0u, 0u, 0u};
        *(LAS u32x4*)(pl + lane * 16u) = zero;
        *(LAS u32x4*)(pl + lane * 16u + 1024u) = zero;
        u32 hp;
        for (;;) {
            hp = __hip_atomic_load(prog + 32u + (lw8 & 6u) + (lane & 1u), __ATOMIC_ACQUIRE, __HIP_MEMORY_SCOPE_WORKGROUP);
            if (__all(hp != 0u)) break;
            __builtin_amdgcn_s_sleep(1);
        }
        FENCE();
        STAMP_AT(3);
        u32 hk[2];
        u32x4 l[2], n[2];
        u32 x[2];
#pragma unroll
        for (int k = 0; k < 2; ++k) {
            hk[k] = __builtin_amdgcn_readlane(hp, k);
            const bool raw = hk[k] & H_RAW;
            const u32 so = OFF_DATA + (idp + k) * BSTR;
            x[k] = __builtin_amdgcn_raw_buffer_load_b16(mine, lane < ((hk[k] & H_LONG) ? 64u : 32u) ? lane * 2u : OOB,
                                                        so + 2816u, SYS);
            n[k] = __builtin_amdgcn_raw_buffer_load_b128(mine, lane * (raw ? 16u : 12u),
                                                         so + (raw ? hf * 2048u + 1024u : 2048u), SYS);
            l[k] = __builtin_amdgcn_raw_buffer_load_b128(mine, lane * 16u, so + hf * (raw ? 2048u : 1024u), SYS);
        }
        const u32 bad = uni(__hip_atomic_load((GAS u32*)(myws + OFF_TMO), __ATOMIC_RELAXED,
                                              __HIP_MEMORY_SCOPE_AGENT)) == call;
        u32x4 val[2][2];
#pragma unroll
        for (int k = 0; k < 2; ++k) {
            const bool use = x[k] != 0u && ((x[k] >> 10) & 1u) == hf;
            FENCE();
            if (use) *(LAS u16*)(pl + ((x[k] & 0x3FFu) << 1)) = (u16)((x[k] >> 1) & 0x7C00u);
            FENCE();
            const u32x4 p0 = *(LAS u32x4*)(pl + lane * 16u), p1 = *(LAS u32x4*)(pl + lane * 16u + 1024u);
            FENCE();
            if (k == 0 && use) *(LAS u16*)(pl + ((x[k] & 0x3FFu) << 1)) = 0;
            const u32 b7 = in_vgpr((hk[k] & 0xFFu) * 0x00800080u);
            val[k][0] = (hf ? unpack_piece(l[k], n[k], b7, m80, 2) : unpack_piece(l[k], n[k], b7, m80, 0)) ^ p0;
            val[k][1] = (hf ? unpack_piece(l[k], n[k], b7, m80, 3) : unpack_piece(l[k], n[k], b7, m80, 1)) ^ p1;
            if (__builtin_expect(hk[k] & H_RAW, 0)) { val[k][0] = l[k]; val[k][1] = n[k]; }
        }
        const rsrc_t ro = make_rsrc(P.out, 4u << 20);
        const u32 oo = (unit >> 3) * 32u * 1024u + (unit & 7u) * 128u + hf * 64u + (lane & 31u) * 1024u + (lane >> 5) * 16u;
#pragma unroll
        for (int j = 0; j < 2; ++j) {
            u32x4 o;
#pragma unroll
            for (int d = 0; d < 4; ++d) {
                const u32 a = val[0][j][d], b = val[1][j][d];
                float y0 = __builtin_bit_cast(float, a << 16) + __builtin_bit_cast(float, b << 16);
                float y1 = __builtin_bit_cast(float, a & 0xFFFF0000u) + __builtin_bit_cast(float, b & 0xFFFF0000u);
                o[d] = cvt2(silu(y0), silu(y1)) | (bad ? 0x7FC07FC0u : 0u);
            }
            __builtin_amdgcn_raw_buffer_store_b128(o, ro, oo + j * 32u, 0, 0);
        }
        STAMP_AT(4);
        return;
    }

    const u32 mw = wave - W_MFMA, wm = mw >> 1, wn = mw & 1u;
    const u32 rdA = (64u * wm + (lane & 31u)) * 128u + (((lane >> 5) ^ ((lane >> 1) & 7u)) * 16u);
    const u32 rdB = 16384u + (8u * (lane >> 5) + ((lane >> 2) & 3u)) * 256u +
                    (((wn * 8u + ((lane >> 4) & 1u) * 2u + (lane & 1u)) ^ (((lane >> 2) & 3u) << 2)) * 16u) +
                    ((lane >> 1) & 1u) * 8u;
    f32x16 acc[2][2];
#pragma unroll
    for (int a = 0; a < 2; ++a)
#pragma unroll
        for (int b = 0; b < 2; ++b)
#pragma unroll
            for (int r = 0; r < 16; ++r) acc[a][b][r] = 0.f;

    lds_u32p lprog = prog + W_LOAD + (lane & 7u);
    u32 lseen = 0;
    bf16x8 X[2][2], W[2][2];
    auto rd_x = [&](u32 buf, int ks, int mi) {
        return *(LAS bf16x8*)(lds + buf + ((rdA ^ (ks * 32u)) + mi * 4096u));
    };
    auto rd_w = [&](u32 buf, int ks, int ni) {
        u32 a = buf + (rdB ^ (ni * 64u)) + ks * 4096u;
        s16x4 lo = __builtin_amdgcn_ds_read_tr16_b64_v4i16((LAS s16x4*)(lds + a));
        s16x4 hi = __builtin_amdgcn_ds_read_tr16_b64_v4i16((LAS s16x4*)(lds + a + 1024u));
        return __builtin_bit_cast(bf16x8, __builtin_shufflevector(lo, hi, 0, 1, 2, 3, 4, 5, 6, 7));
    };
#define MMA(mi, ni, st) acc[mi][ni] = __builtin_amdgcn_mfma_f32_32x32x16_bf16(W[st][ni], X[st][mi], acc[mi][ni], 0, 0, 0)
    auto step = [&](u32 s, u32 buf, u32 nxt, bool more) {
#pragma unroll
        for (int ks = 0; ks < 4; ++ks) {
            const int st = ks & 1, k2 = (ks + 2) & 3;
            const u32 src = ks < 2 ? buf : nxt;
            const bool ld = ks < 2 || more;
            MMA(0, 0, st); SB();
            if (ks == 1 && more) lseen = prog_read(lprog);
            if (ks == 2 && more) prog_set(prog + wave, s + 1u);
            SB();
            MMA(0, 1, st); SB();
            if (ks == 2 && more) prog_wait(lprog, lseen, s + 2u);
            if (ld) X[st][0] = rd_x(src, k2, 0);
            SB();
            MMA(1, 1, st); SB();
            if (ld) W[st][1] = rd_w(src, k2, 1);
            SB();
            MMA(1, 0, st); SB();
            if (ld) { W[st][0] = rd_w(src, k2, 0); X[st][1] = rd_x(src, k2, 1); }
            SB();
        }
    };
    prog_wait(lprog, lseen, 1u);
#pragma unroll
    for (int st = 0; st < 2; ++st) {
        W[st][0] = rd_w(0, st, 0); X[st][0] = rd_x(0, st, 0);
        W[st][1] = rd_w(0, st, 1); X[st][1] = rd_x(0, st, 1);
    }
    for (u32 s = 0; s < (u32)STEPS - 4u; s += 4) {
        step(s, 0, SLOT, true);
        step(s + 1u, SLOT, 2 * SLOT, true);
        step(s + 2u, 2 * SLOT, 3 * SLOT, true);
        step(s + 3u, 3 * SLOT, 0, true);
    }
    step(28u, 0, SLOT, true);
    step(29u, SLOT, 2 * SLOT, true);
    step(30u, 2 * SLOT, 3 * SLOT, true);
    step(31u, 3 * SLOT, 0, false);
    STAMP_AT(2);

    {
        char* ows = P.ws[owner & 7u];
        const rsrc_t dst = make_rsrc(ows, WS_NEED);
        const u32 row = 512u * srcx + 256u * (cell & 1u) + 128u * (tm & 1u) + 64u * wm;
        const u32 col = 256u * ((cell >> 1) & 1u) + 128u * (tn & 1u) + 64u * wn;
        const u32 id = ID_PART + ((row >> 5) * 8u + (col >> 6)) * 2u + (cell >> 2);
        const u32 bad = __hip_atomic_load((GAS u32*)(myws + OFF_TMO), __ATOMIC_RELAXED, __HIP_MEMORY_SCOPE_AGENT);
        auto pack_half = [&](int h, Wire& p) {
            u32x4 v[4];
#pragma unroll
            for (int j = 0; j < 4; ++j) {
                const f32x16 c = acc[h][j >> 1];
                const int o = 8 * (j & 1);
                v[j] = u32x4{cvt2(c[o], c[o + 1]), cvt2(c[o + 2], c[o + 3]), cvt2(c[o + 4], c[o + 5]), cvt2(c[o + 6], c[o + 7])};
            }
            pack_block<false>(v, p, lds + LDS_PACK + wave * 1024u, lane);
        };
        Wire p0, p1;
        pack_half(0, p0);
        STAMP_AT(4);
        const u32 chain = (tm & 1u) ? (u32)CHAIN1 : (tn & 1u) ? (u32)CHAIN01 : (u32)CHAIN0;
        if (owner != rank) prog_wait(prog + ((mw & (chain - 1u)) ? wave - 1u : W_WATCH), 0u, 64u);
        STAMP_AT(5);
        if (__builtin_expect(p0.hdr & H_RAW, 0)) store_tail(dst, id, p0, lane);
        store_main(dst, id, p0, lane);
        pack_half(1, p1);
        asm volatile("s_waitcnt vmcnt(0)" ::: "memory");
        if (lane == 0u && bad != call) publish(ows, id, call, p0.hdr);
        STAMP_AT(6);
        if (__builtin_expect(p1.hdr & H_RAW, 0)) store_tail(dst, id + 16u, p1, lane);
        store_main(dst, id + 16u, p1, lane);
        asm volatile("s_waitcnt vmcnt(0)" ::: "memory");
        prog_set(prog + wave, 64u);
        if (lane == 0u && bad != call) publish(ows, id + 16u, call, p1.hdr);
    }
    STAMP_AT(3);
}

extern "C" void kernel_launch(int rank, int nranks, ncclComm_t comm, hipStream_t stream,
                              void* const* d_in, const int* in_sizes, int n_in,
                              void* d_out, int out_size, const int* in_dtypes, int out_dtype,
                              void* const* d_out_all, void* const* d_ws_all, size_t ws_bytes) {
    if (nranks != NRANKS || (unsigned)rank >= (unsigned)NRANKS || ws_bytes < (size_t)WS_NEED || n_in != 2 ||
        in_sizes[0] != 512 * 4096 || in_sizes[1] != 4096 * 512 || out_size != 4096 * 512) {
        fprintf(stderr, "kernel_launch: unexpected problem shape\n");
        return;
    }
    Params p;
    p.in[0] = (const char*)d_in[1];
    p.in[1] = (const char*)d_in[0];
    p.out = (char*)d_out;
    for (int r = 0; r < NRANKS; ++r) p.ws[r] = (char*)d_ws_all[r];
    p.rank = (u32)rank;
    p.pad = 0;
    {
        unsigned char ord[128];
        int n = 0, nx[4] = {0, 0, 0, 0};
        const int lag[4] = {0, LAG_R, 0, LAG_B};
        for (int t = 1; t <= STEPS; ++t)
            for (int st = 0; st < 4; ++st)
                while (nx[st] < (t * (STEPS - lag[st]) + STEPS - 1) / STEPS) { ord[n++] = (unsigned char)(st * 32 + nx[st]); ++nx[st]; }
        while (n < 128)
            for (int st = 1; st < 4; st += 2)
                if (nx[st] < STEPS) { ord[n++] = (unsigned char)(st * 32 + nx[st]); ++nx[st]; }
        for (int i = 0; i < 32; ++i)
            p.order[i] = ord[4 * i] | (ord[4 * i + 1] << 8) | (ord[4 * i + 2] << 16) | ((u32)ord[4 * i + 3] << 24);
    }
    hipLaunchKernelGGL(ag_gemm_silu, dim3(N_WG), dim3(N_WAVES * 64), 0, stream, p);
    HIP_CHECK(hipGetLastError());
}
```

```cpp
#include <hip/hip_runtime.h>
#include <rccl/rccl.h>
#include <cstdio>
#include <cstdint>
#include <cstddef>

#define HIP_CHECK(expr) do { hipError_t _e = (expr); if (_e != hipSuccess) { \
    fprintf(stderr, "HIP error %s at %s:%d\n", hipGetErrorString(_e), __FILE__, __LINE__); \
    return; } } while (0)

typedef unsigned u32;
typedef unsigned long long u64;
typedef unsigned short u16;
typedef u32 u32x4 __attribute__((ext_vector_type(4)));
typedef float f32x16 __attribute__((ext_vector_type(16)));
typedef short s16x4 __attribute__((ext_vector_type(4)));
typedef unsigned short u16x2 __attribute__((ext_vector_type(2)));
typedef __bf16 bf16x8 __attribute__((ext_vector_type(8)));
typedef __bf16 bf16x2 __attribute__((ext_vector_type(2)));
typedef float f32x2 __attribute__((ext_vector_type(2)));
#define GAS __attribute__((address_space(1)))
#define LAS __attribute__((address_space(3)))
typedef __amdgpu_buffer_rsrc_t rsrc_t;
typedef LAS u32* lds_u32p;

#define CHAIN0 2
#define CHAIN1 2
#define CHAIN01 CHAIN0
#define POLL_SLEEP 24
#define POLL_FAST 28
#define RED_SLEEP 4
#define SPIN 1
#define PLANAR_IN 0
#define ADD64
#define TF1 99
#define TF0 99
#define TAIL_SLEEP 1
#define N_RELAY 16
#define RELAY_W 32
#define RELAY_SLEEP 8
#define RELAY_SHIFT 16
#define RELAY_LAST 122
#define PUSHERS 120
#define LAG_R 6
#define LAG_B 14
#define PUSH_PER_CU 1
#define PUSH_PRIO 3
#define GATE_AT 1920

constexpr int NRANKS = 8;
constexpr int STEPS = 32;
constexpr int N_WG = 256;
constexpr u32 W_WATCH = PUSH_PER_CU, W_MFMA = W_WATCH + 1, W_LOAD = W_MFMA + 4;
constexpr int N_WAVES = W_LOAD + 8;
constexpr int N_IN = 4096;
constexpr int N_PART = 2048;
constexpr int N_SRC = 2048;
constexpr int ID_PART = 2 * N_IN;
constexpr int ID_OWN = ID_PART + N_PART;
constexpr int N_BLK = ID_OWN + N_SRC;
constexpr int PUSH_W = PUSHERS;

constexpr u32 OFF_CNT = 0;
constexpr u32 OFF_TMO = 32768;
constexpr u32 OFF_TKT = 33024;
constexpr u32 OFF_GATE = 33280;
constexpr u32 OFF_FLAG = 65536;
constexpr u32 OFF_DBG = 262144;
#define BSTRIDE 4352
constexpr u32 OFF_DATA = 1u << 20;
constexpr u32 BSTR = BSTRIDE;
constexpr u32 WS_NEED = OFF_DATA + N_BLK * BSTR;
constexpr u64 DEADLINE_TICKS = 40000000ull;
constexpr int SYS = 17;
constexpr u32 OOB = 0x80000000u;

constexpr u32 H_VALID = 0x100, H_LONG = 0x200, H_RAW = 0x400;

constexpr u32 SLOT = 32768;
constexpr u32 LDS_CTL = 4 * SLOT;
constexpr u32 LDS_TAB = LDS_CTL + 256;
constexpr u32 LDS_PACK = LDS_TAB + 1024;
constexpr u32 LDS_RED = LDS_PACK + 8192;
constexpr u32 LDS_SIZE = LDS_RED + 8 * 2048;

struct Params {
    const char* in[2];
    char* out;
    char* ws[NRANKS];
    u32 rank;
    u32 pad;
    u32 order[32];
    u32 relay[8];
    u32 role[64];
    char* myws;
};

__device__ __forceinline__ rsrc_t make_rsrc(const void* p, u32 bytes) {
    return __builtin_amdgcn_make_buffer_rsrc((void*)p, 0, bytes, 0x00020000);
}
__device__ __forceinline__ u32 uni(u32 v) { return __builtin_amdgcn_readfirstlane(v); }
__device__ __forceinline__ u32 rotr(u32 v, u32 s) { return __builtin_amdgcn_alignbit(v, v, s); }
#define SB() __builtin_amdgcn_sched_barrier(0)
#define FENCE() asm volatile("" ::: "memory")

#define STAMP_AT(slot) do {} while (0)

struct Wire { u32x4 a, b, c, d; u32 hdr; };

__device__ __forceinline__ u32 wave_sum_u32(u32 v) {
    v += (u32)__builtin_amdgcn_update_dpp(0, (int)v, 0x111, 0xf, 0xf, true);
    v += (u32)__builtin_amdgcn_update_dpp(0, (int)v, 0x112, 0xf, 0xf, true);
    v += (u32)__builtin_amdgcn_update_dpp(0, (int)v, 0x114, 0xf, 0xf, true);
    v += (u32)__builtin_amdgcn_update_dpp(0, (int)v, 0x118, 0xf, 0xf, true);
    return __builtin_amdgcn_readlane(v, 15) + __builtin_amdgcn_readlane(v, 31) +
           __builtin_amdgcn_readlane(v, 47) + __builtin_amdgcn_readlane(v, 63);
}

__device__ __forceinline__ u32 in_vgpr(u32 x) { asm("" : "+v"(x)); return x; }
__device__ __forceinline__ u32 bsel(u32 m, u32 a, u32 b) {
    return __builtin_amdgcn_bitop3_b32(m, a, b, 0xCA);
}

template <bool PLANAR>
__device__ __forceinline__ void pack_block(const u32x4 (&v)[4], Wire& o, LAS char* scr, u32 lane) {
    *(LAS u16*)(scr + 768u + lane * 2u) = 0;
    const u32 m80 = in_vgpr(0x80808080u), m7c = in_vgpr(0x7C7C7C7Cu);
    u32 hv[8], sum = 0;
#pragma unroll
    for (int k = 0; k < 8; ++k) {
        hv[k] = __builtin_amdgcn_perm(v[k >> 1][2 * (k & 1) + 1], v[k >> 1][2 * (k & 1)], 0x07050301u);
        if (k & 1) sum = __builtin_amdgcn_sad_u8(hv[k] & 0x7F7F7F7Fu, 0u, sum);
    }
    const u32 tot = wave_sum_u32(sum);
    const u32 base = min((max(tot, 1895u) - 1895u) >> 9, 248u);
    const u16x2 b7 = __builtin_bit_cast(u16x2, in_vgpr(base * 0x00800080u));
    u32 lo[8], hd[8], px[8], todo = 0;
#pragma unroll
    for (int k = 0; k < 8; ++k) {
        const u32 v0 = v[k >> 1][2 * (k & 1)], v1 = v[k >> 1][2 * (k & 1) + 1];
        const u32 d0 = __builtin_bit_cast(u32, __builtin_bit_cast(u16x2, v0) - b7);
        const u32 d1 = __builtin_bit_cast(u32, __builtin_bit_cast(u16x2, v1) - b7);
        lo[k] = __builtin_amdgcn_perm(d1, d0, 0x06040200u);
        hd[k] = __builtin_amdgcn_perm(d1, d0, 0x07050301u);
        const u32 q = hd[k] & m7c;
        px[k] = ((hv[k] | m80) - q) ^ hv[k];
        todo = bsel(m80, q + m7c, todo >> 1);
    }
    const u32 m83 = in_vgpr(0x83838383u), m1c = in_vgpr(0x1C1C1C1Cu);
    lds_u32p fp = (lds_u32p)(scr + lane * 12u);
#pragma unroll
    for (int k = 0; k < 3; ++k) {
        const u32 spare = k < 2 ? hd[6 + k] << 5 : bsel(0x40404040u, hv[6] >> 1, hv[7] >> 2);
        fp[k] = bsel(m83, bsel(m80, hv[2 * k], hd[2 * k]),
                     bsel(m1c, rotr(bsel(m80, hv[2 * k + 1], hd[2 * k + 1]), 29), spare));
    }
    u32 cnt = 0;
    u64 m = __ballot(todo != 0u);
    while (m != 0ull) {
        u32 i;
        asm("v_ffbl_b32 %0, %1" : "=v"(i) : "v"(todo));
        const u32 m0 = (u32)__builtin_amdgcn_sbfe((int)i, 0, 1), m1 = (u32)__builtin_amdgcn_sbfe((int)i, 1, 1),
                  m2 = (u32)__builtin_amdgcn_sbfe((int)i, 2, 1);
        const u32 pk = bsel(m2, bsel(m1, bsel(m0, px[7], px[6]), bsel(m0, px[5], px[4])),
                            bsel(m1, bsel(m0, px[3], px[2]), bsel(m0, px[1], px[0])));
        const u32 pos = PLANAR ? (__umul24(i, 0x108u) & 0x780u) | ((i >> 3) & 1u) | (lane * 2u)
                               : (__umul24(i, 0x104u) & 0x604u) | (i >> 3) | (lane * 8u);
        const u32 ent = bsel(0x7FFu, pos, (pk >> (i & 24u)) << 9);
        const u32 slot = __builtin_amdgcn_mbcnt_hi((u32)(m >> 32), __builtin_amdgcn_mbcnt_lo((u32)m, cnt));
        *(LAS u16*)(scr + 768u + min(slot | (i & 64u), 64u) * 2u) = (u16)ent;
        todo &= todo - 1u;
        cnt += (u32)__builtin_popcountll(m);
        m = __ballot(todo != 0u);
        if (cnt > 64u) break;
    }
    FENCE();
    o.a = u32x4{lo[0], lo[1], lo[2], lo[3]};
    o.b = u32x4{lo[4], lo[5], lo[6], lo[7]};
    o.c = *(LAS u32x4*)(scr + lane * 16u);
    o.d = v[3];
    FENCE();
    o.hdr = base | H_VALID | (cnt > 32u ? H_LONG : 0u);
    if (__builtin_expect(cnt > 64u, 0)) {
        o.a = v[0]; o.b = v[1]; o.c = v[2];
        o.hdr = H_VALID | H_RAW;
    }
}

__device__ __forceinline__ void store_main(rsrc_t dst, u32 id, const Wire& p, u32 lane) {
    const u32 vo = OFF_DATA + id * BSTR + lane * 16u;
    const u32 nc = (p.hdr & H_RAW) ? 64u : (p.hdr & H_LONG) ? 56u : 52u;
    __builtin_amdgcn_raw_buffer_store_b128(p.a, dst, vo, 0, SYS);
    __builtin_amdgcn_raw_buffer_store_b128(p.b, dst, vo + 1024u, 0, SYS);
    __builtin_amdgcn_raw_buffer_store_b128(p.c, dst, lane < nc ? vo + 2048u : OOB, 0, SYS);
}
__device__ __forceinline__ void store_tail(rsrc_t dst, u32 id, const Wire& p, u32 lane) {
    __builtin_amdgcn_raw_buffer_store_b128(p.d, dst, OFF_DATA + id * BSTR + lane * 16u + 3072u, 0, SYS);
}

__device__ __forceinline__ u32 field(const u32x4& n, int k, u32 m80) {
    if (k < 6) return ((k & 1) ? rotr(n[k >> 1], 3) : n[k >> 1]) & 0x83838383u;
    u32 sg = n[2];
    asm("v_add_u32 %0, %1, %1" : "=v"(sg) : "v"(sg));
    if (k == 7) asm("v_add_u32 %0, %1, %1" : "=v"(sg) : "v"(sg));
    return bsel(m80, sg, n[k - 6] >> 5) & 0x83838383u;
}
typedef u64 base_t;
__device__ __forceinline__ base_t make_base(u32 h) { u32 b = (h & 0xFFu) * 0x00800080u; u64 bb = ((u64)b << 32) | b; asm("" : "+s"(bb)); return bb; }
__device__ __forceinline__ u32x4 unpack_piece(const u32x4& l, const u32x4& n, base_t b7, u32 m80, int j) {
    const u32 h0 = field(n, 2 * j, m80), h1 = field(n, 2 * j + 1, m80);
    const u32 a = l[2 * (j & 1)], b = l[2 * (j & 1) + 1];
    typedef u64 u64x2 __attribute__((ext_vector_type(2)));
    const u32x4 t = {__builtin_amdgcn_perm(h0, a, 0x05010400u), __builtin_amdgcn_perm(h0, a, 0x07030602u),
                     __builtin_amdgcn_perm(h1, b, 0x05010400u), __builtin_amdgcn_perm(h1, b, 0x07030602u)};
    return __builtin_bit_cast(u32x4, __builtin_bit_cast(u64x2, t) + b7);
}

__device__ __forceinline__ void set_m0(u32 x) { asm volatile("s_mov_b32 m0, %0" :: "s"(x) : "memory"); }
template <int OFF>
__device__ __forceinline__ void put_rows(const u32x4& v) {
    asm volatile("ds_write_addtid_b32 %0 offset:%4\n\tds_write_addtid_b32 %1 offset:%5\n\t"
                 "ds_write_addtid_b32 %2 offset:%6\n\tds_write_addtid_b32 %3 offset:%7"
                 :: "v"(v[0]), "v"(v[1]), "v"(v[2]), "v"(v[3]), "i"(OFF), "i"(OFF + 256), "i"(OFF + 512),
                    "i"(OFF + 768) : "memory");
}

__device__ __forceinline__ void publish(char* dst_ws, u32 id, u32 call, u32 hdr) {
    __hip_atomic_store((GAS u64*)(dst_ws + OFF_FLAG + id * 8u), ((u64)call << 32) | hdr,
                       __ATOMIC_RELAXED, __HIP_MEMORY_SCOPE_SYSTEM);
}

__device__ __forceinline__ u32 flag_of(u32 id) { return (id & ~127u) | ((id & 31u) << 2) | ((id >> 5) & 3u); }

__device__ __forceinline__ bool give_up(char* myws, u32 call, u64 t0) {
    u64 dt = __builtin_amdgcn_s_memrealtime() - t0;
    if (dt < 100000ull) return false;
    GAS u32* tmo = (GAS u32*)(myws + OFF_TMO);
    u32 t = uni(__hip_atomic_load(tmo, __ATOMIC_RELAXED, __HIP_MEMORY_SCOPE_AGENT));
    if (t == call || dt > DEADLINE_TICKS) {
        __hip_atomic_store(tmo, call, __ATOMIC_RELAXED, __HIP_MEMORY_SCOPE_AGENT);
        asm volatile("s_waitcnt vmcnt(0)" ::: "memory");
        return true;
    }
    return false;
}

__device__ __forceinline__ float silu(float y) {
    float c = __builtin_amdgcn_fmed3f(y, -60.0f, 60.0f);
    float e = __builtin_amdgcn_exp2f(c * -1.4426950408889634f);
    return y * __builtin_amdgcn_rcpf(1.0f + e);
}
__device__ __forceinline__ u32 cvt2(float a, float b) {
    f32x2 f = {a, b};
    return __builtin_bit_cast(u32, __builtin_convertvector(f, bf16x2));
}

struct Tabs { u32 ka, off, ids, w0l, w0h, w1l, w1h, lox, low; };
struct Src { u32 id, own, idx, two; };
__device__ __forceinline__ char* mk_ptr(u32 lo, u32 hi) { return (char*)(((u64)hi << 32) | lo); }
__device__ __forceinline__ char* tab_ptr(u32 lo, u32 hi, u32 i) {
    return mk_ptr(__builtin_amdgcn_readlane(lo, i), __builtin_amdgcn_readlane(hi, i));
}
__device__ __forceinline__ Tabs make_tabs(u32 ka, u32 lane, u32 rank) {
    const u32 isx = (lane >> 4) & 1u, kh = lane & 1u, half = (lane >> 1) & 1u, q = (lane >> 2) & 3u;
    Tabs T;
    T.ka = ka;
    T.off = isx ? (256u * half + 32u * q) * 8192u + 4096u * kh : (2048u * kh + 16u * q) * 1024u + half * 512u;
    u32 d0 = isx ? half | (kh << 2) | ((q & 1u) << 1) : (half << 1) | (kh << 2) | (q & 1u);
    u32 d1 = d0 ^ (isx ? 2u : 1u);
    const u32 skip = isx ? rank : rank ^ 2u;
    if (d0 == rank || d0 == skip) { d0 = d1; d1 = rank; }
    const u32 two = d1 != rank && d1 != skip;
    T.ids = ((((isx ^ 1u) * 8u + rank) * 8u + q) * 32u) | (((isx ^ 1u) * 1024u + ((half | (kh << 1)) * 8u + q) * 32u) << 12) |
            (two << 31);
    const u32 wsd = offsetof(Params, ws) / 4;
    T.w0l = __shfl(ka, wsd + d0 * 2u); T.w0h = __shfl(ka, wsd + d0 * 2u + 1u);
    T.w1l = __shfl(ka, wsd + d1 * 2u); T.w1h = __shfl(ka, wsd + d1 * 2u + 1u);
    T.lox = (lane >> 3) * 8192u + (((lane & 7u) ^ (lane >> 4)) * 16u);
    T.low = (lane >> 4) * 1024u + (((lane & 15u) ^ ((lane >> 4) << 2)) * 16u);
    return T;
}
__device__ __forceinline__ Src fetch_src(const Tabs& T, u32 e, bool on, u32x4 (&v)[4]) {
    const u32 gi = (e >> 4) & 127u;
    const u32 code = (u32)__builtin_amdgcn_readlane(T.ka, offsetof(Params, order) / 4 + (gi >> 2)) >> ((gi & 3u) * 8u);
    const u32 t = code & 31u, hi = (code >> 5) & 1u, isx = (code >> 6) & 1u;
    Src s;
    s.idx = isx * 16u + (e & 15u);
    const u32 off = __builtin_amdgcn_readlane(T.off, s.idx) + (isx ? t * 128u + hi * 1048576u : t * 65536u + hi * 256u);
    const u32 pin = offsetof(Params, in) / 4 + isx * 2u;
    const rsrc_t rs = make_rsrc(mk_ptr(__builtin_amdgcn_readlane(T.ka, pin), __builtin_amdgcn_readlane(T.ka, pin + 1u)),
                                on ? 4u << 20 : 0u);
    const u32 lo = isx ? T.lox : T.low, lo1 = lo ^ (isx * 64u), stride = isx ? 65536u : 4096u;
#pragma unroll
    for (int j = 0; j < 4; ++j)
        v[j] = __builtin_amdgcn_raw_buffer_load_b128(rs, (j & 1) ? lo1 : lo, off + j * stride, 0);
    const u32 ids = __builtin_amdgcn_readlane(T.ids, s.idx);
    s.id = (ids & 0xFFFu) + hi * 128u + t;
    s.own = ID_OWN + ((ids >> 12) & 0x7FFu) + hi * 128u + t;
    s.two = (ids >> 31) & ~(code >> 7) & 1u;
    return s;
}

__device__ __forceinline__ u32 prog_read(lds_u32p p) {
    return __hip_atomic_load(p, __ATOMIC_RELAXED, __HIP_MEMORY_SCOPE_WORKGROUP);
}
__device__ __forceinline__ void prog_wait(lds_u32p p, u32 seen, u32 need) {
    if (__builtin_expect(!__all(seen >= need), 0)) {
        while (!__all(prog_read(p) >= need)) __builtin_amdgcn_s_sleep(SPIN);
    }
    FENCE();
}
__device__ __forceinline__ void prog_set(lds_u32p p, u32 v) {
    FENCE();
    __hip_atomic_store(p, v, __ATOMIC_RELAXED, __HIP_MEMORY_SCOPE_WORKGROUP);
    FENCE();
}

struct Set { u32x4 l0, l1, n; u32 x; };
template <int N> struct IC { static constexpr int value = N; };

__global__ void __launch_bounds__(N_WAVES * 64, 4) ag_gemm_silu(Params P) {
    __shared__ __attribute__((aligned(16))) char smem[LDS_SIZE];
    LAS char* lds = (LAS char*)smem;
    const u32 lane = threadIdx.x & 63u;
    const u32 wave = uni(threadIdx.x >> 6);
    const u32 gwave = blockIdx.x * 16u + wave;
    static_assert(offsetof(Params, role) == 256, "kernel argument layout");
    const u32 ka = __hip_atomic_load((GAS u32*)(u64)__builtin_amdgcn_kernarg_segment_ptr() + lane, __ATOMIC_RELAXED,
                                     __HIP_MEMORY_SCOPE_WAVEFRONT);
    u32 rank = P.rank, rolew = P.role[(blockIdx.x >> 2) & 63u];
    char* myws = P.myws;
    asm volatile("" : "+s"(rank), "+s"(rolew), "+s"(myws) :: "memory");
    rank &= 7u;
    const rsrc_t mine = make_rsrc(myws, WS_NEED);
    STAMP_AT(0);
    lds_u32p prog = (lds_u32p)(lds + LDS_CTL);
    lds_u32p tab = (lds_u32p)(lds + LDS_TAB);
    const u32 xcd = blockIdx.x & 7u, idx = blockIdx.x >> 3;
    const u32 tm = (xcd >> 1) * 4u + (idx & 3u), tn = (idx >> 2) * 2u + (((idx >> 2) + xcd) & 1u);

    const bool diag = (tm >> 1) == (tn >> 1), jcol = !diag && (tn >> 1) == (rank ^ 2u);
    const u32 cell = diag ? tm >> 1 : jcol ? rank ^ 2u : rank;
    const u32 srcx = diag ? rank : jcol && (tm >> 1) == rank ? rank ^ 2u : tm >> 1;
    const u32 owner = diag || jcol ? rank : tn >> 1;
    auto first_id = [&](u32 par, u32 isB) {
        const u32 h = (isB ? tn : tm) & 1u, src = isB ? owner : srcx;
        const u32 own = ID_OWN + isB * 1024u + ((((cell >> isB) & 1u) | ((cell >> 2) << 1)) * 2u + h) * 128u;
        return src == rank ? own : par * N_IN + (isB * 16u + src * 2u + h) * 128u;
    };

    GAS u32* cnt = (GAS u32*)(myws + OFF_CNT + gwave * 8u);

    if (wave < W_WATCH) {
        u32 e = (rolew >> ((blockIdx.x & 3u) * 8u)) & 0xFFu;
        if (e >= (u32)PUSH_W) {
            const u32 cv = __hip_atomic_load(cnt, __ATOMIC_RELAXED, __HIP_MEMORY_SCOPE_AGENT);
            __builtin_amdgcn_s_barrier();
            if (e == 255u) return;
            const u32 call = uni(cv) + 1u;
            if (lane == 0) __hip_atomic_store(cnt, call, __ATOMIC_RELAXED, __HIP_MEMORY_SCOPE_AGENT);
            char* jws = P.ws[rank ^ 2u];
            const rsrc_t dst = make_rsrc(jws, WS_NEED);
            const u64 t0 = __builtin_amdgcn_s_memrealtime();
            for (u32 r = e - 160u; r < 12u * N_RELAY; r += RELAY_W) {
                const u32 g = r / 12u, a = (r % 12u) >> 1;
                const u32 code = (P.relay[(g >> 2) & 7u] >> ((g & 3u) * 8u)) & 0xFFu;
                const u32 src = rank ^ (a + (a ? 2u : 1u));
                const u32 id = (call & 1u) * N_IN +
                               (src * 8u + ((code >> 5) & 1u) * 4u + ((rank >> 1) & 1u) + (r & 1u) * 2u) * 32u + (code & 31u);
                u64 f;
                for (;;) {
                    f = __hip_atomic_load((GAS u64*)(myws + OFF_FLAG + flag_of(id) * 8u), __ATOMIC_RELAXED,
                                          __HIP_MEMORY_SCOPE_SYSTEM);
                    if (uni((u32)(f >> 32)) >= call) break;
                    if (give_up(myws, call, t0)) return;
                    __builtin_amdgcn_s_sleep(RELAY_SLEEP);
                }
                FENCE();
                Wire w;
                w.hdr = uni((u32)f);
                w.a = __builtin_amdgcn_raw_buffer_load_b128(mine, lane * 16u, OFF_DATA + id * BSTR, SYS);
                w.b = __builtin_amdgcn_raw_buffer_load_b128(mine, lane * 16u + 1024u, OFF_DATA + id * BSTR, SYS);
                w.c = __builtin_amdgcn_raw_buffer_load_b128(mine, lane * 16u + 2048u, OFF_DATA + id * BSTR, SYS);
                store_main(dst, id, w, lane);
                if (__builtin_expect(w.hdr & H_RAW, 0)) {
                    w.d = __builtin_amdgcn_raw_buffer_load_b128(mine, lane * 16u + 3072u, OFF_DATA + id * BSTR, SYS);
                    store_tail(dst, id, w, lane);
                }
                asm volatile("s_waitcnt vmcnt(0)" ::: "memory");
                if (lane == 0) publish(jws, flag_of(id), call, w.hdr);
                FENCE();
            }
            STAMP_AT(7);
            return;
        }
        __builtin_amdgcn_s_setprio(PUSH_PRIO);
        u32x4 v[4], u[4];
        const u32 cv = __hip_atomic_load(cnt, __ATOMIC_RELAXED, __HIP_MEMORY_SCOPE_AGENT);
        const Tabs T = make_tabs(ka, lane, rank);
        Src sa = fetch_src(T, e, true, u);
        Src sb = fetch_src(T, e + PUSH_W, true, v);
        __builtin_amdgcn_s_barrier();
        const u32 call = uni(cv) + 1u;
        const u32 idp = (call & 1u) * N_IN;
        const u32 tkoff = lane ? OOB : OFF_TKT;
        const u32 tbase = (call - 1u) * (u32)(N_SRC - 2 * PUSH_W) - 3u * (u32)PUSH_W;
        u32 tk = e + 2u * (u32)PUSH_W + tbase;
        Wire p;
        pack_block<false>(u, p, lds + LDS_PACK + wave * 1024u, lane);
        asm volatile("" : "+v"(v[0]), "+v"(v[1]), "+v"(v[2]), "+v"(v[3]));
        STAMP_AT(4);
        char* fptr = myws + OFF_CNT + gwave * 8u - OFF_FLAG;
        u32 fn = 1u, fh = call;
        bool hasb = true, go;
#define TQ(x)
        do {
            TQ(tq0);
            if (lane < fn) publish(fptr, 0u, call, fh);
            FENCE();
            char* w0 = tab_ptr(T.w0l, T.w0h, sa.idx);
            char* w1 = tab_ptr(T.w1l, T.w1h, sa.idx);
            const rsrc_t r0 = make_rsrc(w0, WS_NEED), r1 = make_rsrc(w1, sa.two ? WS_NEED : 0u);
            const u32 id = sa.id + idp;
            store_main(r0, id, p, lane);
            if (sa.two) store_main(r1, id, p, lane);
            store_main(mine, sa.own, p, lane);
            if (__builtin_expect(p.hdr & H_RAW, 0)) {
                store_tail(r0, id, p, lane);
                store_tail(r1, id, p, lane);
                store_tail(mine, sa.own, p, lane);
            }
            FENCE();
            e = uni(tk) - tbase;
            const bool hasc = hasb && e < (u32)N_SRC;
            const Src sc = fetch_src(T, e, hasc, u);
            tk = (u32)__builtin_amdgcn_raw_ptr_buffer_atomic_add_i32(1, mine, hasc ? tkoff : OOB, 0, 0);
            if (hasb && e == (u32)GATE_AT)
                __builtin_amdgcn_raw_buffer_store_b32(call, mine, lane ? OOB : OFF_GATE, 0, 16);
            fptr = (lane == 0u ? w0 : lane == 1u ? myws : w1) + (lane == 1u ? flag_of(sa.own) : flag_of(id)) * 8u;
            fn = 2u + sa.two;
            fh = p.hdr;
            TQ(tq1);
            if (hasb) pack_block<false>(v, p, lds + LDS_PACK + wave * 1024u, lane);
            TQ(tq2);
            asm volatile("s_waitcnt vmcnt(0)"
                         : "+v"(fptr), "+v"(tk), "+v"(u[0]), "+v"(u[1]), "+v"(u[2]), "+v"(u[3]) :: "memory");
            TQ(tq3);
#pragma unroll
            for (int j = 0; j < 4; ++j) v[j] = u[j];
            sa = sb; sb = sc;
            go = hasb; hasb = hasc;
        } while (go);
        if (lane < fn) publish(fptr, 0u, call, fh);
        STAMP_AT(1);
        return;
    }

    if (wave == W_MFMA) prog[lane] = lane - W_MFMA < 4u ? 3u : 0u;
    if (wave >= W_LOAD && wave < W_LOAD + 4u) tab[(wave - W_LOAD) * 64u + lane] = 0u;
    const u32 cv = __hip_atomic_load(cnt, __ATOMIC_RELAXED, __HIP_MEMORY_SCOPE_AGENT);
    asm volatile("s_waitcnt lgkmcnt(0)" ::: "memory");
    __builtin_amdgcn_s_barrier();
    const u32 call = uni(cv) + 1u;
    if (lane == 0) __hip_atomic_store(cnt, call, __ATOMIC_RELAXED, __HIP_MEMORY_SCOPE_AGENT);
    const u32 par = call & 1u;

    if (wave == W_WATCH) {
        const u32 op = lane >> 5;
        GAS u64* fl = (GAS u64*)(myws + OFF_FLAG + ((op ? first_id(par, 1u) : first_id(par, 0u)) + (lane & 31u)) * 8u);
        lds_u32p ent = tab + (op * 4u + (lane & 3u)) * 32u + ((lane >> 2) & 7u);
        const u64 t0 = __builtin_amdgcn_s_memrealtime();
        u32 fx = 0, fw = 0;
        for (;;) {
            const u32 f0 = op ? fw : fx;
            const u64 f = __hip_atomic_load(fl + f0 * 4u, __ATOMIC_RELAXED, __HIP_MEMORY_SCOPE_SYSTEM);
            const bool ok = (u32)(f >> 32) >= call;
            __hip_atomic_store(ent + f0, ok ? (u32)f : 0u, __ATOMIC_RELEASE, __HIP_MEMORY_SCOPE_WORKGROUP);
            const u64 miss = ~__ballot(ok);
            if (miss == 0ull && fx == 24u && fw == 24u) break;
            const u32 mx = (u32)miss, mw = (u32)(miss >> 32);
            const u32 nx = min(fx + (mx == 0u ? 8u : (mx & 0xFFFFu) ? 0u : 4u), 24u);
            const u32 nw = min(fw + (mw == 0u ? 8u : (mw & 0xFFFFu) ? 0u : 4u), 24u);
            const bool full = (mx == 0u && nx != fx) || (mw == 0u && nw != fw);
            const bool tail = (fx == 24u && fw == 24u && ((mx | mw) & 0xFFFFu) == 0u) || min(fx, fw) >= ((tm & 1u) ? (u32)TF1 : (u32)TF0);
            fx = nx; fw = nw;
            if (full) continue;
            if (give_up(myws, call, t0)) {
                prog_set(prog + 48u, 1u);
#pragma unroll
                for (int i = 0; i < 4; ++i)
                    __hip_atomic_store(tab + i * 64 + lane, H_VALID, __ATOMIC_RELEASE, __HIP_MEMORY_SCOPE_WORKGROUP);
                break;
            }
            if (tail) __builtin_amdgcn_s_sleep(TAIL_SLEEP);
            else __builtin_amdgcn_s_sleep(POLL_SLEEP);
        }
        STAMP_AT(1);
        while (uni(__hip_atomic_load((GAS u32*)(myws + OFF_GATE), __ATOMIC_RELAXED, __HIP_MEMORY_SCOPE_AGENT)) < call) {
            if (give_up(myws, call, t0)) break;
            __builtin_amdgcn_s_sleep(4);
        }
        prog_set(prog + W_WATCH, 64u);
        STAMP_AT(2);
        GAS u64* fp = (GAS u64*)(myws + OFF_FLAG + (ID_PART + blockIdx.x * 8u + (lane & 7u)) * 8u);
        for (;;) {
            const u64 f = __hip_atomic_load(fp, __ATOMIC_RELAXED, __HIP_MEMORY_SCOPE_SYSTEM);
            const bool ok = (u32)(f >> 32) >= call;
            __hip_atomic_store(prog + 32u + (lane & 7u), ok ? (u32)f : 0u, __ATOMIC_RELEASE, __HIP_MEMORY_SCOPE_WORKGROUP);
            if (__all(ok)) break;
            if (give_up(myws, call, t0)) {
                __hip_atomic_store(prog + 32u + (lane & 7u), H_VALID, __ATOMIC_RELEASE, __HIP_MEMORY_SCOPE_WORKGROUP);
                break;
            }
            __builtin_amdgcn_s_sleep(RED_SLEEP);
        }
        return;
    }

    if (wave >= W_LOAD) {
        const u32 lw8 = wave - W_LOAD, lw = lw8 & 3u, isB = lw8 >> 2;
        const u32 id0 = first_id(par, isB) + lw * 32u;
        const u32 blk = lw8 * 4096u;
        const u32 m80 = in_vgpr(0x80808080u);
        const u32 doff = OFF_DATA + id0 * BSTR + lane * 16u;
        const u32 noff = OFF_DATA + id0 * BSTR + lane * 12u;
        const u32 xlong = OFF_DATA + id0 * BSTR + lane * 2u, xshort = lane < 32u ? xlong : OOB;
        lds_u32p row = tab + lw8 * 32u + (lane & 31u);
        lds_u32p mprog = prog + W_MFMA + (lane & 3u);
        lds_u32p myprog = (lds_u32p)(size_t)in_vgpr(LDS_CTL + wave * 4u);
        rsrc_t lrs = mine;
        asm volatile("" : "+s"(lrs));
        Set R[4];
        u32 H[4] = {0, 0, 0, 0};
#pragma unroll
        for (int q = 0; q < 4; ++q)
            asm volatile("" : "=v"(R[q].l0), "=v"(R[q].l1), "=v"(R[q].n), "=v"(R[q].x));
        u32 rel = 0, hv = 0;
        auto fetch = [&](int d, u32 s, u32 avail, Set& q, u32& h) {
            u32 tv, tw, ts;
            asm volatile("s_cmp_lg_u32 %[rel], %[d]\n\ts_cbranch_scc1 .Lskip%=\n\t"
                         "s_cmp_le_u32 %[av], %[d]\n\ts_cbranch_scc1 .Lskip%=\n\t"
                         "s_add_u32 %[ts], %[s], %[d]\n\t"
                         "v_readlane_b32 %[h], %[hv], %[ts]\n\t"
                         "s_add_u32 %[rel], %[rel], 1\n\t"
                         "s_mul_i32 %[ts], %[ts], %[bs]\n\t"
                         "s_bitcmp1_b32 %[h], 10\n\ts_cselect_b64 vcc, -1, 0\n\ts_nop 1\n\t"
                         "v_cndmask_b32 %[tv], %[noff], %[doff], vcc\n\t"
                         "s_bitcmp1_b32 %[h], 9\n\ts_cselect_b64 vcc, -1, 0\n\ts_nop 1\n\t"
                         "v_cndmask_b32 %[tw], %[xs], %[xl], vcc\n\t"
                         "buffer_load_dwordx4 %[n], %[tv], %[rs], %[ts] offen offset:2048 sc0 sc1\n\t"
                         "buffer_load_dwordx4 %[l0], %[doff], %[rs], %[ts] offen sc0 sc1\n\t"
                         "buffer_load_dwordx4 %[l1], %[doff], %[rs], %[ts] offen offset:1024 sc0 sc1\n\t"
                         "buffer_load_ushort %[x], %[tw], %[rs], %[ts] offen offset:2816 sc0 sc1\n"
                         ".Lskip%=:"
                         : [x] "+v"(q.x), [n] "+v"(q.n), [l0] "+v"(q.l0), [l1] "+v"(q.l1), [h] "+s"(h),
                           [rel] "+s"(rel), [tv] "=&v"(tv), [tw] "=&v"(tw), [ts] "=&s"(ts)
                         : [d] "i"(d), [av] "s"(avail), [s] "s"(s), [hv] "v"(hv), [noff] "v"(noff),
                           [doff] "v"(doff), [xs] "v"(xshort), [xl] "v"(xlong), [rs] "s"(lrs), [bs] "i"(BSTR)
                         : "vcc", "scc", "memory");
        };
        auto landed = [&](Set& q) {
            asm volatile("s_cmp_lt_u32 %[rel], 4\n\ts_cbranch_scc1 .La%=\n\ts_waitcnt vmcnt(12)\n\ts_branch .Lz%=\n"
                         ".La%=:\n\ts_cmp_lt_u32 %[rel], 3\n\ts_cbranch_scc1 .Lb%=\n\ts_waitcnt vmcnt(8)\n\ts_branch .Lz%=\n"
                         ".Lb%=:\n\ts_cmp_lt_u32 %[rel], 2\n\ts_cbranch_scc1 .Lc%=\n\ts_waitcnt vmcnt(4)\n\ts_branch .Lz%=\n"
                         ".Lc%=:\n\ts_waitcnt vmcnt(0)\n.Lz%=:"
                         : "+v"(q.x), "+v"(q.n), "+v"(q.l0), "+v"(q.l1) : [rel] "s"(rel) : "scc", "memory");
        };
        auto arrived = [&]() {
            const u32 ok = (u32)__ballot(hv != 0u);
            return ok == 0xFFFFFFFFu ? 32u : (u32)__builtin_ctz(~ok);
        };
        u32 lim = 0;
        for (u32 s0 = 0; s0 < (u32)STEPS; s0 += 4) {
#pragma unroll
            for (int q = 0; q < 4; ++q) {
                const u32 s = s0 + q;
                Set& c = R[q];
                LAS char* dst = lds + q * SLOT;
                const u32 hn = prog_read(row), mseen = prog_read(mprog);
                const u32 lim0 = lim;
                if (__builtin_expect(s + 3u >= lim0, 0)) {
                    u32 rdy = arrived();
                    while (rdy <= s) {
                        __builtin_amdgcn_s_sleep(SPIN);
                        hv = __hip_atomic_load(row, __ATOMIC_ACQUIRE, __HIP_MEMORY_SCOPE_WORKGROUP);
                        rdy = arrived();
                    }
                    const u32 avail = rdy - s;
#pragma unroll
                    for (int d = 0; d < 4; ++d) fetch(d, s, avail, R[(q + d) & 3], H[(q + d) & 3]);
                    landed(c);
                    rel -= 1u;
                    const u64 xm = (__ballot(hv >= H_LONG) | (1ull << 32)) >> (s + 4u);
                    lim = rel == 3u ? min(rdy, s + 4u + (u32)__builtin_ctzll(xm | (1ull << 40))) : 0u;
                }
                {
                    Set& nx = R[(q + 3) & 3];
                    u32 ts;
                    asm volatile("s_cmp_ge_u32 %[s3], %[lim]\n\ts_cbranch_scc1 .Lslow%=\n\t"
                                 "s_mul_i32 %[ts], %[s3], %[bs]\n\t"
                                 "v_readlane_b32 %[h], %[hv], %[s3]\n\t"
                                 "buffer_load_dwordx4 %[n], %[noff], %[rs], %[ts] offen offset:2048 sc0 sc1\n\t"
                                 "buffer_load_dwordx4 %[l0], %[doff], %[rs], %[ts] offen sc0 sc1\n\t"
                                 "buffer_load_dwordx4 %[l1], %[doff], %[rs], %[ts] offen offset:1024 sc0 sc1\n\t"
                                 "buffer_load_ushort %[x], %[xs], %[rs], %[ts] offen offset:2816 sc0 sc1\n\t"
                                 "s_waitcnt vmcnt(12)\n.Lslow%=:"
                                 : [n] "+v"(nx.n), [l0] "+v"(nx.l0), [l1] "+v"(nx.l1), [x] "+v"(nx.x), "+v"(c.x),
                                   "+v"(c.n), "+v"(c.l0), "+v"(c.l1), [h] "+s"(H[(q + 3) & 3]), [ts] "=&s"(ts)
                                 : [s3] "s"(s + 3u), [lim] "s"(lim0), [hv] "v"(hv), [doff] "v"(doff), [noff] "v"(noff),
                                   [xs] "v"(xshort), [rs] "s"(lrs), [bs] "i"(BSTR)
                                 : "scc", "memory");
                }
                const base_t b7 = make_base(H[q]);
                auto put = [&](auto jc, const u32x4& t) {
                    constexpr int j = decltype(jc)::value;
                    *(LAS u32x4*)(dst + blk + lane * 16u + j * 1024u) = t;
                    SB();
                };
                u32x4 t0 = unpack_piece(c.l0, c.n, b7, m80, 0);
                asm volatile("" : "+v"(t0));
                hv |= hn;
                prog_wait(mprog, mseen, s);
                put(IC<0>{}, t0);
                put(IC<1>{}, unpack_piece(c.l0, c.n, b7, m80, 1));
                put(IC<2>{}, unpack_piece(c.l1, c.n, b7, m80, 2));
                put(IC<3>{}, unpack_piece(c.l1, c.n, b7, m80, 3));
                if (c.x != 0u)
                    __hip_atomic_fetch_xor((lds_u32p)(dst + blk + ((c.x & 0x7FEu) << 1)),
                                           ((c.x >> 1) & 0x7C00u) << ((c.x & 1u) << 4), __ATOMIC_RELAXED,
                                           __HIP_MEMORY_SCOPE_WORKGROUP);
                FENCE();
                if (__builtin_expect(H[q] >= H_RAW, 0)) {
                    put(IC<0>{}, c.l0);
                    put(IC<1>{}, c.l1);
                    put(IC<2>{}, c.n);
                    put(IC<3>{}, __builtin_amdgcn_raw_buffer_load_b128(mine, doff + 3072u, s * BSTR, SYS));
                }
                prog_set(myprog, s + 1u);
                SB();
            }
        }
        STAMP_AT(2);
        const u32 wt = blockIdx.x * 2u + (lw8 >> 2), uh = (lw8 >> 1) & 1u, hf = lw8 & 1u;
        const u32 idp = ID_PART + wt * 4u + uh;
        LAS char* pl = lds + LDS_RED + lw8 * 2048u;
        const u32x4 zero = {0u, 0u, 0u, 0u};
        *(LAS u32x4*)(pl + lane * 16u) = zero;
        *(LAS u32x4*)(pl + lane * 16u + 1024u) = zero;
        const u32 rb = OFF_DATA + idp * BSTR;
        const u32 ox = in_vgpr(rb + 2816u + lane * 2u), on = in_vgpr(rb + 2048u + lane * 12u),
                  ol = in_vgpr(rb + hf * 1024u + lane * 16u);
        u32 hp;
        for (;;) {
            hp = __hip_atomic_load(prog + 32u + (lw8 & 4u) + uh + (lane & 1u) * 2u, __ATOMIC_ACQUIRE,
                                   __HIP_MEMORY_SCOPE_WORKGROUP);
            if (__all(hp != 0u)) break;
            __builtin_amdgcn_s_sleep(1);
        }
        FENCE();
        u32x4 l[2], n[2];
        u32 x[2];
#pragma unroll
        for (int k = 0; k < 2; ++k) {
            x[k] = __builtin_amdgcn_raw_buffer_load_b16(lrs, ox, k * 2u * BSTR, SYS);
            n[k] = __builtin_amdgcn_raw_buffer_load_b128(lrs, on, k * 2u * BSTR, SYS);
            l[k] = __builtin_amdgcn_raw_buffer_load_b128(lrs, ol, k * 2u * BSTR, SYS);
        }
        const u32 tmo = __hip_atomic_load((GAS u32*)(myws + OFF_TMO), __ATOMIC_RELAXED, __HIP_MEMORY_SCOPE_AGENT);
        SB();
        STAMP_AT(3);
        u32x4 val[2][2];
#pragma unroll
        for (int k = 0; k < 2; ++k) {
            const u32 hk = __builtin_amdgcn_readlane(hp, k);
            const bool use = x[k] != 0u && ((x[k] >> 10) & 1u) == hf && lane < ((hk & H_LONG) ? 64u : 32u);
            FENCE();
            if (use) *(LAS u16*)(pl + ((x[k] & 0x3FFu) << 1)) = (u16)((x[k] >> 1) & 0x7C00u);
            FENCE();
            const u32x4 p0 = *(LAS u32x4*)(pl + lane * 16u), p1 = *(LAS u32x4*)(pl + lane * 16u + 1024u);
            FENCE();
            if (k == 0 && use) *(LAS u16*)(pl + ((x[k] & 0x3FFu) << 1)) = 0;
            const base_t b7 = make_base(hk);
            val[k][0] = (hf ? unpack_piece(l[k], n[k], b7, m80, 2) : unpack_piece(l[k], n[k], b7, m80, 0)) ^ p0;
            val[k][1] = (hf ? unpack_piece(l[k], n[k], b7, m80, 3) : unpack_piece(l[k], n[k], b7, m80, 1)) ^ p1;
            if (__builtin_expect(hk & H_RAW, 0)) {
                val[k][0] = __builtin_amdgcn_raw_buffer_load_b128(lrs, ol + hf * 1024u, k * 2u * BSTR, SYS);
                val[k][1] = __builtin_amdgcn_raw_buffer_load_b128(lrs, ol + hf * 1024u + 1024u, k * 2u * BSTR, SYS);
            }
        }
        const u32 poison = tmo == call ? 0x7FC07FC0u : 0u;
        const rsrc_t ro = make_rsrc(P.out, 4u << 20);
        const u32 oo = ((wt >> 3) * 64u + uh * 32u + (lane & 31u)) * 1024u + (wt & 7u) * 128u + hf * 64u + (lane >> 5) * 16u;
#pragma unroll
        for (int j = 0; j < 2; ++j) {
            u32x4 o;
#pragma unroll
            for (int d = 0; d < 4; ++d) {
                const u32 a = val[0][j][d], b = val[1][j][d];
                float y0 = __builtin_bit_cast(float, a << 16) + __builtin_bit_cast(float, b << 16);
                float y1 = __builtin_bit_cast(float, a & 0xFFFF0000u) + __builtin_bit_cast(float, b & 0xFFFF0000u);
                o[d] = cvt2(silu(y0), silu(y1)) | poison;
            }
            __builtin_amdgcn_raw_buffer_store_b128(o, ro, oo + j * 32u, 0, 0);
        }
        STAMP_AT(4);
        return;
    }

    const u32 mw = wave - W_MFMA, wm = mw >> 1, wn = mw & 1u;
    const u32 rdA = (64u * wm + (lane & 31u)) * 128u + (((lane >> 5) ^ ((lane >> 1) & 7u)) * 16u);
    const u32 rdB = 16384u + (8u * (lane >> 5) + ((lane >> 2) & 3u)) * 256u +
                    (((wn * 8u + ((lane >> 4) & 1u) * 2u + (lane & 1u)) ^ (((lane >> 2) & 3u) << 2)) * 16u) +
                    ((lane >> 1) & 1u) * 8u;
    f32x16 acc[2][2];
#pragma unroll
    for (int a = 0; a < 2; ++a)
#pragma unroll
        for (int b = 0; b < 2; ++b)
#pragma unroll
            for (int r = 0; r < 16; ++r) acc[a][b][r] = 0.f;

    lds_u32p lprog = prog + W_LOAD + (lane & 7u);
    u32 lseen = 0;
    bf16x8 X[2][2], W[2][2];
    auto rd_x = [&](u32 buf, int ks, int mi) {
        return *(LAS bf16x8*)(lds + buf + ((rdA ^ (ks * 32u)) + mi * 4096u));
    };
    auto rd_w = [&](u32 buf, int ks, int ni) {
        u32 a = buf + (rdB ^ (ni * 64u)) + ks * 4096u;
        s16x4 lo = __builtin_amdgcn_ds_read_tr16_b64_v4i16((LAS s16x4*)(lds + a));
        s16x4 hi = __builtin_amdgcn_ds_read_tr16_b64_v4i16((LAS s16x4*)(lds + a + 1024u));
        return __builtin_bit_cast(bf16x8, __builtin_shufflevector(lo, hi, 0, 1, 2, 3, 4, 5, 6, 7));
    };
#define MMA(mi, ni, st) acc[mi][ni] = __builtin_amdgcn_mfma_f32_32x32x16_bf16(W[st][ni], X[st][mi], acc[mi][ni], 0, 0, 0)
    auto step = [&](u32 s, u32 buf, u32 nxt, bool more) {
#pragma unroll
        for (int ks = 0; ks < 4; ++ks) {
            const int st = ks & 1, k2 = (ks + 2) & 3;
            const u32 src = ks < 2 ? buf : nxt;
            const bool ld = ks < 2 || more;
            MMA(0, 0, st); SB();
            if (ks == 1 && more) lseen = prog_read(lprog);
            if (ks == 2 && more) prog_set(prog + wave, s + 4u);
            SB();
            MMA(0, 1, st); SB();
            if (ks == 2 && more) prog_wait(lprog, lseen, s + 2u);
            if (ld) X[st][0] = rd_x(src, k2, 0);
            SB();
            MMA(1, 1, st); SB();
            if (ld) W[st][1] = rd_w(src, k2, 1);
            SB();
            MMA(1, 0, st); SB();
            if (ld) { W[st][0] = rd_w(src, k2, 0); X[st][1] = rd_x(src, k2, 1); }
            SB();
        }
    };
    prog_wait(lprog, lseen, 1u);
#pragma unroll
    for (int st = 0; st < 2; ++st) {
        W[st][0] = rd_w(0, st, 0); X[st][0] = rd_x(0, st, 0);
        W[st][1] = rd_w(0, st, 1); X[st][1] = rd_x(0, st, 1);
    }
    for (u32 s = 0; s < (u32)STEPS - 4u; s += 4) {
        step(s, 0, SLOT, true);
        step(s + 1u, SLOT, 2 * SLOT, true);
        step(s + 2u, 2 * SLOT, 3 * SLOT, true);
        step(s + 3u, 3 * SLOT, 0, true);
    }
    step(28u, 0, SLOT, true);
    step(29u, SLOT, 2 * SLOT, true);
    step(30u, 2 * SLOT, 3 * SLOT, true);
    step(31u, 3 * SLOT, 0, false);
    STAMP_AT(2);

    {
        char* ows = P.ws[owner & 7u];
        const rsrc_t dst = make_rsrc(ows, WS_NEED);
        const u32 row = 512u * srcx + 256u * (cell & 1u) + 128u * (tm & 1u) + 64u * wm;
        const u32 col = 256u * ((cell >> 1) & 1u) + 128u * (tn & 1u) + 64u * wn;
        const u32 id = ID_PART + ((row >> 6) * 8u + (col >> 6)) * 4u + (cell >> 2) * 2u;
        const u32 bad = prog_read(prog + 48u);
        auto pack_half = [&](int h, Wire& p) {
            u32x4 v[4];
#pragma unroll
            for (int j = 0; j < 4; ++j) {
                const f32x16 c = acc[h][j >> 1];
                const int o = 8 * (j & 1);
                v[j] = u32x4{cvt2(c[o], c[o + 1]), cvt2(c[o + 2], c[o + 3]), cvt2(c[o + 4], c[o + 5]), cvt2(c[o + 6], c[o + 7])};
            }
            pack_block<false>(v, p, lds + LDS_PACK + wave * 1024u, lane);
        };
        Wire p0, p1;
        pack_half(0, p0);
        STAMP_AT(4);
        const u32 chain = (tm & 1u) ? (u32)CHAIN1 : (tn & 1u) ? (u32)CHAIN01 : (u32)CHAIN0;
        if (owner != rank) prog_wait(prog + ((mw & (chain - 1u)) ? wave - 1u : W_WATCH), 0u, 64u);
        STAMP_AT(5);
        if (__builtin_expect(p0.hdr & H_RAW, 0)) store_tail(dst, id, p0, lane);
        store_main(dst, id, p0, lane);
        pack_half(1, p1);
        asm volatile("s_waitcnt vmcnt(0)" ::: "memory");
        STAMP_AT(6);
        if (__builtin_expect(p1.hdr & H_RAW, 0)) store_tail(dst, id + 1u, p1, lane);
        store_main(dst, id + 1u, p1, lane);
        asm volatile("s_waitcnt vmcnt(0)" ::: "memory");
        if (lane < 2u && !bad) publish(ows, id + lane, call, lane ? p1.hdr : p0.hdr);
        prog_set(prog + wave, 64u);
    }
    STAMP_AT(3);
}

extern "C" void kernel_launch(int rank, int nranks, ncclComm_t comm, hipStream_t stream,
                              void* const* d_in, const int* in_sizes, int n_in,
                              void* d_out, int out_size, const int* in_dtypes, int out_dtype,
                              void* const* d_out_all, void* const* d_ws_all, size_t ws_bytes) {
    if (nranks != NRANKS || (unsigned)rank >= (unsigned)NRANKS || ws_bytes < (size_t)WS_NEED || n_in != 2 ||
        in_sizes[0] != 512 * 4096 || in_sizes[1] != 4096 * 512 || out_size != 4096 * 512) {
        fprintf(stderr, "kernel_launch: unexpected problem shape\n");
        return;
    }
    Params p;
    p.in[0] = (const char*)d_in[1];
    p.in[1] = (const char*)d_in[0];
    p.out = (char*)d_out;
    for (int r = 0; r < NRANKS; ++r) p.ws[r] = (char*)d_ws_all[r];
    p.myws = p.ws[rank];
    p.rank = (u32)rank;
    p.pad = 0;
    {
        unsigned char ord[128];
        int n = 0, nx[4] = {0, 0, 0, 0};
        const int lag[4] = {0, LAG_R, 0, LAG_B};
        for (int t = 1; t <= STEPS; ++t)
            for (int st = 0; st < 4; ++st)
                while (nx[st] < (t * (STEPS - lag[st]) + STEPS - 1) / STEPS) { ord[n++] = (unsigned char)(st * 32 + nx[st]); ++nx[st]; }
        while (n < 128)
            for (int st = 1; st < 4; st += 2)
                if (nx[st] < STEPS) { ord[n++] = (unsigned char)(st * 32 + nx[st]); ++nx[st]; }
        unsigned char rel[32] = {0};
        int nrel = 0;
        for (int k = 0; k < N_RELAY; ++k) {
            int pos = RELAY_SHIFT + (RELAY_LAST - RELAY_SHIFT) * k / (N_RELAY > 1 ? N_RELAY - 1 : 1);
            while (pos < 128 && (ord[pos] < 64 || ord[pos] >= 128)) ++pos;
            if (pos >= 128) { fprintf(stderr, "kernel_launch: bad relay plan\n"); return; }
            const unsigned char c = ord[pos] | 128;
            for (int q = pos; q > pos - RELAY_SHIFT; --q) ord[q] = ord[q - 1];
            ord[pos - RELAY_SHIFT] = c;
            rel[nrel++] = c;
        }
        unsigned char role[256];
        int np = 0, nr = 0;
        for (int b = 0; b < 256; ++b) role[b] = 255;
        for (int pass = 0; pass < 3; ++pass)
            for (int b = 0; b < 256; ++b) {
                const int xcd = b & 7, idx = b >> 3, tm = (xcd >> 1) * 4 + (idx & 3), po = idx >> 2;
                const bool local = (tm >> 1) == po || po == rank || po == (rank ^ 2);
                if (role[b] != 255 || (pass == 0 && !local) || (pass == 1 && !(tm & 1))) continue;
                if (np < PUSH_W) role[b] = (unsigned char)np++;
                else if (nr < RELAY_W) role[b] = (unsigned char)(160 + nr++);
            }
        auto word = [](const unsigned char* c) { return c[0] | (c[1] << 8) | (c[2] << 16) | ((u32)c[3] << 24); };
        for (int i = 0; i < 32; ++i) p.order[i] = word(ord + 4 * i);
        for (int i = 0; i < 8; ++i) p.relay[i] = word(rel + 4 * i);
        for (int i = 0; i < 64; ++i) p.role[i] = word(role + 4 * i);
    }
    hipLaunchKernelGGL(ag_gemm_silu, dim3(N_WG), dim3(N_WAVES * 64), 0, stream, p);
    HIP_CHECK(hipGetLastError());
}
```

```cpp
#include <hip/hip_runtime.h>
#include <rccl/rccl.h>
#include <cstdio>
#include <cstdint>
#include <cstddef>

#define HIP_CHECK(expr) do { hipError_t _e = (expr); if (_e != hipSuccess) { \
    fprintf(stderr, "HIP error %s at %s:%d\n", hipGetErrorString(_e), __FILE__, __LINE__); \
    return; } } while (0)

typedef unsigned u32;
typedef unsigned long long u64;
typedef unsigned short u16;
typedef u32 u32x4 __attribute__((ext_vector_type(4)));
typedef u32 u32x2 __attribute__((ext_vector_type(2)));
typedef float f32x16 __attribute__((ext_vector_type(16)));
typedef short s16x4 __attribute__((ext_vector_type(4)));
typedef unsigned short u16x2 __attribute__((ext_vector_type(2)));
typedef __bf16 bf16x8 __attribute__((ext_vector_type(8)));
typedef __bf16 bf16x2 __attribute__((ext_vector_type(2)));
typedef float f32x2 __attribute__((ext_vector_type(2)));
#define GAS __attribute__((address_space(1)))
#define LAS __attribute__((address_space(3)))
typedef __amdgpu_buffer_rsrc_t rsrc_t;
typedef LAS u32* lds_u32p;

constexpr int CHAIN0 = 2;
constexpr int CHAIN1 = 2;
constexpr int CHAIN01 = CHAIN0;
constexpr int POLL_SLEEP = 24;
constexpr int RED_SLEEP = 4;
constexpr int N_RELAY = 12;
constexpr int RELAY_W = 32;
constexpr int RELAY_SLEEP = 8;
constexpr int RELAY_SHIFT = 16;
constexpr int RELAY_LAST = 122;
constexpr int GATE_D0 = 300;
constexpr int GATE_D1 = 450;
constexpr int GATE_D2 = 600;
constexpr int OWN_EARLY = 1;
constexpr int OWN_RATE = 100;
constexpr int PUSHERS = 120;
constexpr int LAG_R = 8;
constexpr int LAG_B = 16;
constexpr int PUSH_PRIO = 3;
constexpr int GATE_AT = 2047;
constexpr int BSTRIDE = 4352;

constexpr int NRANKS = 8;
constexpr int STEPS = 32;
constexpr int N_WG = 256;
constexpr u32 W_WATCH = 1, W_MFMA = W_WATCH + 1, W_LOAD = W_MFMA + 4;
constexpr int N_WAVES = W_LOAD + 8;
constexpr int N_IN = 4096;
constexpr int N_PART = 2048;
constexpr int N_SRC = 2048;
constexpr int ID_PART = 2 * N_IN;
constexpr int ID_OWN = ID_PART + N_PART;
constexpr int N_BLK = ID_OWN + N_SRC;
constexpr int PUSH_W = PUSHERS;

constexpr u32 OFF_CNT = 0;
constexpr u32 OFF_TMO = 32768;
constexpr u32 OFF_TKT = 33024;
constexpr u32 OFF_GATE = 33280;
constexpr u32 OFF_FLAG = 65536;
constexpr u32 OFF_DBG = 262144;
constexpr u32 OFF_DATA = 1u << 20;
constexpr u32 BSTR = BSTRIDE;
constexpr u32 WS_NEED = OFF_DATA + N_BLK * BSTR;
constexpr u64 DEADLINE_TICKS = 40000000ull;
constexpr int SYS = 17;
constexpr u32 OOB = 0x80000000u;

constexpr u32 H_VALID = 1, H_LONG = 2, H_RAW = 4, H_BASE = 0x7F807F80;

constexpr u32 SLOT = 32768;
constexpr u32 LDS_CTL = 4 * SLOT;
constexpr u32 LDS_TAB = LDS_CTL + 256;
constexpr u32 LDS_PACK = LDS_TAB + 1024;
constexpr u32 LDS_RED = LDS_PACK + 8192;
constexpr u32 LDS_SIZE = LDS_RED + 8 * 2048;

struct Params {
    const char* in[2];
    char* out;
    char* ws[NRANKS];
    u32 rank;
    u32 pad;
    u32 fill[32];
    u32 relay[8];
    u32 role[64];
    u32 order[256];
    char* myws;
};

__device__ __forceinline__ rsrc_t make_rsrc(const void* p, u32 bytes) {
    return __builtin_amdgcn_make_buffer_rsrc((void*)p, 0, bytes, 0x00020000);
}
__device__ __forceinline__ u32 uni(u32 v) { return __builtin_amdgcn_readfirstlane(v); }
__device__ __forceinline__ u32 rotr(u32 v, u32 s) { return __builtin_amdgcn_alignbit(v, v, s); }
#define SB() __builtin_amdgcn_sched_barrier(0)
#define FENCE() asm volatile("" ::: "memory")

struct Wire { u32x4 a, b, c, d; u32 hdr; };

__device__ __forceinline__ u32 wave_sum_u32(u32 v) {
    v += (u32)__builtin_amdgcn_update_dpp(0, (int)v, 0x111, 0xf, 0xf, true);
    v += (u32)__builtin_amdgcn_update_dpp(0, (int)v, 0x112, 0xf, 0xf, true);
    v += (u32)__builtin_amdgcn_update_dpp(0, (int)v, 0x114, 0xf, 0xf, true);
    v += (u32)__builtin_amdgcn_update_dpp(0, (int)v, 0x118, 0xf, 0xf, true);
    return __builtin_amdgcn_readlane(v, 15) + __builtin_amdgcn_readlane(v, 31) +
           __builtin_amdgcn_readlane(v, 47) + __builtin_amdgcn_readlane(v, 63);
}

__device__ __forceinline__ u32 in_vgpr(u32 x) { asm("" : "+v"(x)); return x; }
__device__ __forceinline__ u32 bsel(u32 m, u32 a, u32 b) {
    return __builtin_amdgcn_bitop3_b32(m, a, b, 0xCA);
}

struct NoEarly { __device__ __forceinline__ void operator()(const Wire&) const {} };
template <class Early = NoEarly>
__device__ __forceinline__ void pack_block(const u32x4 (&v)[4], Wire& o, LAS char* scr, u32 lane, Early early = Early()) {
    *(LAS u16*)(scr + 768u + lane * 2u) = 0;
    const u32 m80 = in_vgpr(0x80808080u), m7c = in_vgpr(0x7C7C7C7Cu);
    u32 hv[8];
#pragma unroll
    for (int k = 0; k < 8; ++k)
        hv[k] = __builtin_amdgcn_perm(v[k >> 1][2 * (k & 1) + 1], v[k >> 1][2 * (k & 1)], 0x07050301u);
    u32 sum = 0;
#pragma unroll
    for (int k = 1; k < 8; k += 2) sum = __builtin_amdgcn_sad_u8(hv[k] & 0x7F7F7F7Fu, 0u, sum);
    const u32 base = min((max(wave_sum_u32(sum), 1895u) - 1895u) >> 9, 248u);
    const u16x2 b7 = __builtin_bit_cast(u16x2, in_vgpr(base * 0x00800080u));
    u32 lo[8], d[16], hd[8], px[8], todo = 0;
#pragma unroll
    for (int k = 0; k < 16; ++k) {
        const u32 t = v[k >> 2][k & 3];
        d[k] = __builtin_bit_cast(u32, __builtin_bit_cast(u16x2, t) - b7);
    }
#pragma unroll
    for (int k = 0; k < 8; ++k) lo[k] = __builtin_amdgcn_perm(d[2 * k + 1], d[2 * k], 0x06040200u);
    o.a = u32x4{lo[0], lo[1], lo[2], lo[3]};
    o.b = u32x4{lo[4], lo[5], lo[6], lo[7]};
    early(o);
#pragma unroll
    for (int k = 0; k < 8; ++k) {
        hd[k] = __builtin_amdgcn_perm(d[2 * k + 1], d[2 * k], 0x07050301u);
        const u32 q = hd[k] & m7c;
        px[k] = ((hv[k] | m80) - q) ^ hv[k];
        todo = bsel(m80, q + m7c, todo >> 1);
    }
    const u32 m83 = in_vgpr(0x83838383u), m1c = in_vgpr(0x1C1C1C1Cu);
    lds_u32p fp = (lds_u32p)(scr + lane * 12u);
#pragma unroll
    for (int k = 0; k < 3; ++k) {
        const u32 spare = k < 2 ? hd[6 + k] << 5 : bsel(0x40404040u, hv[6] >> 1, hv[7] >> 2);
        fp[k] = bsel(m83, bsel(m80, hv[2 * k], hd[2 * k]),
                     bsel(m1c, rotr(bsel(m80, hv[2 * k + 1], hd[2 * k + 1]), 29), spare));
    }
    const u32 mine_n = (u32)__builtin_popcount(todo);
    u32 inc = mine_n;
    inc += (u32)__builtin_amdgcn_update_dpp(0, (int)inc, 0x111, 0xf, 0xf, true);
    inc += (u32)__builtin_amdgcn_update_dpp(0, (int)inc, 0x112, 0xf, 0xf, true);
    inc += (u32)__builtin_amdgcn_update_dpp(0, (int)inc, 0x114, 0xf, 0xf, true);
    inc += (u32)__builtin_amdgcn_update_dpp(0, (int)inc, 0x118, 0xf, 0xf, true);
    inc += (u32)__builtin_amdgcn_update_dpp(0, (int)inc, 0x142, 0xa, 0xf, false);
    inc += (u32)__builtin_amdgcn_update_dpp(0, (int)inc, 0x143, 0xc, 0xf, false);
    u32 cnt = __builtin_amdgcn_readlane(inc, 63);
    u32 slot2 = (inc - mine_n) * 2u;
    auto emit = [&]() {
        u32 i;
        asm("v_ffbl_b32 %0, %1" : "=v"(i) : "v"(todo));
        const u32 m0 = (u32)__builtin_amdgcn_sbfe((int)i, 0, 1), m1 = (u32)__builtin_amdgcn_sbfe((int)i, 1, 1),
                  m2 = (u32)__builtin_amdgcn_sbfe((int)i, 2, 1);
        const u32 pk = bsel(m2, bsel(m1, bsel(m0, px[7], px[6]), bsel(m0, px[5], px[4])),
                            bsel(m1, bsel(m0, px[3], px[2]), bsel(m0, px[1], px[0])));
        const u32 pos = (__umul24(i, 0x104u) & 0x604u) | (i >> 3) | (lane * 8u);
        const u32 ent = bsel(0x7FFu, pos - slot2, (pk >> (i & 24u)) << 9);
        *(LAS u16*)(scr + 768u + min(slot2 | (i & 128u), 128u)) = (u16)ent;
        todo &= todo - 1u;
        slot2 += 2u;
    };
    emit();
    emit();
    if (cnt <= 64u)
        while (__ballot(todo != 0u) != 0ull) emit();
    FENCE();
    o.c = *(LAS u32x4*)(scr + lane * 16u);
    o.d = v[3];
    FENCE();
    o.hdr = base * 0x00800080u | H_VALID | (cnt > 32u ? H_LONG : 0u);
    if (__builtin_expect(cnt > 64u, 0)) {
        o.a = v[0]; o.b = v[1]; o.c = v[2];
        o.hdr = H_VALID | H_RAW;
    }
}

__device__ __forceinline__ void store_ab(rsrc_t dst, u32 id, const Wire& p, u32 lane) {
    const u32 vo = OFF_DATA + id * BSTR + lane * 16u;
    __builtin_amdgcn_raw_buffer_store_b128(p.a, dst, vo, 0, SYS);
    __builtin_amdgcn_raw_buffer_store_b128(p.b, dst, vo + 1024u, 0, SYS);
}
__device__ __forceinline__ void store_c(rsrc_t dst, u32 id, const Wire& p, u32 lane) {
    const u32 nc = (p.hdr & H_RAW) ? 64u : (p.hdr & H_LONG) ? 56u : 52u;
    __builtin_amdgcn_raw_buffer_store_b128(p.c, dst, lane < nc ? OFF_DATA + id * BSTR + lane * 16u + 2048u : OOB, 0, SYS);
}
__device__ __forceinline__ void store_main(rsrc_t dst, u32 id, const Wire& p, u32 lane) {
    store_ab(dst, id, p, lane);
    store_c(dst, id, p, lane);
}
__device__ __forceinline__ void store_tail(rsrc_t dst, u32 id, const Wire& p, u32 lane) {
    __builtin_amdgcn_raw_buffer_store_b128(p.d, dst, OFF_DATA + id * BSTR + lane * 16u + 3072u, 0, SYS);
}

__device__ __forceinline__ u32 field(const u32x4& n, int k, u32 m80) {
    if (k < 6) return ((k & 1) ? rotr(n[k >> 1], 3) : n[k >> 1]) & 0x83838383u;
    u32 sg = n[2];
    asm("v_add_u32 %0, %1, %1" : "=v"(sg) : "v"(sg));
    if (k == 7) asm("v_add_u32 %0, %1, %1" : "=v"(sg) : "v"(sg));
    return bsel(m80, sg, n[k - 6] >> 5) & 0x83838383u;
}
__device__ __forceinline__ u64 make_base(u32 h) {
    u64 b = h & H_BASE;
    b |= b << 32;
    asm("" : "+s"(b));
    return b;
}
__device__ __forceinline__ u32x4 unpack_piece(const u32x4& l, const u32x4& n, u64 b7, u32 m80, int j) {
    const u32 h0 = field(n, 2 * j, m80), h1 = field(n, 2 * j + 1, m80);
    const u32 a = l[2 * (j & 1)], b = l[2 * (j & 1) + 1];
    typedef u64 u64x2 __attribute__((ext_vector_type(2)));
    const u32x4 t = {__builtin_amdgcn_perm(h0, a, 0x05010400u), __builtin_amdgcn_perm(h0, a, 0x07030602u),
                     __builtin_amdgcn_perm(h1, b, 0x05010400u), __builtin_amdgcn_perm(h1, b, 0x07030602u)};
    return __builtin_bit_cast(u32x4, __builtin_bit_cast(u64x2, t) + b7);
}

__device__ __forceinline__ void publish(char* dst_ws, u32 id, u32 call, u32 hdr) {
    __hip_atomic_store((GAS u64*)(dst_ws + OFF_FLAG + id * 8u), ((u64)call << 32) | hdr,
                       __ATOMIC_RELAXED, __HIP_MEMORY_SCOPE_SYSTEM);
}

__device__ __forceinline__ u32 flag_of(u32 id) { return (id & ~127u) | ((id & 31u) << 2) | ((id >> 5) & 3u); }

__device__ __forceinline__ bool give_up(char* myws, u32 call, u64 t0) {
    u64 dt = __builtin_amdgcn_s_memrealtime() - t0;
    if (dt < 100000ull) return false;
    GAS u32* tmo = (GAS u32*)(myws + OFF_TMO);
    u32 t = uni(__hip_atomic_load(tmo, __ATOMIC_RELAXED, __HIP_MEMORY_SCOPE_AGENT));
    if (t == call || dt > DEADLINE_TICKS) {
        __hip_atomic_store(tmo, call, __ATOMIC_RELAXED, __HIP_MEMORY_SCOPE_AGENT);
        asm volatile("s_waitcnt vmcnt(0)" ::: "memory");
        return true;
    }
    return false;
}

__device__ __forceinline__ float silu(float y) {
    float c = __builtin_amdgcn_fmed3f(y, -60.0f, 60.0f);
    float e = __builtin_amdgcn_exp2f(c * -1.4426950408889634f);
    return y * __builtin_amdgcn_rcpf(1.0f + e);
}
__device__ __forceinline__ u32 cvt2(float a, float b) {
    f32x2 f = {a, b};
    return __builtin_bit_cast(u32, __builtin_convertvector(f, bf16x2));
}

struct Tabs { u32x4 ord; u32 ka, off, ids, w0l, w0h, w1l, w1h, lox, low; };
struct Src { u32 id, own, idx, two; };
__device__ __forceinline__ char* mk_ptr(u32 lo, u32 hi) { return (char*)(((u64)hi << 32) | lo); }
__device__ __forceinline__ char* tab_ptr(u32 lo, u32 hi, u32 i) {
    return mk_ptr(__builtin_amdgcn_readlane(lo, i), __builtin_amdgcn_readlane(hi, i));
}
__device__ __forceinline__ Tabs make_tabs(u32 ka, u32x4 ord, u32 lane, u32 rank) {
    const u32 isx = (lane >> 4) & 1u, kh = lane & 1u, half = (lane >> 1) & 1u, q = (lane >> 2) & 3u;
    Tabs T;
    T.ka = ka;
    T.ord = ord;
    T.off = isx ? (256u * half + 32u * q) * 8192u + 4096u * kh : (2048u * kh + 16u * q) * 1024u + half * 512u;
    u32 d0 = isx ? half | (kh << 2) | ((q & 1u) << 1) : (half << 1) | (kh << 2) | (q & 1u);
    u32 d1 = d0 ^ (isx ? 2u : 1u);
    const u32 skip = isx ? rank : rank ^ 2u;
    if (d0 == rank || d0 == skip) { d0 = d1; d1 = rank; }
    const u32 two = d1 != rank && d1 != skip;
    T.ids = ((((isx ^ 1u) * 8u + rank) * 8u + q) * 32u) | (((isx ^ 1u) * 1024u + ((half | (kh << 1)) * 8u + q) * 32u) << 12) |
            (two << 31);
    const u32 wsd = offsetof(Params, ws) / 4;
    T.w0l = __shfl(ka, wsd + d0 * 2u); T.w0h = __shfl(ka, wsd + d0 * 2u + 1u);
    T.w1l = __shfl(ka, wsd + d1 * 2u); T.w1h = __shfl(ka, wsd + d1 * 2u + 1u);
    T.lox = (lane >> 3) * 8192u + (((lane & 7u) ^ (lane >> 4)) * 16u);
    T.low = (lane >> 4) * 1024u + (((lane & 15u) ^ ((lane >> 4) << 2)) * 16u);
    return T;
}
__device__ __forceinline__ Src fetch_src(const Tabs& T, u32 e, bool on, u32x4 (&v)[4]) {
    const u32 ln = (e >> 3) & 63u;
    const u32 o0 = __builtin_amdgcn_readlane(T.ord[0], ln), o1 = __builtin_amdgcn_readlane(T.ord[1], ln),
              o2 = __builtin_amdgcn_readlane(T.ord[2], ln), o3 = __builtin_amdgcn_readlane(T.ord[3], ln);
    const u32 code = ((e & 1024u) ? ((e & 512u) ? o3 : o2) : ((e & 512u) ? o1 : o0)) >> ((e & 4u) * 4u);
    const u32 t = code & 31u, hi = (code >> 5) & 1u, isx = (code >> 6) & 1u;
    Src s;
    s.idx = isx * 16u + (e & 3u) * 4u + ((code >> 8) & 3u);
    const u32 off = __builtin_amdgcn_readlane(T.off, s.idx) + (isx ? t * 128u + hi * 1048576u : t * 65536u + hi * 256u);
    const u32 pin = offsetof(Params, in) / 4 + isx * 2u;
    const rsrc_t rs = make_rsrc(mk_ptr(__builtin_amdgcn_readlane(T.ka, pin), __builtin_amdgcn_readlane(T.ka, pin + 1u)),
                                on ? 4u << 20 : 0u);
    const u32 lo = isx ? T.lox : T.low, lo1 = lo ^ (isx * 64u), stride = isx ? 65536u : 4096u;
#pragma unroll
    for (int j = 0; j < 4; ++j)
        v[j] = __builtin_amdgcn_raw_buffer_load_b128(rs, (j & 1) ? lo1 : lo, off + j * stride, 0);
    const u32 ids = __builtin_amdgcn_readlane(T.ids, s.idx);
    s.id = (ids & 0xFFFu) + hi * 128u + t;
    s.own = ID_OWN + ((ids >> 12) & 0x7FFu) + hi * 128u + t;
    s.two = (ids >> 31) & ~(code >> 7) & 1u;
    return s;
}

__device__ __forceinline__ u32 prog_read(lds_u32p p) {
    return __hip_atomic_load(p, __ATOMIC_RELAXED, __HIP_MEMORY_SCOPE_WORKGROUP);
}
__device__ __forceinline__ void prog_wait(lds_u32p p, u32 seen, u32 need) {
    if (__builtin_expect(__ballot(seen < need) != 0ull, 0)) {
        while (__ballot(prog_read(p) < need) != 0ull) __builtin_amdgcn_s_sleep(1);
    }
    FENCE();
}
__device__ __forceinline__ void prog_set(lds_u32p p, u32 v) {
    FENCE();
    __hip_atomic_store(p, v, __ATOMIC_RELAXED, __HIP_MEMORY_SCOPE_WORKGROUP);
    FENCE();
}

struct Set { u32x4 l0, l1, n; u32 x; };

__global__ void __launch_bounds__(N_WAVES * 64, 4) ag_gemm_silu(Params P) {
    __shared__ __attribute__((aligned(16))) char smem[LDS_SIZE];
    LAS char* lds = (LAS char*)smem;
    const u32 lane = threadIdx.x & 63u;
    const u32 wave = uni(threadIdx.x >> 6);
    const u32 gwave = blockIdx.x * 16u + wave;
    static_assert(offsetof(Params, role) == 256, "kernel argument layout");
    u32 ka = 0;
    u32x4 ord = {0u, 0u, 0u, 0u};
    if (wave == 0u) {
        GAS u32* kp = (GAS u32*)(u64)__builtin_amdgcn_kernarg_segment_ptr() + lane;
        ka = __hip_atomic_load(kp, __ATOMIC_RELAXED, __HIP_MEMORY_SCOPE_WAVEFRONT);
#pragma unroll
        for (int k = 0; k < 4; ++k)
            ord[k] = __hip_atomic_load(kp + offsetof(Params, order) / 4 + k * 64, __ATOMIC_RELAXED, __HIP_MEMORY_SCOPE_WAVEFRONT);
    }
    u32 rank = P.rank, rolew = P.role[(blockIdx.x >> 2) & 63u];
    char* myws = P.myws;
    asm volatile("" : "+s"(rank), "+s"(rolew), "+s"(myws) :: "memory");
    rank &= 7u;
    const rsrc_t mine = make_rsrc(myws, WS_NEED);
    lds_u32p prog = (lds_u32p)(lds + LDS_CTL);
    lds_u32p tab = (lds_u32p)(lds + LDS_TAB);
    const u32 xcd = blockIdx.x & 7u, idx = blockIdx.x >> 3;
    const u32 tm = (xcd >> 1) * 4u + (idx & 3u), tn = (idx >> 2) * 2u + (((idx >> 2) + xcd) & 1u);

    const bool diag = (tm >> 1) == (tn >> 1), jcol = !diag && (tn >> 1) == (rank ^ 2u);
    const u32 cell = diag ? tm >> 1 : jcol ? rank ^ 2u : rank;
    const u32 srcx = diag ? rank : jcol && (tm >> 1) == rank ? rank ^ 2u : tm >> 1;
    const u32 owner = diag || jcol ? rank : tn >> 1;
    const bool late = (tm & 1u) && (srcx | 2u) != (rank | 2u);
    auto first_id = [&](u32 par, u32 isB) {
        const u32 h = (isB ? tn : tm) & 1u, src = isB ? owner : srcx;
        const u32 own = ID_OWN + isB * 1024u + ((((cell >> isB) & 1u) | ((cell >> 2) << 1)) * 2u + h) * 128u;
        return src == rank ? own : par * N_IN + (isB * 16u + src * 2u + h) * 128u;
    };

    GAS u32* cnt = (GAS u32*)(myws + OFF_CNT + gwave * 8u);

    if (wave < W_WATCH) {
        u32 e = (rolew >> ((blockIdx.x & 3u) * 8u)) & 0xFFu;
        if (e >= (u32)PUSH_W) {
            const u32 cv = __hip_atomic_load(cnt, __ATOMIC_RELAXED, __HIP_MEMORY_SCOPE_AGENT);
            __builtin_amdgcn_s_barrier();
            if (e == 255u) return;
            const u32 call = uni(cv) + 1u;
            if (lane == 0) __hip_atomic_store(cnt, call, __ATOMIC_RELAXED, __HIP_MEMORY_SCOPE_AGENT);
            char* jws = P.ws[rank ^ 2u];
            const rsrc_t dst = make_rsrc(jws, WS_NEED);
            const u64 t0 = __builtin_amdgcn_s_memrealtime();
            for (u32 r = e - 160u; r < 12u * N_RELAY; r += RELAY_W) {
                const u32 g = r / 12u, a = (r % 12u) >> 1;
                const u32 code = (P.relay[(g >> 2) & 7u] >> ((g & 3u) * 8u)) & 0xFFu;
                const u32 src = rank ^ (a + (a ? 2u : 1u));
                const u32 id = (call & 1u) * N_IN +
                               (src * 8u + ((code >> 5) & 1u) * 4u + ((rank >> 1) & 1u) + (r & 1u) * 2u) * 32u + (code & 31u);
                u64 f;
                for (;;) {
                    f = __hip_atomic_load((GAS u64*)(myws + OFF_FLAG + flag_of(id) * 8u), __ATOMIC_RELAXED,
                                          __HIP_MEMORY_SCOPE_SYSTEM);
                    if (uni((u32)(f >> 32)) >= call) break;
                    if (give_up(myws, call, t0)) return;
                    __builtin_amdgcn_s_sleep(RELAY_SLEEP);
                }
                FENCE();
                Wire w;
                w.hdr = uni((u32)f);
                w.a = __builtin_amdgcn_raw_buffer_load_b128(mine, lane * 16u, OFF_DATA + id * BSTR, SYS);
                w.b = __builtin_amdgcn_raw_buffer_load_b128(mine, lane * 16u + 1024u, OFF_DATA + id * BSTR, SYS);
                w.c = __builtin_amdgcn_raw_buffer_load_b128(mine, lane * 16u + 2048u, OFF_DATA + id * BSTR, SYS);
                store_main(dst, id, w, lane);
                if (__builtin_expect(w.hdr & H_RAW, 0)) {
                    w.d = __builtin_amdgcn_raw_buffer_load_b128(mine, lane * 16u + 3072u, OFF_DATA + id * BSTR, SYS);
                    store_tail(dst, id, w, lane);
                }
                asm volatile("s_waitcnt vmcnt(0)" ::: "memory");
                if (lane == 0) publish(jws, flag_of(id), call, w.hdr);
                FENCE();
            }
            return;
        }
        __builtin_amdgcn_s_setprio(PUSH_PRIO);
        u32x4 v[4], u[4];
        const u32 cv = __hip_atomic_load(cnt, __ATOMIC_RELAXED, __HIP_MEMORY_SCOPE_AGENT);
        const Tabs T = make_tabs(ka, ord, lane, rank);
        Src sa = fetch_src(T, e, true, u);
        Src sb = fetch_src(T, e + PUSH_W, true, v);
        __builtin_amdgcn_s_barrier();
        const u32 call = uni(cv) + 1u;
        const u32 idp = (call & 1u) * N_IN;
        const u32 tkoff = lane ? OOB : OFF_TKT;
        const u32 tbase = (call - 1u) * (u32)(N_SRC - 2 * PUSH_W) - 3u * (u32)PUSH_W;
        u32 tk = e + 2u * (u32)PUSH_W + tbase;
        Wire p;
        pack_block(u, p, lds + LDS_PACK + wave * 1024u, lane, [&](const Wire& w) {
            FENCE();
            store_ab(make_rsrc(tab_ptr(T.w0l, T.w0h, sa.idx), WS_NEED), sa.id + idp, w, lane);
            store_ab(make_rsrc(tab_ptr(T.w1l, T.w1h, sa.idx), sa.two ? WS_NEED : 0u), sa.id + idp, w, lane);
            FENCE();
        });
        bool ab = p.hdr & H_RAW;
        if (ab) asm volatile("s_waitcnt vmcnt(0)" ::: "memory");
        asm volatile("" : "+v"(v[0]), "+v"(v[1]), "+v"(v[2]), "+v"(v[3]));
        char* fptr = myws + OFF_CNT + gwave * 8u - OFF_FLAG;
        u32 fn = 1u, fh = call;
        bool hasb = true, go;
        do {
            if (lane < fn) publish(fptr, 0u, call, fh);
            FENCE();
            char* w0 = tab_ptr(T.w0l, T.w0h, sa.idx);
            char* w1 = tab_ptr(T.w1l, T.w1h, sa.idx);
            const rsrc_t r0 = make_rsrc(w0, WS_NEED), r1 = make_rsrc(w1, sa.two ? WS_NEED : 0u);
            const u32 id = sa.id + idp;
            if (__builtin_expect(ab, 1)) store_ab(r0, id, p, lane);
            store_c(r0, id, p, lane);
            if (sa.two) {
                if (__builtin_expect(ab, 1)) store_ab(r1, id, p, lane);
                store_c(r1, id, p, lane);
            }
            ab = true;
            store_main(mine, sa.own, p, lane);
            if (__builtin_expect(p.hdr & H_RAW, 0)) {
                store_tail(r0, id, p, lane);
                store_tail(r1, id, p, lane);
                store_tail(mine, sa.own, p, lane);
            }
            FENCE();
            e = uni(tk) - tbase;
            const bool hasc = hasb && e < (u32)N_SRC;
            const Src sc = fetch_src(T, e, hasc, u);
            tk = (u32)__builtin_amdgcn_raw_ptr_buffer_atomic_add_i32(1, mine, hasc ? tkoff : OOB, 0, 0);
            if (hasb && e == (u32)GATE_AT)
                __builtin_amdgcn_raw_buffer_store_b64(u32x2{(u32)__builtin_amdgcn_s_memrealtime(), call}, mine,
                                                      lane ? OOB : OFF_GATE, 0, 16);
            fptr = (lane == 0u ? w0 : lane == 1u ? myws : w1) + (lane == 1u ? flag_of(sa.own) : flag_of(id)) * 8u;
            fn = 2u + sa.two;
            fh = p.hdr;
            if (hasb) pack_block(v, p, lds + LDS_PACK + wave * 1024u, lane);
            asm volatile("s_waitcnt vmcnt(0)"
                         : "+v"(fptr), "+v"(tk), "+v"(u[0]), "+v"(u[1]), "+v"(u[2]), "+v"(u[3]) :: "memory");
#pragma unroll
            for (int j = 0; j < 4; ++j) v[j] = u[j];
            sa = sb; sb = sc;
            go = hasb; hasb = hasc;
        } while (go);
        if (lane < fn) publish(fptr, 0u, call, fh);
        return;
    }

    if (wave == W_MFMA) prog[lane] = lane - W_MFMA < 4u ? 3u : 0u;
    if (wave >= W_LOAD && wave < W_LOAD + 4u) tab[(wave - W_LOAD) * 64u + lane] = 0u;
    const u32 cv = __hip_atomic_load(cnt, __ATOMIC_RELAXED, __HIP_MEMORY_SCOPE_AGENT);
    asm volatile("s_waitcnt lgkmcnt(0)" ::: "memory");
    __builtin_amdgcn_s_barrier();
    const u32 call = uni(cv) + 1u;
    if (lane == 0) __hip_atomic_store(cnt, call, __ATOMIC_RELAXED, __HIP_MEMORY_SCOPE_AGENT);
    const u32 par = call & 1u;

    if (wave == W_WATCH) {
        const u32 op = lane >> 5;
        GAS u64* fl = (GAS u64*)(myws + OFF_FLAG + ((op ? first_id(par, 1u) : first_id(par, 0u)) + (lane & 31u)) * 8u);
        lds_u32p ent = tab + (op * 4u + (lane & 3u)) * 32u + ((lane >> 2) & 7u);
        const u64 t0 = __builtin_amdgcn_s_memrealtime();
        u32 fx = 0, fw = 0;
        for (;;) {
            const u32 f0 = op ? fw : fx;
            const u64 f = __hip_atomic_load(fl + f0 * 4u, __ATOMIC_RELAXED, __HIP_MEMORY_SCOPE_SYSTEM);
            const bool ok = (u32)(f >> 32) >= call;
            __hip_atomic_store(ent + f0, ok ? (u32)f : 0u, __ATOMIC_RELEASE, __HIP_MEMORY_SCOPE_WORKGROUP);
            const u64 miss = ~__ballot(ok);
            if (miss == 0ull && fx == 24u && fw == 24u) break;
            const u32 mx = (u32)miss, mw = (u32)(miss >> 32);
            const u32 nx = min(fx + (mx == 0u ? 8u : (mx & 0xFFFFu) ? 0u : 4u), 24u);
            const u32 nw = min(fw + (mw == 0u ? 8u : (mw & 0xFFFFu) ? 0u : 4u), 24u);
            const bool full = (mx == 0u && nx != fx) || (mw == 0u && nw != fw);
            const bool tail = fx == 24u && fw == 24u && ((mx | mw) & 0xFFFFu) == 0u;
            fx = nx; fw = nw;
            if (full) continue;
            if (give_up(myws, call, t0)) {
                prog_set(prog + 48u, 1u);
#pragma unroll
                for (int i = 0; i < 4; ++i)
                    __hip_atomic_store(tab + i * 64 + lane, H_VALID, __ATOMIC_RELEASE, __HIP_MEMORY_SCOPE_WORKGROUP);
                break;
            }
            if (tail) __builtin_amdgcn_s_sleep(1);
            else __builtin_amdgcn_s_sleep(POLL_SLEEP);
        }
        for (;;) {
            const u64 g = __hip_atomic_load((GAS u64*)(myws + OFF_GATE), __ATOMIC_RELAXED, __HIP_MEMORY_SCOPE_AGENT);
            if (uni((u32)(g >> 32)) >= call) {
                const u32 hold = late ? GATE_D2 : (tn & 1u) ? GATE_D1 : GATE_D0;
                while ((u32)__builtin_amdgcn_s_memrealtime() - uni((u32)g) < hold) __builtin_amdgcn_s_sleep(2);
                break;
            }
            if (give_up(myws, call, t0)) break;
            __builtin_amdgcn_s_sleep(4);
        }
        prog_set(prog + W_WATCH, 64u);
        GAS u64* fp = (GAS u64*)(myws + OFF_FLAG + (ID_PART + blockIdx.x * 8u + (lane & 7u)) * 8u);
        for (;;) {
            const u64 f = __hip_atomic_load(fp, __ATOMIC_RELAXED, __HIP_MEMORY_SCOPE_SYSTEM);
            const bool ok = (u32)(f >> 32) >= call;
            __hip_atomic_store(prog + 32u + (lane & 7u), ok ? (u32)f : 0u, __ATOMIC_RELEASE, __HIP_MEMORY_SCOPE_WORKGROUP);
            if (__all(ok)) break;
            if (give_up(myws, call, t0)) {
                __hip_atomic_store(prog + 32u + (lane & 7u), H_VALID, __ATOMIC_RELEASE, __HIP_MEMORY_SCOPE_WORKGROUP);
                break;
            }
            __builtin_amdgcn_s_sleep(RED_SLEEP);
        }
        return;
    }

    if (wave >= W_LOAD) {
        const u32 lw8 = wave - W_LOAD, lw = lw8 & 3u, isB = lw8 >> 2;
        const u32 id0 = first_id(par, isB) + lw * 32u;
        const u32 blk = lw8 * 4096u;
        const u32 m80 = in_vgpr(0x80808080u), mffc = in_vgpr(0xFFCu), vblk = in_vgpr(blk), lane4 = lane * 4u;
        const u32 doff = OFF_DATA + id0 * BSTR + lane * 16u;
        const u32 noff = OFF_DATA + id0 * BSTR + lane * 12u;
        const u32 xlong = OFF_DATA + id0 * BSTR + lane * 2u, xshort = lane < 32u ? xlong : OOB;
        lds_u32p row = lane < 32u ? tab + lw8 * 32u + lane : prog + W_MFMA + (lane & 3u);
        const u32 inc = lane >> 5;
        u32 need = 0;
        lds_u32p myprog = (lds_u32p)(size_t)in_vgpr(LDS_CTL + wave * 4u);
        rsrc_t lrs = mine;
        asm volatile("" : "+s"(lrs));
        Set R[4];
        u32 H[4] = {0, 0, 0, 0};
#pragma unroll
        for (int q = 0; q < 4; ++q)
            asm volatile("" : "=v"(R[q].l0), "=v"(R[q].l1), "=v"(R[q].n), "=v"(R[q].x));
        u32 rel = 0, hv = 0;
        auto fetch = [&](int d, u32 s, u32 avail, Set& q, u32& h) {
            u32 tv, tw, ts;
            asm volatile("s_cmp_lg_u32 %[rel], %[d]\n\ts_cbranch_scc1 .Lskip%=\n\t"
                         "s_cmp_le_u32 %[av], %[d]\n\ts_cbranch_scc1 .Lskip%=\n\t"
                         "s_add_u32 %[ts], %[s], %[d]\n\t"
                         "v_readlane_b32 %[h], %[hv], %[ts]\n\t"
                         "s_add_u32 %[rel], %[rel], 1\n\t"
                         "s_mul_i32 %[ts], %[ts], %[bs]\n\t"
                         "s_bitcmp1_b32 %[h], 2\n\ts_cselect_b64 vcc, -1, 0\n\ts_nop 1\n\t"
                         "v_cndmask_b32 %[tv], %[noff], %[doff], vcc\n\t"
                         "s_bitcmp1_b32 %[h], 1\n\ts_cselect_b64 vcc, -1, 0\n\ts_nop 1\n\t"
                         "v_cndmask_b32 %[tw], %[xs], %[xl], vcc\n\t"
                         "buffer_load_dwordx4 %[n], %[tv], %[rs], %[ts] offen offset:2048 sc0 sc1\n\t"
                         "buffer_load_dwordx4 %[l0], %[doff], %[rs], %[ts] offen sc0 sc1\n\t"
                         "buffer_load_dwordx4 %[l1], %[doff], %[rs], %[ts] offen offset:1024 sc0 sc1\n\t"
                         "buffer_load_ushort %[x], %[tw], %[rs], %[ts] offen offset:2816 sc0 sc1\n"
                         ".Lskip%=:"
                         : [x] "+v"(q.x), [n] "+v"(q.n), [l0] "+v"(q.l0), [l1] "+v"(q.l1), [h] "+s"(h),
                           [rel] "+s"(rel), [tv] "=&v"(tv), [tw] "=&v"(tw), [ts] "=&s"(ts)
                         : [d] "i"(d), [av] "s"(avail), [s] "s"(s), [hv] "v"(hv), [noff] "v"(noff),
                           [doff] "v"(doff), [xs] "v"(xshort), [xl] "v"(xlong), [rs] "s"(lrs), [bs] "i"(BSTR)
                         : "vcc", "scc", "memory");
        };
        auto landed = [&](Set& q) {
            asm volatile("s_cmp_lt_u32 %[rel], 4\n\ts_cbranch_scc1 .La%=\n\ts_waitcnt vmcnt(12)\n\ts_branch .Lz%=\n"
                         ".La%=:\n\ts_cmp_lt_u32 %[rel], 3\n\ts_cbranch_scc1 .Lb%=\n\ts_waitcnt vmcnt(8)\n\ts_branch .Lz%=\n"
                         ".Lb%=:\n\ts_cmp_lt_u32 %[rel], 2\n\ts_cbranch_scc1 .Lc%=\n\ts_waitcnt vmcnt(4)\n\ts_branch .Lz%=\n"
                         ".Lc%=:\n\ts_waitcnt vmcnt(0)\n.Lz%=:"
                         : "+v"(q.x), "+v"(q.n), "+v"(q.l0), "+v"(q.l1) : [rel] "s"(rel) : "scc", "memory");
        };
        auto arrived = [&]() {
            const u32 ok = (u32)__ballot(hv != 0u);
            return ok == 0xFFFFFFFFu ? 32u : (u32)__builtin_ctz(~ok);
        };
        u32 lim = 0;
        for (u32 s0 = 0; s0 < (u32)STEPS; s0 += 4) {
#pragma unroll
            for (int q = 0; q < 4; ++q) {
                const u32 s = s0 + q;
                Set& c = R[q];
                LAS char* dst = lds + q * SLOT;
                const u32 hn = prog_read(row);
                const u32 lim0 = lim;
                if (__builtin_expect(s + 3u >= lim0, 0)) {
                    u32 rdy = arrived();
                    while (rdy <= s) {
                        __builtin_amdgcn_s_sleep(1);
                        hv = __hip_atomic_load(row, __ATOMIC_ACQUIRE, __HIP_MEMORY_SCOPE_WORKGROUP);
                        rdy = arrived();
                    }
                    const u32 avail = rdy - s;
#pragma unroll
                    for (int d = 0; d < 4; ++d) fetch(d, s, avail, R[(q + d) & 3], H[(q + d) & 3]);
                    landed(c);
                    rel -= 1u;
                    const u64 xm = (__ballot((hv & (H_LONG | H_RAW)) != 0u) | (1ull << 32)) >> (s + 4u);
                    lim = rel == 3u ? min(rdy, s + 4u + (u32)__builtin_ctzll(xm | (1ull << 40))) : 0u;
                }
                {
                    Set& nx = R[(q + 3) & 3];
                    u32 ts;
                    asm volatile("s_cmp_ge_u32 %[s3], %[lim]\n\ts_cbranch_scc1 .Lslow%=\n\t"
                                 "s_mul_i32 %[ts], %[s3], %[bs]\n\t"
                                 "v_readlane_b32 %[h], %[hv], %[s3]\n\t"
                                 "buffer_load_dwordx4 %[n], %[noff], %[rs], %[ts] offen offset:2048 sc0 sc1\n\t"
                                 "buffer_load_dwordx4 %[l0], %[doff], %[rs], %[ts] offen sc0 sc1\n\t"
                                 "buffer_load_dwordx4 %[l1], %[doff], %[rs], %[ts] offen offset:1024 sc0 sc1\n\t"
                                 "buffer_load_ushort %[x], %[xs], %[rs], %[ts] offen offset:2816 sc0 sc1\n\t"
                                 "s_waitcnt vmcnt(12)\n.Lslow%=:"
                                 : [n] "+v"(nx.n), [l0] "+v"(nx.l0), [l1] "+v"(nx.l1), [x] "+v"(nx.x), "+v"(c.x),
                                   "+v"(c.n), "+v"(c.l0), "+v"(c.l1), [h] "+s"(H[(q + 3) & 3]), [ts] "=&s"(ts)
                                 : [s3] "s"(s + 3u), [lim] "s"(lim0), [hv] "v"(hv), [doff] "v"(doff), [noff] "v"(noff),
                                   [xs] "v"(xshort), [rs] "s"(lrs), [bs] "i"(BSTR)
                                 : "scc", "memory");
                }
                const u64 b7 = make_base(H[q]);
                auto put = [&](int j, const u32x4& t) { *(LAS u32x4*)(dst + blk + lane * 16u + j * 1024u) = t; SB(); };
                u32x4 t0 = unpack_piece(c.l0, c.n, b7, m80, 0);
                asm volatile("" : "+v"(t0));
                hv |= hn;
                prog_wait(row, hn, need);
                need += inc;
                put(0, t0);
                put(1, unpack_piece(c.l0, c.n, b7, m80, 1));
                put(2, unpack_piece(c.l1, c.n, b7, m80, 2));
                put(3, unpack_piece(c.l1, c.n, b7, m80, 3));
                __hip_atomic_fetch_xor((lds_u32p)(dst + bsel(mffc, c.x + c.x + lane4, vblk)),
                                       ((c.x >> 1) & 0x7C00u) << ((c.x & 1u) << 4), __ATOMIC_RELAXED,
                                       __HIP_MEMORY_SCOPE_WORKGROUP);
                FENCE();
                if (__builtin_expect(H[q] & H_RAW, 0)) {
                    put(0, c.l0);
                    put(1, c.l1);
                    put(2, c.n);
                    put(3, __builtin_amdgcn_raw_buffer_load_b128(mine, doff + 3072u, s * BSTR, SYS));
                }
                prog_set(myprog, s + 1u);
                SB();
            }
        }
        const u32 wt = blockIdx.x * 2u + (lw8 >> 2), uh = (lw8 >> 1) & 1u, hf = lw8 & 1u;
        const u32 idp = ID_PART + wt * 4u + uh;
        LAS char* pl = lds + LDS_RED + lw8 * 2048u;
        const u32x4 zero = {0u, 0u, 0u, 0u};
        *(LAS u32x4*)(pl + lane * 16u) = zero;
        *(LAS u32x4*)(pl + lane * 16u + 1024u) = zero;
        const u32 rb = OFF_DATA + idp * BSTR;
        const u32 ox = in_vgpr(rb + 2816u + lane * 2u), on = in_vgpr(rb + 2048u + lane * 12u),
                  ol = in_vgpr(rb + hf * 1024u + lane * 16u);
        u32 hp;
        for (;;) {
            hp = __hip_atomic_load(prog + 32u + (lw8 & 4u) + uh + (lane & 1u) * 2u, __ATOMIC_ACQUIRE,
                                   __HIP_MEMORY_SCOPE_WORKGROUP);
            if (__all(hp != 0u)) break;
            __builtin_amdgcn_s_sleep(1);
        }
        FENCE();
        u32x4 l[2], n[2];
        u32 x[2];
#pragma unroll
        for (int k = 0; k < 2; ++k) {
            x[k] = __builtin_amdgcn_raw_buffer_load_b16(lrs, ox, k * 2u * BSTR, SYS);
            n[k] = __builtin_amdgcn_raw_buffer_load_b128(lrs, on, k * 2u * BSTR, SYS);
            l[k] = __builtin_amdgcn_raw_buffer_load_b128(lrs, ol, k * 2u * BSTR, SYS);
        }
        const u32 tmo = __hip_atomic_load((GAS u32*)(myws + OFF_TMO), __ATOMIC_RELAXED, __HIP_MEMORY_SCOPE_AGENT);
        SB();
        u32x4 val[2][2];
#pragma unroll
        for (int k = 0; k < 2; ++k) {
            const u32 hk = __builtin_amdgcn_readlane(hp, k);
            const u32 pos = x[k] + lane * 2u;
            const bool use = x[k] >= 0x800u && ((pos >> 10) & 1u) == hf && lane < ((hk & H_LONG) ? 64u : 32u);
            FENCE();
            if (use) *(LAS u16*)(pl + ((pos & 0x3FFu) << 1)) = (u16)((x[k] >> 1) & 0x7C00u);
            FENCE();
            const u32x4 p0 = *(LAS u32x4*)(pl + lane * 16u), p1 = *(LAS u32x4*)(pl + lane * 16u + 1024u);
            FENCE();
            if (k == 0 && use) *(LAS u16*)(pl + ((pos & 0x3FFu) << 1)) = 0;
            const u64 b7 = make_base(hk);
            val[k][0] = (hf ? unpack_piece(l[k], n[k], b7, m80, 2) : unpack_piece(l[k], n[k], b7, m80, 0)) ^ p0;
            val[k][1] = (hf ? unpack_piece(l[k], n[k], b7, m80, 3) : unpack_piece(l[k], n[k], b7, m80, 1)) ^ p1;
            if (__builtin_expect(hk & H_RAW, 0)) {
                val[k][0] = __builtin_amdgcn_raw_buffer_load_b128(lrs, ol + hf * 1024u, k * 2u * BSTR, SYS);
                val[k][1] = __builtin_amdgcn_raw_buffer_load_b128(lrs, ol + hf * 1024u + 1024u, k * 2u * BSTR, SYS);
            }
        }
        const u32 poison = tmo == call ? 0x7FC07FC0u : 0u;
        const rsrc_t ro = make_rsrc(P.out, 4u << 20);
        const u32 oo = ((wt >> 3) * 64u + uh * 32u + (lane & 31u)) * 1024u + (wt & 7u) * 128u + hf * 64u + (lane >> 5) * 16u;
#pragma unroll
        for (int j = 0; j < 2; ++j) {
            u32x4 o;
#pragma unroll
            for (int d = 0; d < 4; ++d) {
                const u32 a = val[0][j][d], b = val[1][j][d];
                float y0 = __builtin_bit_cast(float, a << 16) + __builtin_bit_cast(float, b << 16);
                float y1 = __builtin_bit_cast(float, a & 0xFFFF0000u) + __builtin_bit_cast(float, b & 0xFFFF0000u);
                o[d] = cvt2(silu(y0), silu(y1)) | poison;
            }
            __builtin_amdgcn_raw_buffer_store_b128(o, ro, oo + j * 32u, 0, 0);
        }
        return;
    }

    const u32 mw = wave - W_MFMA, wm = mw >> 1, wn = mw & 1u;
    const u32 rdA = (64u * wm + (lane & 31u)) * 128u + (((lane >> 5) ^ ((lane >> 1) & 7u)) * 16u);
    const u32 rdB = 16384u + (8u * (lane >> 5) + ((lane >> 2) & 3u)) * 256u +
                    (((wn * 8u + ((lane >> 4) & 1u) * 2u + (lane & 1u)) ^ (((lane >> 2) & 3u) << 2)) * 16u) +
                    ((lane >> 1) & 1u) * 8u;
    f32x16 acc[2][2];
#pragma unroll
    for (int a = 0; a < 2; ++a)
#pragma unroll
        for (int b = 0; b < 2; ++b)
#pragma unroll
            for (int r = 0; r < 16; ++r) acc[a][b][r] = 0.f;

    lds_u32p lprog = prog + W_LOAD + (lane & 7u);
    u32 lseen = 0;
    bf16x8 X[2][2], W[2][2];
    auto rd_x = [&](u32 buf, int ks, int mi) {
        return *(LAS bf16x8*)(lds + buf + ((rdA ^ (ks * 32u)) + mi * 4096u));
    };
    auto rd_w = [&](u32 buf, int ks, int ni) {
        u32 a = buf + (rdB ^ (ni * 64u)) + ks * 4096u;
        s16x4 lo = __builtin_amdgcn_ds_read_tr16_b64_v4i16((LAS s16x4*)(lds + a));
        s16x4 hi = __builtin_amdgcn_ds_read_tr16_b64_v4i16((LAS s16x4*)(lds + a + 1024u));
        return __builtin_bit_cast(bf16x8, __builtin_shufflevector(lo, hi, 0, 1, 2, 3, 4, 5, 6, 7));
    };
#define MMA(mi, ni, st) acc[mi][ni] = __builtin_amdgcn_mfma_f32_32x32x16_bf16(W[st][ni], X[st][mi], acc[mi][ni], 0, 0, 0)
    auto step = [&](u32 s, u32 buf, u32 nxt, bool more) {
#pragma unroll
        for (int ks = 0; ks < 4; ++ks) {
            const int st = ks & 1, k2 = (ks + 2) & 3;
            const u32 src = ks < 2 ? buf : nxt;
            const bool ld = ks < 2 || more;
            MMA(0, 0, st); SB();
            if (ks == 1 && more) lseen = prog_read(lprog);
            if (ks == 2 && more) prog_set(prog + wave, s + 4u);
            SB();
            MMA(0, 1, st); SB();
            if (ks == 2 && more) prog_wait(lprog, lseen, s + 2u);
            if (ld) X[st][0] = rd_x(src, k2, 0);
            SB();
            MMA(1, 1, st); SB();
            if (ld) W[st][1] = rd_w(src, k2, 1);
            SB();
            MMA(1, 0, st); SB();
            if (ld) { W[st][0] = rd_w(src, k2, 0); X[st][1] = rd_x(src, k2, 1); }
            SB();
        }
    };
    prog_wait(lprog, lseen, 1u);
#pragma unroll
    for (int st = 0; st < 2; ++st) {
        W[st][0] = rd_w(0, st, 0); X[st][0] = rd_x(0, st, 0);
        W[st][1] = rd_w(0, st, 1); X[st][1] = rd_x(0, st, 1);
    }
    for (u32 s = 0; s < (u32)STEPS - 4u; s += 4) {
        step(s, 0, SLOT, true);
        step(s + 1u, SLOT, 2 * SLOT, true);
        step(s + 2u, 2 * SLOT, 3 * SLOT, true);
        step(s + 3u, 3 * SLOT, 0, true);
    }
    step(28u, 0, SLOT, true);
    step(29u, SLOT, 2 * SLOT, true);
    step(30u, 2 * SLOT, 3 * SLOT, true);
    step(31u, 3 * SLOT, 0, false);

    {
        char* ows = P.ws[owner & 7u];
        const rsrc_t dst = make_rsrc(ows, WS_NEED);
        const u32 row = 512u * srcx + 256u * (cell & 1u) + 128u * (tm & 1u) + 64u * wm;
        const u32 col = 256u * ((cell >> 1) & 1u) + 128u * (tn & 1u) + 64u * wn;
        const u32 id = ID_PART + ((row >> 6) * 8u + (col >> 6)) * 4u + (cell >> 2) * 2u;
        const u32 bad = prog_read(prog + 48u);
        auto pack_half = [&](int h, Wire& p) {
            u32x4 v[4];
#pragma unroll
            for (int j = 0; j < 4; ++j) {
                const f32x16 c = acc[h][j >> 1];
                const int o = 8 * (j & 1);
                v[j] = u32x4{cvt2(c[o], c[o + 1]), cvt2(c[o + 2], c[o + 3]), cvt2(c[o + 4], c[o + 5]), cvt2(c[o + 6], c[o + 7])};
            }
            pack_block(v, p, lds + LDS_PACK + wave * 1024u, lane);
        };
        Wire p0, p1;
        pack_half(0, p0);
        const u32 chain = late ? (u32)CHAIN1 : (tn & 1u) ? (u32)CHAIN01 : (u32)CHAIN0;
        if (owner != rank) prog_wait(prog + ((mw & (chain - 1u)) ? wave - 1u : W_WATCH), 0u, 64u);
        if (__builtin_expect(p0.hdr & H_RAW, 0)) store_tail(dst, id, p0, lane);
        store_main(dst, id, p0, lane);
        pack_half(1, p1);
        asm volatile("s_waitcnt vmcnt(0)" ::: "memory");
        if (__builtin_expect(p1.hdr & H_RAW, 0)) store_tail(dst, id + 1u, p1, lane);
        store_main(dst, id + 1u, p1, lane);
        asm volatile("s_waitcnt vmcnt(0)" ::: "memory");
        if (lane < 2u && !bad) publish(ows, id + lane, call, lane ? p1.hdr : p0.hdr);
        prog_set(prog + wave, 64u);
    }
}

extern "C" void kernel_launch(int rank, int nranks, ncclComm_t comm, hipStream_t stream,
                              void* const* d_in, const int* in_sizes, int n_in,
                              void* d_out, int out_size, const int* in_dtypes, int out_dtype,
                              void* const* d_out_all, void* const* d_ws_all, size_t ws_bytes) {
    if (nranks != NRANKS || (unsigned)rank >= (unsigned)NRANKS || ws_bytes < (size_t)WS_NEED || n_in != 2 ||
        in_sizes[0] != 512 * 4096 || in_sizes[1] != 4096 * 512 || out_size != 4096 * 512) {
        fprintf(stderr, "kernel_launch: unexpected problem shape\n");
        return;
    }
    Params p;
    p.in[0] = (const char*)d_in[1];
    p.in[1] = (const char*)d_in[0];
    p.out = (char*)d_out;
    for (int r = 0; r < NRANKS; ++r) p.ws[r] = (char*)d_ws_all[r];
    p.myws = p.ws[rank];
    p.rank = (u32)rank;
    p.pad = 0;
    {
        unsigned char ord[128];
        int n = 0, nx[4] = {0, 0, 0, 0};
        const int lag[4] = {0, LAG_R, 0, LAG_B};
        for (int t = 1; t <= STEPS; ++t)
            for (int st = 0; st < 4; ++st)
                while (nx[st] < (t * (STEPS - lag[st]) + STEPS - 1) / STEPS) { ord[n++] = (unsigned char)(st * 32 + nx[st]); ++nx[st]; }
        while (n < 128)
            for (int st = 1; st < 4; st += 2)
                if (nx[st] < STEPS) { ord[n++] = (unsigned char)(st * 32 + nx[st]); ++nx[st]; }
        unsigned char rel[32] = {0};
        for (int k = 0; k < N_RELAY; ++k) {
            int pos = RELAY_SHIFT + (RELAY_LAST - RELAY_SHIFT) * k / (N_RELAY > 1 ? N_RELAY - 1 : 1);
            while (pos < 128 && (ord[pos] < 64 || ord[pos] >= 128)) ++pos;
            if (pos >= 128) { fprintf(stderr, "kernel_launch: bad relay plan\n"); return; }
            rel[k] = ord[pos] |= 128;
        }
        int top[32];
        for (int g = 0; g < 128; ++g) if ((ord[g] & 96) == 64) top[ord[g] & 31] = g;
        const int ownq = ((rank & 1) << 1) | (rank >> 2);
        int key[512];
        unsigned short quad[512];
        for (int g = 0; g < 128; ++g)
            for (int c = 0; c < 4; ++c) {
                const bool own = (ord[g] & 64) && c == ownq;
                int k = own ? (OWN_EARLY ? top[ord[g] & 31] * OWN_RATE / 100 : g) : (ord[g] & 128) ? g - RELAY_SHIFT : g;
                int q = 4 * g + c;
                const unsigned short code = (unsigned short)(ord[g] | (c << 8));
                for (; q > 0 && key[q - 1] > k; --q) { key[q] = key[q - 1]; quad[q] = quad[q - 1]; }
                key[q] = k; quad[q] = code;
            }
        unsigned char role[256];
        int np = 0, nr = 0;
        for (int b = 0; b < 256; ++b) role[b] = 255;
        for (int pass = 0; pass < 3; ++pass)
            for (int b = 0; b < 256; ++b) {
                const int xcd = b & 7, idx = b >> 3, tm = (xcd >> 1) * 4 + (idx & 3), po = idx >> 2;
                const bool local = (tm >> 1) == po || po == rank || po == (rank ^ 2);
                if (role[b] != 255 || (pass == 0 && !local) || (pass == 1 && (!(tm & 1) || ((tm >> 1) | 2) == (rank | 2)))) continue;
                if (np < PUSH_W) role[b] = (unsigned char)np++;
                else if (nr < RELAY_W) role[b] = (unsigned char)(160 + nr++);
            }
        auto word = [](const unsigned char* c) { return c[0] | (c[1] << 8) | (c[2] << 16) | ((u32)c[3] << 24); };
        for (int i = 0; i < 32; ++i) p.fill[i] = 0;
        for (int i = 0; i < 256; ++i) p.order[i] = quad[2 * i] | ((u32)quad[2 * i + 1] << 16);
        for (int i = 0; i < 8; ++i) p.relay[i] = word(rel + 4 * i);
        for (int i = 0; i < 64; ++i) p.role[i] = word(role + 4 * i);
    }
    hipLaunchKernelGGL(ag_gemm_silu, dim3(N_WG), dim3(N_WAVES * 64), 0, stream, p);
    HIP_CHECK(hipGetLastError());
}
```

```cpp
#include <hip/hip_runtime.h>
#include <rccl/rccl.h>
#include <cstdio>
#include <cstdint>
#include <cstddef>

#define HIP_CHECK(expr) do { hipError_t _e = (expr); if (_e != hipSuccess) { \
    fprintf(stderr, "HIP error %s at %s:%d\n", hipGetErrorString(_e), __FILE__, __LINE__); \
    return; } } while (0)

typedef unsigned u32;
typedef unsigned long long u64;
typedef unsigned short u16;
typedef u32 u32x4 __attribute__((ext_vector_type(4)));
typedef u32 u32x2 __attribute__((ext_vector_type(2)));
typedef float f32x16 __attribute__((ext_vector_type(16)));
typedef short s16x4 __attribute__((ext_vector_type(4)));
typedef unsigned short u16x2 __attribute__((ext_vector_type(2)));
typedef __bf16 bf16x8 __attribute__((ext_vector_type(8)));
typedef __bf16 bf16x2 __attribute__((ext_vector_type(2)));
typedef float f32x2 __attribute__((ext_vector_type(2)));
#define GAS __attribute__((address_space(1)))
#define LAS __attribute__((address_space(3)))
typedef __amdgpu_buffer_rsrc_t rsrc_t;
typedef LAS u32* lds_u32p;

constexpr int CHAIN0 = 2;
constexpr int CHAIN1 = 2;
constexpr int CHAIN01 = CHAIN0;
constexpr int POLL_SLEEP = 4;
constexpr int RED_SLEEP = 1;
constexpr int N_RELAY = 13;
constexpr int RELAY_W = 24;
constexpr int RELAY_SLEEP = 8;
constexpr int RELAY_SHIFT = 10;
constexpr int RELAY_LAST = 122;
constexpr int GATE_D0 = 300;
constexpr int GATE_D1 = 450;
constexpr int GATE_D2 = 600;
constexpr int OWN_RATE = 110;
constexpr int PUSHERS = 124;
constexpr int LAG_R = 8;
constexpr int LAG_B = 16;
constexpr int PUSH_PRIO = 3;
constexpr int GATE_AT = 2047;
constexpr int BSTRIDE = 4352;

constexpr int NRANKS = 8;
constexpr int STEPS = 32;
constexpr int N_WG = 256;
constexpr u32 W_WATCH = 1, W_MFMA = W_WATCH + 1, W_LOAD = W_MFMA + 4;
constexpr int N_WAVES = W_LOAD + 8;
constexpr int N_IN = 4096;
constexpr int N_PART = 2048;
constexpr int N_SRC = 2048;
constexpr int ID_PART = 2 * N_IN;
constexpr int ID_OWN = ID_PART + N_PART;
constexpr int N_BLK = ID_OWN + N_SRC;
constexpr int PUSH_W = PUSHERS;

constexpr u32 OFF_CNT = 0;
constexpr u32 OFF_TMO = 32768;
constexpr u32 OFF_TKT = 33024;
constexpr u32 OFF_GATE = 33280;
constexpr u32 OFF_FLAG = 65536;
constexpr u32 OFF_DATA = 1u << 20;
constexpr u32 BSTR = BSTRIDE;
constexpr u32 WS_NEED = OFF_DATA + N_BLK * BSTR;
constexpr u64 DEADLINE_TICKS = 40000000ull;
constexpr int SYS = 17;
constexpr u32 OOB = 0x80000000u;

constexpr u32 H_VALID = 1, H_LONG = 2, H_RAW = 4, H_BASE = 0x7F807F80;

constexpr u32 SLOT = 32768;
constexpr u32 LDS_CTL = 4 * SLOT;
constexpr u32 LDS_TAB = LDS_CTL + 256;
constexpr u32 LDS_PACK = LDS_TAB + 1024;
constexpr u32 LDS_RED = LDS_PACK + 8192;
constexpr u32 LDS_SIZE = LDS_RED + 8 * 2048;

struct Params {
    const char* in[2];
    char* out;
    char* ws[NRANKS];
    u32 rank;
    u32 pad;
    u32 fill[32];
    u32 relay[8];
    u32 role[64];
    u32 order[256];
    char* myws;
};

__device__ __forceinline__ rsrc_t make_rsrc(const void* p, u32 bytes) {
    return __builtin_amdgcn_make_buffer_rsrc((void*)p, 0, bytes, 0x00020000);
}
__device__ __forceinline__ u32 uni(u32 v) { return __builtin_amdgcn_readfirstlane(v); }
__device__ __forceinline__ u32 rotr(u32 v, u32 s) { return __builtin_amdgcn_alignbit(v, v, s); }
#define SB() __builtin_amdgcn_sched_barrier(0)
#define FENCE() asm volatile("" ::: "memory")

struct Wire { u32x4 a, b, c, d; u32 hdr; };

__device__ __forceinline__ u32 wave_sum_u32(u32 v) {
    v += (u32)__builtin_amdgcn_update_dpp(0, (int)v, 0x111, 0xf, 0xf, true);
    v += (u32)__builtin_amdgcn_update_dpp(0, (int)v, 0x112, 0xf, 0xf, true);
    v += (u32)__builtin_amdgcn_update_dpp(0, (int)v, 0x114, 0xf, 0xf, true);
    v += (u32)__builtin_amdgcn_update_dpp(0, (int)v, 0x118, 0xf, 0xf, true);
    return __builtin_amdgcn_readlane(v, 15) + __builtin_amdgcn_readlane(v, 31) +
           __builtin_amdgcn_readlane(v, 47) + __builtin_amdgcn_readlane(v, 63);
}

__device__ __forceinline__ u32 in_vgpr(u32 x) { asm("" : "+v"(x)); return x; }
__device__ __forceinline__ u32 bsel(u32 m, u32 a, u32 b) {
    return __builtin_amdgcn_bitop3_b32(m, a, b, 0xCA);
}

struct NoEarly { __device__ __forceinline__ void operator()(const Wire&) const {} };
template <class Early = NoEarly>
__device__ __forceinline__ void pack_block(const u32x4 (&v)[4], Wire& o, LAS char* scr, u32 lane, Early early = Early()) {
    *(LAS u16*)(scr + 768u + lane * 2u) = 0;
    const u32 m80 = in_vgpr(0x80808080u), m7c = in_vgpr(0x7C7C7C7Cu);
    u32 hv[8];
#pragma unroll
    for (int k = 0; k < 8; ++k)
        hv[k] = __builtin_amdgcn_perm(v[k >> 1][2 * (k & 1) + 1], v[k >> 1][2 * (k & 1)], 0x07050301u);
    u32 sum = 0;
#pragma unroll
    for (int k = 1; k < 8; k += 2) sum = __builtin_amdgcn_sad_u8(hv[k] & 0x7F7F7F7Fu, 0u, sum);
    const u32 base = min((max(wave_sum_u32(sum), 1895u) - 1895u) >> 9, 248u);
    const u16x2 b7 = __builtin_bit_cast(u16x2, in_vgpr(base * 0x00800080u));
    u32 lo[8], d[16], hd[8], px[8], todo = 0;
#pragma unroll
    for (int k = 0; k < 16; ++k) {
        const u32 t = v[k >> 2][k & 3];
        d[k] = __builtin_bit_cast(u32, __builtin_bit_cast(u16x2, t) - b7);
    }
#pragma unroll
    for (int k = 0; k < 8; ++k) lo[k] = __builtin_amdgcn_perm(d[2 * k + 1], d[2 * k], 0x06040200u);
    o.a = u32x4{lo[0], lo[1], lo[2], lo[3]};
    o.b = u32x4{lo[4], lo[5], lo[6], lo[7]};
    early(o);
#pragma unroll
    for (int k = 0; k < 8; ++k) {
        hd[k] = __builtin_amdgcn_perm(d[2 * k + 1], d[2 * k], 0x07050301u);
        const u32 q = hd[k] & m7c;
        px[k] = ((hv[k] | m80) - q) ^ hv[k];
        todo = bsel(m80, q + m7c, todo >> 1);
    }
    const u32 m83 = in_vgpr(0x83838383u), m1c = in_vgpr(0x1C1C1C1Cu);
    lds_u32p fp = (lds_u32p)(scr + lane * 12u);
#pragma unroll
    for (int k = 0; k < 3; ++k) {
        const u32 spare = k < 2 ? hd[6 + k] << 5 : bsel(0x40404040u, hv[6] >> 1, hv[7] >> 2);
        fp[k] = bsel(m83, bsel(m80, hv[2 * k], hd[2 * k]),
                     bsel(m1c, rotr(bsel(m80, hv[2 * k + 1], hd[2 * k + 1]), 29), spare));
    }
    const u32 mine_n = (u32)__builtin_popcount(todo);
    u32 inc = mine_n;
    inc += (u32)__builtin_amdgcn_update_dpp(0, (int)inc, 0x111, 0xf, 0xf, true);
    inc += (u32)__builtin_amdgcn_update_dpp(0, (int)inc, 0x112, 0xf, 0xf, true);
    inc += (u32)__builtin_amdgcn_update_dpp(0, (int)inc, 0x114, 0xf, 0xf, true);
    inc += (u32)__builtin_amdgcn_update_dpp(0, (int)inc, 0x118, 0xf, 0xf, true);
    inc += (u32)__builtin_amdgcn_update_dpp(0, (int)inc, 0x142, 0xa, 0xf, false);
    inc += (u32)__builtin_amdgcn_update_dpp(0, (int)inc, 0x143, 0xc, 0xf, false);
    u32 cnt = __builtin_amdgcn_readlane(inc, 63);
    u32 slot2 = (inc - mine_n) * 2u;
    auto emit = [&]() {
        u32 i;
        asm("v_ffbl_b32 %0, %1" : "=v"(i) : "v"(todo));
        const u32 m0 = (u32)__builtin_amdgcn_sbfe((int)i, 0, 1), m1 = (u32)__builtin_amdgcn_sbfe((int)i, 1, 1),
                  m2 = (u32)__builtin_amdgcn_sbfe((int)i, 2, 1);
        const u32 pk = bsel(m2, bsel(m1, bsel(m0, px[7], px[6]), bsel(m0, px[5], px[4])),
                            bsel(m1, bsel(m0, px[3], px[2]), bsel(m0, px[1], px[0])));
        const u32 pos = (__umul24(i, 0x104u) & 0x604u) | (i >> 3) | (lane * 8u);
        const u32 ent = bsel(0x7FFu, pos - slot2, (pk >> (i & 24u)) << 9);
        *(LAS u16*)(scr + 768u + min(slot2 | (i & 128u), 128u)) = (u16)ent;
        todo &= todo - 1u;
        slot2 += 2u;
    };
    emit();
    emit();
    if (cnt <= 64u)
        while (__ballot(todo != 0u) != 0ull) emit();
    FENCE();
    o.c = *(LAS u32x4*)(scr + lane * 16u);
    o.d = v[3];
    FENCE();
    o.hdr = base * 0x00800080u | H_VALID | (cnt > 32u ? H_LONG : 0u);
    if (__builtin_expect(cnt > 64u, 0)) {
        o.a = v[0]; o.b = v[1]; o.c = v[2];
        o.hdr = H_VALID | H_RAW;
    }
}

__device__ __forceinline__ void store_ab(rsrc_t dst, u32 id, const Wire& p, u32 lane) {
    const u32 vo = OFF_DATA + id * BSTR + lane * 16u;
    __builtin_amdgcn_raw_buffer_store_b128(p.a, dst, vo, 0, SYS);
    __builtin_amdgcn_raw_buffer_store_b128(p.b, dst, vo + 1024u, 0, SYS);
}
__device__ __forceinline__ void store_c(rsrc_t dst, u32 id, const Wire& p, u32 lane) {
    const u32 nc = (p.hdr & H_RAW) ? 64u : (p.hdr & H_LONG) ? 56u : 52u;
    __builtin_amdgcn_raw_buffer_store_b128(p.c, dst, lane < nc ? OFF_DATA + id * BSTR + lane * 16u + 2048u : OOB, 0, SYS);
}
__device__ __forceinline__ void store_main(rsrc_t dst, u32 id, const Wire& p, u32 lane) {
    store_ab(dst, id, p, lane);
    store_c(dst, id, p, lane);
}
__device__ __forceinline__ void store_tail(rsrc_t dst, u32 id, const Wire& p, u32 lane) {
    __builtin_amdgcn_raw_buffer_store_b128(p.d, dst, OFF_DATA + id * BSTR + lane * 16u + 3072u, 0, SYS);
}

__device__ __forceinline__ u32 field(const u32x4& n, int k, u32 m80) {
    if (k < 6) return ((k & 1) ? rotr(n[k >> 1], 3) : n[k >> 1]) & 0x83838383u;
    u32 sg = n[2];
    asm("v_add_u32 %0, %1, %1" : "=v"(sg) : "v"(sg));
    if (k == 7) asm("v_add_u32 %0, %1, %1" : "=v"(sg) : "v"(sg));
    return bsel(m80, sg, n[k - 6] >> 5) & 0x83838383u;
}
__device__ __forceinline__ u64 make_base(u32 h) {
    u64 b = h & H_BASE;
    b |= b << 32;
    asm("" : "+s"(b));
    return b;
}
__device__ __forceinline__ u32x4 unpack_piece(const u32x4& l, const u32x4& n, u64 b7, u32 m80, int j) {
    const u32 h0 = field(n, 2 * j, m80), h1 = field(n, 2 * j + 1, m80);
    const u32 a = l[2 * (j & 1)], b = l[2 * (j & 1) + 1];
    typedef u64 u64x2 __attribute__((ext_vector_type(2)));
    const u32x4 t = {__builtin_amdgcn_perm(h0, a, 0x05010400u), __builtin_amdgcn_perm(h0, a, 0x07030602u),
                     __builtin_amdgcn_perm(h1, b, 0x05010400u), __builtin_amdgcn_perm(h1, b, 0x07030602u)};
    return __builtin_bit_cast(u32x4, __builtin_bit_cast(u64x2, t) + b7);
}

__device__ __forceinline__ void publish(char* dst_ws, u32 id, u32 call, u32 hdr) {
    __hip_atomic_store((GAS u64*)(dst_ws + OFF_FLAG + id * 8u), ((u64)call << 32) | hdr,
                       __ATOMIC_RELAXED, __HIP_MEMORY_SCOPE_SYSTEM);
}

__device__ __forceinline__ u64 poll_flag(GAS u64* p) {
    u64 zero = 0;
    asm("" : "+v"(zero));
    return __hip_atomic_fetch_or(p, zero, __ATOMIC_RELAXED, __HIP_MEMORY_SCOPE_SYSTEM);
}
__device__ __forceinline__ u32 flag_of(u32 id) { return (id & ~127u) | ((id & 31u) << 2) | ((id >> 5) & 3u); }

__device__ __forceinline__ bool give_up(char* myws, u32 call, u64 t0) {
    u64 dt = __builtin_amdgcn_s_memrealtime() - t0;
    if (dt < 100000ull) return false;
    GAS u32* tmo = (GAS u32*)(myws + OFF_TMO);
    u32 t = uni(__hip_atomic_load(tmo, __ATOMIC_RELAXED, __HIP_MEMORY_SCOPE_AGENT));
    if (t == call || dt > DEADLINE_TICKS) {
        __hip_atomic_store(tmo, call, __ATOMIC_RELAXED, __HIP_MEMORY_SCOPE_AGENT);
        asm volatile("s_waitcnt vmcnt(0)" ::: "memory");
        return true;
    }
    return false;
}

__device__ __forceinline__ float silu(float y) {
    float c = __builtin_amdgcn_fmed3f(y, -60.0f, 60.0f);
    float e = __builtin_amdgcn_exp2f(c * -1.4426950408889634f);
    return y * __builtin_amdgcn_rcpf(1.0f + e);
}
__device__ __forceinline__ u32 cvt2(float a, float b) {
    f32x2 f = {a, b};
    return __builtin_bit_cast(u32, __builtin_convertvector(f, bf16x2));
}

struct Tabs { u32x4 ord; u32 ka, off, ids, w0l, w0h, w1l, w1h, lox, low; };
struct Src { u32 id, own, idx, two; };
__device__ __forceinline__ char* mk_ptr(u32 lo, u32 hi) { return (char*)(((u64)hi << 32) | lo); }
__device__ __forceinline__ char* tab_ptr(u32 lo, u32 hi, u32 i) {
    return mk_ptr(__builtin_amdgcn_readlane(lo, i), __builtin_amdgcn_readlane(hi, i));
}
__device__ __forceinline__ Tabs make_tabs(u32 ka, u32x4 ord, u32 lane, u32 rank) {
    const u32 isx = (lane >> 4) & 1u, kh = lane & 1u, half = (lane >> 1) & 1u, q = (lane >> 2) & 3u;
    Tabs T;
    T.ka = ka;
    T.ord = ord;
    T.off = isx ? (256u * half + 32u * q) * 8192u + 4096u * kh : (2048u * kh + 16u * q) * 1024u + half * 512u;
    u32 d0 = isx ? half | (kh << 2) | ((q & 1u) << 1) : (half << 1) | (kh << 2) | (q & 1u);
    u32 d1 = d0 ^ (isx ? 2u : 1u);
    const u32 skip = isx ? rank : rank ^ 2u;
    if (d0 == rank || d0 == skip) { d0 = d1; d1 = rank; }
    const u32 two = d1 != rank && d1 != skip;
    T.ids = ((((isx ^ 1u) * 8u + rank) * 8u + q) * 32u) | (((isx ^ 1u) * 1024u + ((half | (kh << 1)) * 8u + q) * 32u) << 12) |
            (two << 31);
    const u32 wsd = offsetof(Params, ws) / 4;
    T.w0l = __shfl(ka, wsd + d0 * 2u); T.w0h = __shfl(ka, wsd + d0 * 2u + 1u);
    T.w1l = __shfl(ka, wsd + d1 * 2u); T.w1h = __shfl(ka, wsd + d1 * 2u + 1u);
    T.lox = (lane >> 3) * 8192u + (((lane & 7u) ^ (lane >> 4)) * 16u);
    T.low = (lane >> 4) * 1024u + (((lane & 15u) ^ ((lane >> 4) << 2)) * 16u);
    return T;
}
__device__ __forceinline__ Src fetch_src(const Tabs& T, u32 e, bool on, u32x4 (&v)[4]) {
    const u32 ln = (e >> 3) & 63u;
    const u32 o0 = __builtin_amdgcn_readlane(T.ord[0], ln), o1 = __builtin_amdgcn_readlane(T.ord[1], ln),
              o2 = __builtin_amdgcn_readlane(T.ord[2], ln), o3 = __builtin_amdgcn_readlane(T.ord[3], ln);
    const u32 code = ((e & 1024u) ? ((e & 512u) ? o3 : o2) : ((e & 512u) ? o1 : o0)) >> ((e & 4u) * 4u);
    const u32 t = code & 31u, hi = (code >> 5) & 1u, isx = (code >> 6) & 1u;
    Src s;
    s.idx = isx * 16u + (e & 3u) * 4u + ((code >> 8) & 3u);
    const u32 off = __builtin_amdgcn_readlane(T.off, s.idx) + (isx ? t * 128u + hi * 1048576u : t * 65536u + hi * 256u);
    const u32 pin = offsetof(Params, in) / 4 + isx * 2u;
    const rsrc_t rs = make_rsrc(mk_ptr(__builtin_amdgcn_readlane(T.ka, pin), __builtin_amdgcn_readlane(T.ka, pin + 1u)),
                                on ? 4u << 20 : 0u);
    const u32 lo = isx ? T.lox : T.low, lo1 = lo ^ (isx * 64u), stride = isx ? 65536u : 4096u;
#pragma unroll
    for (int j = 0; j < 4; ++j)
        v[j] = __builtin_amdgcn_raw_buffer_load_b128(rs, (j & 1) ? lo1 : lo, off + j * stride, 0);
    const u32 ids = __builtin_amdgcn_readlane(T.ids, s.idx);
    s.id = (ids & 0xFFFu) + hi * 128u + t;
    s.own = ID_OWN + ((ids >> 12) & 0x7FFu) + hi * 128u + t;
    s.two = (ids >> 31) & ~(code >> 7) & 1u;
    return s;
}

__device__ __forceinline__ u32 prog_read(lds_u32p p) {
    return __hip_atomic_load(p, __ATOMIC_RELAXED, __HIP_MEMORY_SCOPE_WORKGROUP);
}
__device__ __forceinline__ void prog_wait(lds_u32p p, u32 seen, u32 need) {
    if (__builtin_expect(__ballot(seen < need) != 0ull, 0)) {
        while (__ballot(prog_read(p) < need) != 0ull) __builtin_amdgcn_s_sleep(1);
    }
    FENCE();
}
__device__ __forceinline__ void prog_set(lds_u32p p, u32 v) {
    FENCE();
    __hip_atomic_store(p, v, __ATOMIC_RELAXED, __HIP_MEMORY_SCOPE_WORKGROUP);
    FENCE();
}

struct Set { u32x4 l0, l1, n; u32 x; };

__global__ void __launch_bounds__(N_WAVES * 64, 4) ag_gemm_silu(Params P) {
    __shared__ __attribute__((aligned(16))) char smem[LDS_SIZE];
    LAS char* lds = (LAS char*)smem;
    const u32 lane = threadIdx.x & 63u;
    const u32 wave = uni(threadIdx.x >> 6);
    const u32 gwave = blockIdx.x * 16u + wave;
    static_assert(offsetof(Params, role) == 256, "kernel argument layout");
    u32 ka = 0;
    u32x4 ord = {0u, 0u, 0u, 0u};
    if (wave == 0u) {
        GAS u32* kp = (GAS u32*)(u64)__builtin_amdgcn_kernarg_segment_ptr() + lane;
        ka = __hip_atomic_load(kp, __ATOMIC_RELAXED, __HIP_MEMORY_SCOPE_WAVEFRONT);
#pragma unroll
        for (int k = 0; k < 4; ++k)
            ord[k] = __hip_atomic_load(kp + offsetof(Params, order) / 4 + k * 64, __ATOMIC_RELAXED, __HIP_MEMORY_SCOPE_WAVEFRONT);
    }
    u32 rank = P.rank, rolew = P.role[(blockIdx.x >> 2) & 63u];
    char* myws = P.myws;
    asm volatile("" : "+s"(rank), "+s"(rolew), "+s"(myws) :: "memory");
    rank &= 7u;
    const rsrc_t mine = make_rsrc(myws, WS_NEED);
    lds_u32p prog = (lds_u32p)(lds + LDS_CTL);
    lds_u32p tab = (lds_u32p)(lds + LDS_TAB);
    const u32 xcd = blockIdx.x & 7u, idx = blockIdx.x >> 3;
    const u32 tm = (xcd >> 1) * 4u + (idx & 3u), tn = (idx >> 2) * 2u + (((idx >> 2) + xcd) & 1u);

    const bool diag = (tm >> 1) == (tn >> 1), jcol = !diag && (tn >> 1) == (rank ^ 2u);
    const u32 cell = diag ? tm >> 1 : jcol ? rank ^ 2u : rank;
    const u32 srcx = diag ? rank : jcol && (tm >> 1) == rank ? rank ^ 2u : tm >> 1;
    const u32 owner = diag || jcol ? rank : tn >> 1;
    const bool late = (tm & 1u) && (srcx | 2u) != (rank | 2u);
    auto first_id = [&](u32 par, u32 isB) {
        const u32 h = (isB ? tn : tm) & 1u, src = isB ? owner : srcx;
        const u32 own = ID_OWN + isB * 1024u + ((((cell >> isB) & 1u) | ((cell >> 2) << 1)) * 2u + h) * 128u;
        return src == rank ? own : par * N_IN + (isB * 16u + src * 2u + h) * 128u;
    };

    GAS u32* cnt = (GAS u32*)(myws + OFF_CNT + gwave * 8u);

    if (wave < W_WATCH) {
        u32 e = (rolew >> ((blockIdx.x & 3u) * 8u)) & 0xFFu;
        if (e >= (u32)PUSH_W) {
            const u32 cv = __hip_atomic_load(cnt, __ATOMIC_RELAXED, __HIP_MEMORY_SCOPE_AGENT);
            __builtin_amdgcn_s_barrier();
            if (e == 255u) return;
            const u32 call = uni(cv) + 1u;
            if (lane == 0) __hip_atomic_store(cnt, call, __ATOMIC_RELAXED, __HIP_MEMORY_SCOPE_AGENT);
            char* jws = P.ws[rank ^ 2u];
            const rsrc_t dst = make_rsrc(jws, WS_NEED);
            const u64 t0 = __builtin_amdgcn_s_memrealtime();
            for (u32 r = e - 160u; r < 12u * N_RELAY; r += RELAY_W) {
                const u32 g = r / 12u, a = (r % 12u) >> 1;
                const u32 code = (P.relay[(g >> 2) & 7u] >> ((g & 3u) * 8u)) & 0xFFu;
                const u32 src = rank ^ (a + (a ? 2u : 1u));
                const u32 id = (call & 1u) * N_IN +
                               (src * 8u + ((code >> 5) & 1u) * 4u + ((rank >> 1) & 1u) + (r & 1u) * 2u) * 32u + (code & 31u);
                u64 f;
                for (;;) {
                    f = 0;
                    if (lane == 0u) f = poll_flag((GAS u64*)(myws + OFF_FLAG + flag_of(id) * 8u));
                    if (uni((u32)(f >> 32)) >= call) break;
                    if (give_up(myws, call, t0)) return;
                    __builtin_amdgcn_s_sleep(RELAY_SLEEP);
                }
                FENCE();
                Wire w;
                w.hdr = uni((u32)f);
                w.a = __builtin_amdgcn_raw_buffer_load_b128(mine, lane * 16u, OFF_DATA + id * BSTR, SYS);
                w.b = __builtin_amdgcn_raw_buffer_load_b128(mine, lane * 16u + 1024u, OFF_DATA + id * BSTR, SYS);
                w.c = __builtin_amdgcn_raw_buffer_load_b128(mine, lane * 16u + 2048u, OFF_DATA + id * BSTR, SYS);
                store_main(dst, id, w, lane);
                if (__builtin_expect(w.hdr & H_RAW, 0)) {
                    w.d = __builtin_amdgcn_raw_buffer_load_b128(mine, lane * 16u + 3072u, OFF_DATA + id * BSTR, SYS);
                    store_tail(dst, id, w, lane);
                }
                asm volatile("s_waitcnt vmcnt(0)" ::: "memory");
                if (lane == 0) publish(jws, flag_of(id), call, w.hdr);
                FENCE();
            }
            return;
        }
        __builtin_amdgcn_s_setprio(PUSH_PRIO);
        u32x4 v[4], u[4];
        const u32 cv = __hip_atomic_load(cnt, __ATOMIC_RELAXED, __HIP_MEMORY_SCOPE_AGENT);
        const Tabs T = make_tabs(ka, ord, lane, rank);
        Src sa = fetch_src(T, e, true, u);
        Src sb = fetch_src(T, e + PUSH_W, true, v);
        __builtin_amdgcn_s_barrier();
        const u32 call = uni(cv) + 1u;
        const u32 idp = (call & 1u) * N_IN;
        const u32 tkoff = lane ? OOB : OFF_TKT;
        const u32 tbase = (call - 1u) * (u32)(N_SRC - 2 * PUSH_W) - 3u * (u32)PUSH_W;
        u32 tk = e + 2u * (u32)PUSH_W + tbase;
        Wire p;
        pack_block(u, p, lds + LDS_PACK + wave * 1024u, lane, [&](const Wire& w) {
            FENCE();
            store_ab(make_rsrc(tab_ptr(T.w0l, T.w0h, sa.idx), WS_NEED), sa.id + idp, w, lane);
            store_ab(make_rsrc(tab_ptr(T.w1l, T.w1h, sa.idx), sa.two ? WS_NEED : 0u), sa.id + idp, w, lane);
            FENCE();
        });
        bool ab = p.hdr & H_RAW;
        if (ab) asm volatile("s_waitcnt vmcnt(0)" ::: "memory");
        asm volatile("" : "+v"(v[0]), "+v"(v[1]), "+v"(v[2]), "+v"(v[3]));
        char* fptr = myws + OFF_CNT + gwave * 8u - OFF_FLAG;
        u32 fn = 1u, fh = call;
        bool hasb = true, go;
        do {
            if (lane < fn) publish(fptr, 0u, call, fh);
            FENCE();
            char* w0 = tab_ptr(T.w0l, T.w0h, sa.idx);
            char* w1 = tab_ptr(T.w1l, T.w1h, sa.idx);
            const rsrc_t r0 = make_rsrc(w0, WS_NEED), r1 = make_rsrc(w1, sa.two ? WS_NEED : 0u);
            const u32 id = sa.id + idp;
            if (__builtin_expect(ab, 1)) store_ab(r0, id, p, lane);
            store_c(r0, id, p, lane);
            if (sa.two) {
                if (__builtin_expect(ab, 1)) store_ab(r1, id, p, lane);
                store_c(r1, id, p, lane);
            }
            ab = true;
            store_main(mine, sa.own, p, lane);
            if (__builtin_expect(p.hdr & H_RAW, 0)) {
                store_tail(r0, id, p, lane);
                store_tail(r1, id, p, lane);
                store_tail(mine, sa.own, p, lane);
            }
            FENCE();
            e = uni(tk) - tbase;
            const bool hasc = hasb && e < (u32)N_SRC;
            const Src sc = fetch_src(T, e, hasc, u);
            tk = (u32)__builtin_amdgcn_raw_ptr_buffer_atomic_add_i32(1, mine, hasc ? tkoff : OOB, 0, 0);
            if (hasb && e == (u32)GATE_AT)
                __builtin_amdgcn_raw_buffer_store_b64(u32x2{(u32)__builtin_amdgcn_s_memrealtime(), call}, mine,
                                                      lane ? OOB : OFF_GATE, 0, 16);
            fptr = (lane == 0u ? w0 : lane == 1u ? myws : w1) + (lane == 1u ? flag_of(sa.own) : flag_of(id)) * 8u;
            fn = 2u + sa.two;
            fh = p.hdr;
            if (hasb) pack_block(v, p, lds + LDS_PACK + wave * 1024u, lane);
            asm volatile("s_waitcnt vmcnt(0)"
                         : "+v"(fptr), "+v"(tk), "+v"(u[0]), "+v"(u[1]), "+v"(u[2]), "+v"(u[3]) :: "memory");
#pragma unroll
            for (int j = 0; j < 4; ++j) v[j] = u[j];
            sa = sb; sb = sc;
            go = hasb; hasb = hasc;
        } while (go);
        if (lane < fn) publish(fptr, 0u, call, fh);
        return;
    }

    if (wave == W_MFMA) prog[lane] = lane - W_MFMA < 4u ? 3u : 0u;
    if (wave >= W_LOAD && wave < W_LOAD + 4u) tab[(wave - W_LOAD) * 64u + lane] = 0u;
    const u32 cv = __hip_atomic_load(cnt, __ATOMIC_RELAXED, __HIP_MEMORY_SCOPE_AGENT);
    asm volatile("s_waitcnt lgkmcnt(0)" ::: "memory");
    __builtin_amdgcn_s_barrier();
    const u32 call = uni(cv) + 1u;
    if (lane == 0) __hip_atomic_store(cnt, call, __ATOMIC_RELAXED, __HIP_MEMORY_SCOPE_AGENT);
    const u32 par = call & 1u;

    if (wave == W_WATCH) {
        const u32 op = lane >> 5;
        GAS u64* fl = (GAS u64*)(myws + OFF_FLAG + ((op ? first_id(par, 1u) : first_id(par, 0u)) + (lane & 31u)) * 8u);
        lds_u32p ent = tab + (op * 4u + (lane & 3u)) * 32u + ((lane >> 2) & 7u);
        const u64 t0 = __builtin_amdgcn_s_memrealtime();
        u32 fx = 0, fw = 0;
        for (;;) {
            const u32 f0 = op ? fw : fx;
            bool ok = prog_read(ent + f0) != 0u;
            if (!ok) {
                const u64 f = poll_flag(fl + f0 * 4u);
                ok = (u32)(f >> 32) >= call;
                if (ok) __hip_atomic_store(ent + f0, (u32)f, __ATOMIC_RELEASE, __HIP_MEMORY_SCOPE_WORKGROUP);
            }
            const u64 miss = ~__ballot(ok);
            if (miss == 0ull && fx == 24u && fw == 24u) break;
            const u32 mx = (u32)miss, mw = (u32)(miss >> 32);
            const u32 nx = min(fx + (mx == 0u ? 8u : (mx & 0xFFFFu) ? 0u : 4u), 24u);
            const u32 nw = min(fw + (mw == 0u ? 8u : (mw & 0xFFFFu) ? 0u : 4u), 24u);
            const bool full = (mx == 0u && nx != fx) || (mw == 0u && nw != fw);
            const bool tail = fx == 24u && fw == 24u && ((mx | mw) & 0xFFFFu) == 0u;
            fx = nx; fw = nw;
            if (full) continue;
            if (give_up(myws, call, t0)) {
                prog_set(prog + 48u, 1u);
#pragma unroll
                for (int i = 0; i < 4; ++i)
                    __hip_atomic_store(tab + i * 64 + lane, H_VALID, __ATOMIC_RELEASE, __HIP_MEMORY_SCOPE_WORKGROUP);
                break;
            }
            if (tail) __builtin_amdgcn_s_sleep(1);
            else __builtin_amdgcn_s_sleep(POLL_SLEEP);
        }
        for (;;) {
            const u64 g = __hip_atomic_load((GAS u64*)(myws + OFF_GATE), __ATOMIC_RELAXED, __HIP_MEMORY_SCOPE_AGENT);
            if (uni((u32)(g >> 32)) >= call) {
                const u32 hold = late ? GATE_D2 : (tn & 1u) ? GATE_D1 : GATE_D0;
                while ((u32)__builtin_amdgcn_s_memrealtime() - uni((u32)g) < hold) __builtin_amdgcn_s_sleep(2);
                break;
            }
            if (give_up(myws, call, t0)) break;
            __builtin_amdgcn_s_sleep(4);
        }
        prog_set(prog + W_WATCH, 64u);
        GAS u64* fp = (GAS u64*)(myws + OFF_FLAG + (ID_PART + blockIdx.x * 8u + (lane & 7u)) * 8u);
        for (;;) {
            u64 f = ~0ull;
            if (lane < 8u) f = poll_flag(fp);
            const bool ok = (u32)(f >> 32) >= call;
            if (lane < 8u)
                __hip_atomic_store(prog + 32u + lane, ok ? (u32)f : 0u, __ATOMIC_RELEASE, __HIP_MEMORY_SCOPE_WORKGROUP);
            if (__all(ok)) break;
            if (give_up(myws, call, t0)) {
                __hip_atomic_store(prog + 32u + (lane & 7u), H_VALID, __ATOMIC_RELEASE, __HIP_MEMORY_SCOPE_WORKGROUP);
                break;
            }
            __builtin_amdgcn_s_sleep(RED_SLEEP);
        }
        return;
    }

    if (wave >= W_LOAD) {
        const u32 lw8 = wave - W_LOAD, lw = lw8 & 3u, isB = lw8 >> 2;
        const u32 id0 = first_id(par, isB) + lw * 32u;
        const u32 blk = lw8 * 4096u;
        const u32 m80 = in_vgpr(0x80808080u), mffc = in_vgpr(0xFFCu), vblk = in_vgpr(blk), lane4 = lane * 4u;
        const u32 doff = OFF_DATA + id0 * BSTR + lane * 16u;
        const u32 noff = OFF_DATA + id0 * BSTR + lane * 12u;
        const u32 xlong = OFF_DATA + id0 * BSTR + lane * 2u, xshort = lane < 32u ? xlong : OOB;
        lds_u32p row = lane < 32u ? tab + lw8 * 32u + lane : prog + W_MFMA + (lane & 3u);
        const u32 inc = lane >> 5;
        u32 need = 0;
        lds_u32p myprog = (lds_u32p)(size_t)in_vgpr(LDS_CTL + wave * 4u);
        rsrc_t lrs = mine;
        asm volatile("" : "+s"(lrs));
        Set R[4];
        u32 H[4] = {0, 0, 0, 0};
#pragma unroll
        for (int q = 0; q < 4; ++q)
            asm volatile("" : "=v"(R[q].l0), "=v"(R[q].l1), "=v"(R[q].n), "=v"(R[q].x));
        u32 rel = 0, hv = 0;
        auto fetch = [&](int d, u32 s, u32 avail, Set& q, u32& h) {
            u32 tv, tw, ts;
            asm volatile("s_cmp_lg_u32 %[rel], %[d]\n\ts_cbranch_scc1 .Lskip%=\n\t"
                         "s_cmp_le_u32 %[av], %[d]\n\ts_cbranch_scc1 .Lskip%=\n\t"
                         "s_add_u32 %[ts], %[s], %[d]\n\t"
                         "v_readlane_b32 %[h], %[hv], %[ts]\n\t"
                         "s_add_u32 %[rel], %[rel], 1\n\t"
                         "s_mul_i32 %[ts], %[ts], %[bs]\n\t"
                         "s_bitcmp1_b32 %[h], 2\n\ts_cselect_b64 vcc, -1, 0\n\ts_nop 1\n\t"
                         "v_cndmask_b32 %[tv], %[noff], %[doff], vcc\n\t"
                         "s_bitcmp1_b32 %[h], 1\n\ts_cselect_b64 vcc, -1, 0\n\ts_nop 1\n\t"
                         "v_cndmask_b32 %[tw], %[xs], %[xl], vcc\n\t"
                         "buffer_load_dwordx4 %[n], %[tv], %[rs], %[ts] offen offset:2048 sc0 sc1\n\t"
                         "buffer_load_dwordx4 %[l0], %[doff], %[rs], %[ts] offen sc0 sc1\n\t"
                         "buffer_load_dwordx4 %[l1], %[doff], %[rs], %[ts] offen offset:1024 sc0 sc1\n\t"
                         "buffer_load_ushort %[x], %[tw], %[rs], %[ts] offen offset:2816 sc0 sc1\n"
                         ".Lskip%=:"
                         : [x] "+v"(q.x), [n] "+v"(q.n), [l0] "+v"(q.l0), [l1] "+v"(q.l1), [h] "+s"(h),
                           [rel] "+s"(rel), [tv] "=&v"(tv), [tw] "=&v"(tw), [ts] "=&s"(ts)
                         : [d] "i"(d), [av] "s"(avail), [s] "s"(s), [hv] "v"(hv), [noff] "v"(noff),
                           [doff] "v"(doff), [xs] "v"(xshort), [xl] "v"(xlong), [rs] "s"(lrs), [bs] "i"(BSTR)
                         : "vcc", "scc", "memory");
        };
        auto landed = [&](Set& q) {
            asm volatile("s_cmp_lt_u32 %[rel], 4\n\ts_cbranch_scc1 .La%=\n\ts_waitcnt vmcnt(12)\n\ts_branch .Lz%=\n"
                         ".La%=:\n\ts_cmp_lt_u32 %[rel], 3\n\ts_cbranch_scc1 .Lb%=\n\ts_waitcnt vmcnt(8)\n\ts_branch .Lz%=\n"
                         ".Lb%=:\n\ts_cmp_lt_u32 %[rel], 2\n\ts_cbranch_scc1 .Lc%=\n\ts_waitcnt vmcnt(4)\n\ts_branch .Lz%=\n"
                         ".Lc%=:\n\ts_waitcnt vmcnt(0)\n.Lz%=:"
                         : "+v"(q.x), "+v"(q.n), "+v"(q.l0), "+v"(q.l1) : [rel] "s"(rel) : "scc", "memory");
        };
        auto arrived = [&]() {
            const u32 ok = (u32)__ballot(hv != 0u);
            return ok == 0xFFFFFFFFu ? 32u : (u32)__builtin_ctz(~ok);
        };
        u32 lim = 0;
        for (u32 s0 = 0; s0 < (u32)STEPS; s0 += 4) {
#pragma unroll
            for (int q = 0; q < 4; ++q) {
                const u32 s = s0 + q;
                Set& c = R[q];
                LAS char* dst = lds + q * SLOT;
                const u32 hn = prog_read(row);
                const u32 lim0 = lim;
                if (__builtin_expect(s + 3u >= lim0, 0)) {
                    u32 rdy = arrived();
                    while (rdy <= s) {
                        __builtin_amdgcn_s_sleep(1);
                        hv = __hip_atomic_load(row, __ATOMIC_ACQUIRE, __HIP_MEMORY_SCOPE_WORKGROUP);
                        rdy = arrived();
                    }
                    const u32 avail = rdy - s;
#pragma unroll
                    for (int d = 0; d < 4; ++d) fetch(d, s, avail, R[(q + d) & 3], H[(q + d) & 3]);
                    landed(c);
                    rel -= 1u;
                    const u64 xm = (__ballot((hv & (H_LONG | H_RAW)) != 0u) | (1ull << 32)) >> (s + 4u);
                    lim = rel == 3u ? min(rdy, s + 4u + (u32)__builtin_ctzll(xm | (1ull << 40))) : 0u;
                }
                {
                    Set& nx = R[(q + 3) & 3];
                    u32 ts;
                    asm volatile("s_cmp_ge_u32 %[s3], %[lim]\n\ts_cbranch_scc1 .Lslow%=\n\t"
                                 "s_mul_i32 %[ts], %[s3], %[bs]\n\t"
                                 "v_readlane_b32 %[h], %[hv], %[s3]\n\t"
                                 "buffer_load_dwordx4 %[n], %[noff], %[rs], %[ts] offen offset:2048 sc0 sc1\n\t"
                                 "buffer_load_dwordx4 %[l0], %[doff], %[rs], %[ts] offen sc0 sc1\n\t"
                                 "buffer_load_dwordx4 %[l1], %[doff], %[rs], %[ts] offen offset:1024 sc0 sc1\n\t"
                                 "buffer_load_ushort %[x], %[xs], %[rs], %[ts] offen offset:2816 sc0 sc1\n\t"
                                 "s_waitcnt vmcnt(12)\n.Lslow%=:"
                                 : [n] "+v"(nx.n), [l0] "+v"(nx.l0), [l1] "+v"(nx.l1), [x] "+v"(nx.x), "+v"(c.x),
                                   "+v"(c.n), "+v"(c.l0), "+v"(c.l1), [h] "+s"(H[(q + 3) & 3]), [ts] "=&s"(ts)
                                 : [s3] "s"(s + 3u), [lim] "s"(lim0), [hv] "v"(hv), [doff] "v"(doff), [noff] "v"(noff),
                                   [xs] "v"(xshort), [rs] "s"(lrs), [bs] "i"(BSTR)
                                 : "scc", "memory");
                }
                const u64 b7 = make_base(H[q]);
                auto put = [&](int j, const u32x4& t) { *(LAS u32x4*)(dst + blk + lane * 16u + j * 1024u) = t; SB(); };
                u32x4 t0 = unpack_piece(c.l0, c.n, b7, m80, 0);
                asm volatile("" : "+v"(t0));
                hv |= hn;
                prog_wait(row, hn, need);
                need += inc;
                put(0, t0);
                put(1, unpack_piece(c.l0, c.n, b7, m80, 1));
                put(2, unpack_piece(c.l1, c.n, b7, m80, 2));
                put(3, unpack_piece(c.l1, c.n, b7, m80, 3));
                __hip_atomic_fetch_xor((lds_u32p)(dst + bsel(mffc, c.x + c.x + lane4, vblk)),
                                       ((c.x >> 1) & 0x7C00u) << ((c.x & 1u) << 4), __ATOMIC_RELAXED,
                                       __HIP_MEMORY_SCOPE_WORKGROUP);
                FENCE();
                if (__builtin_expect(H[q] & H_RAW, 0)) {
                    put(0, c.l0);
                    put(1, c.l1);
                    put(2, c.n);
                    put(3, __builtin_amdgcn_raw_buffer_load_b128(mine, doff + 3072u, s * BSTR, SYS));
                }
                prog_set(myprog, s + 1u);
                SB();
            }
        }
        const u32 wt = blockIdx.x * 2u + (lw8 >> 2), uh = (lw8 >> 1) & 1u, hf = lw8 & 1u;
        const u32 idp = ID_PART + wt * 4u + uh;
        LAS char* pl = lds + LDS_RED + lw8 * 2048u;
        const u32x4 zero = {0u, 0u, 0u, 0u};
        *(LAS u32x4*)(pl + lane * 16u) = zero;
        *(LAS u32x4*)(pl + lane * 16u + 1024u) = zero;
        const u32 rb = OFF_DATA + idp * BSTR;
        const u32 ox = in_vgpr(rb + 2816u + lane * 2u), on = in_vgpr(rb + 2048u + lane * 12u),
                  ol = in_vgpr(rb + hf * 1024u + lane * 16u);
        u32 hp;
        for (;;) {
            hp = __hip_atomic_load(prog + 32u + (lw8 & 4u) + uh + (lane & 1u) * 2u, __ATOMIC_ACQUIRE,
                                   __HIP_MEMORY_SCOPE_WORKGROUP);
            if (__all(hp != 0u)) break;
            __builtin_amdgcn_s_sleep(1);
        }
        FENCE();
        u32x4 l[2], n[2];
        u32 x[2];
#pragma unroll
        for (int k = 0; k < 2; ++k) {
            x[k] = __builtin_amdgcn_raw_buffer_load_b16(lrs, ox, k * 2u * BSTR, SYS);
            n[k] = __builtin_amdgcn_raw_buffer_load_b128(lrs, on, k * 2u * BSTR, SYS);
            l[k] = __builtin_amdgcn_raw_buffer_load_b128(lrs, ol, k * 2u * BSTR, SYS);
        }
        const u32 tmo = __hip_atomic_load((GAS u32*)(myws + OFF_TMO), __ATOMIC_RELAXED, __HIP_MEMORY_SCOPE_AGENT);
        SB();
        u32x4 val[2][2];
#pragma unroll
        for (int k = 0; k < 2; ++k) {
            const u32 hk = __builtin_amdgcn_readlane(hp, k);
            const u32 pos = x[k] + lane * 2u;
            const bool use = x[k] >= 0x800u && ((pos >> 10) & 1u) == hf && lane < ((hk & H_LONG) ? 64u : 32u);
            FENCE();
            if (use) *(LAS u16*)(pl + ((pos & 0x3FFu) << 1)) = (u16)((x[k] >> 1) & 0x7C00u);
            FENCE();
            const u32x4 p0 = *(LAS u32x4*)(pl + lane * 16u), p1 = *(LAS u32x4*)(pl + lane * 16u + 1024u);
            FENCE();
            if (k == 0 && use) *(LAS u16*)(pl + ((pos & 0x3FFu) << 1)) = 0;
            const u64 b7 = make_base(hk);
            val[k][0] = (hf ? unpack_piece(l[k], n[k], b7, m80, 2) : unpack_piece(l[k], n[k], b7, m80, 0)) ^ p0;
            val[k][1] = (hf ? unpack_piece(l[k], n[k], b7, m80, 3) : unpack_piece(l[k], n[k], b7, m80, 1)) ^ p1;
            if (__builtin_expect(hk & H_RAW, 0)) {
                val[k][0] = __builtin_amdgcn_raw_buffer_load_b128(lrs, ol + hf * 1024u, k * 2u * BSTR, SYS);
                val[k][1] = __builtin_amdgcn_raw_buffer_load_b128(lrs, ol + hf * 1024u + 1024u, k * 2u * BSTR, SYS);
            }
        }
        const u32 poison = tmo == call ? 0x7FC07FC0u : 0u;
        const rsrc_t ro = make_rsrc(P.out, 4u << 20);
        const u32 oo = ((wt >> 3) * 64u + uh * 32u + (lane & 31u)) * 1024u + (wt & 7u) * 128u + hf * 64u + (lane >> 5) * 16u;
#pragma unroll
        for (int j = 0; j < 2; ++j) {
            u32x4 o;
#pragma unroll
            for (int d = 0; d < 4; ++d) {
                const u32 a = val[0][j][d], b = val[1][j][d];
                float y0 = __builtin_bit_cast(float, a << 16) + __builtin_bit_cast(float, b << 16);
                float y1 = __builtin_bit_cast(float, a & 0xFFFF0000u) + __builtin_bit_cast(float, b & 0xFFFF0000u);
                o[d] = cvt2(silu(y0), silu(y1)) | poison;
            }
            __builtin_amdgcn_raw_buffer_store_b128(o, ro, oo + j * 32u, 0, 16);
        }
        return;
    }

    const u32 mw = wave - W_MFMA, wm = mw >> 1, wn = mw & 1u;
    const u32 rdA = (64u * wm + (lane & 31u)) * 128u + (((lane >> 5) ^ ((lane >> 1) & 7u)) * 16u);
    const u32 rdB = 16384u + (8u * (lane >> 5) + ((lane >> 2) & 3u)) * 256u +
                    (((wn * 8u + ((lane >> 4) & 1u) * 2u + (lane & 1u)) ^ (((lane >> 2) & 3u) << 2)) * 16u) +
                    ((lane >> 1) & 1u) * 8u;
    f32x16 acc[2][2];
#pragma unroll
    for (int a = 0; a < 2; ++a)
#pragma unroll
        for (int b = 0; b < 2; ++b)
#pragma unroll
            for (int r = 0; r < 16; ++r) acc[a][b][r] = 0.f;

    lds_u32p lprog = prog + W_LOAD + (lane & 7u);
    u32 lseen = 0;
    bf16x8 X[2][2], W[2][2];
    auto rd_x = [&](u32 buf, int ks, int mi) {
        return *(LAS bf16x8*)(lds + buf + ((rdA ^ (ks * 32u)) + mi * 4096u));
    };
    auto rd_w = [&](u32 buf, int ks, int ni) {
        u32 a = buf + (rdB ^ (ni * 64u)) + ks * 4096u;
        s16x4 lo = __builtin_amdgcn_ds_read_tr16_b64_v4i16((LAS s16x4*)(lds + a));
        s16x4 hi = __builtin_amdgcn_ds_read_tr16_b64_v4i16((LAS s16x4*)(lds + a + 1024u));
        return __builtin_bit_cast(bf16x8, __builtin_shufflevector(lo, hi, 0, 1, 2, 3, 4, 5, 6, 7));
    };
#define MMA(mi, ni, st) acc[mi][ni] = __builtin_amdgcn_mfma_f32_32x32x16_bf16(W[st][ni], X[st][mi], acc[mi][ni], 0, 0, 0)
    auto step = [&](u32 s, u32 buf, u32 nxt, bool more) {
#pragma unroll
        for (int ks = 0; ks < 4; ++ks) {
            const int st = ks & 1, k2 = (ks + 2) & 3;
            const u32 src = ks < 2 ? buf : nxt;
            const bool ld = ks < 2 || more;
            MMA(0, 0, st); SB();
            if (ks == 1 && more) lseen = prog_read(lprog);
            if (ks == 2 && more) prog_set(prog + wave, s + 4u);
            SB();
            MMA(0, 1, st); SB();
            if (ks == 2 && more) prog_wait(lprog, lseen, s + 2u);
            if (ld) X[st][0] = rd_x(src, k2, 0);
            SB();
            MMA(1, 1, st); SB();
            if (ld) W[st][1] = rd_w(src, k2, 1);
            SB();
            MMA(1, 0, st); SB();
            if (ld) { W[st][0] = rd_w(src, k2, 0); X[st][1] = rd_x(src, k2, 1); }
            SB();
        }
    };
    prog_wait(lprog, lseen, 1u);
#pragma unroll
    for (int st = 0; st < 2; ++st) {
        W[st][0] = rd_w(0, st, 0); X[st][0] = rd_x(0, st, 0);
        W[st][1] = rd_w(0, st, 1); X[st][1] = rd_x(0, st, 1);
    }
    for (u32 s = 0; s < (u32)STEPS - 4u; s += 4) {
        step(s, 0, SLOT, true);
        step(s + 1u, SLOT, 2 * SLOT, true);
        step(s + 2u, 2 * SLOT, 3 * SLOT, true);
        step(s + 3u, 3 * SLOT, 0, true);
    }
    step(28u, 0, SLOT, true);
    step(29u, SLOT, 2 * SLOT, true);
    step(30u, 2 * SLOT, 3 * SLOT, true);
    step(31u, 3 * SLOT, 0, false);

    {
        char* ows = P.ws[owner & 7u];
        const rsrc_t dst = make_rsrc(ows, WS_NEED);
        const u32 row = 512u * srcx + 256u * (cell & 1u) + 128u * (tm & 1u) + 64u * wm;
        const u32 col = 256u * ((cell >> 1) & 1u) + 128u * (tn & 1u) + 64u * wn;
        const u32 id = ID_PART + ((row >> 6) * 8u + (col >> 6)) * 4u + (cell >> 2) * 2u;
        const u32 bad = prog_read(prog + 48u);
        auto pack_half = [&](int h, Wire& p) {
            u32x4 v[4];
#pragma unroll
            for (int j = 0; j < 4; ++j) {
                const f32x16 c = acc[h][j >> 1];
                const int o = 8 * (j & 1);
                v[j] = u32x4{cvt2(c[o], c[o + 1]), cvt2(c[o + 2], c[o + 3]), cvt2(c[o + 4], c[o + 5]), cvt2(c[o + 6], c[o + 7])};
            }
            pack_block(v, p, lds + LDS_PACK + wave * 1024u, lane);
        };
        Wire p0, p1;
        pack_half(0, p0);
        const u32 chain = late ? (u32)CHAIN1 : (tn & 1u) ? (u32)CHAIN01 : (u32)CHAIN0;
        if (owner != rank) prog_wait(prog + ((mw & (chain - 1u)) ? wave - 1u : W_WATCH), 0u, 64u);
        if (__builtin_expect(p0.hdr & H_RAW, 0)) store_tail(dst, id, p0, lane);
        store_main(dst, id, p0, lane);
        pack_half(1, p1);
        asm volatile("s_waitcnt vmcnt(0)" ::: "memory");
        if (__builtin_expect(p1.hdr & H_RAW, 0)) store_tail(dst, id + 1u, p1, lane);
        store_main(dst, id + 1u, p1, lane);
        asm volatile("s_waitcnt vmcnt(0)" ::: "memory");
        if (lane < 2u && !bad) publish(ows, id + lane, call, lane ? p1.hdr : p0.hdr);
        prog_set(prog + wave, 64u);
    }
}

extern "C" void kernel_launch(int rank, int nranks, ncclComm_t comm, hipStream_t stream,
                              void* const* d_in, const int* in_sizes, int n_in,
                              void* d_out, int out_size, const int* in_dtypes, int out_dtype,
                              void* const* d_out_all, void* const* d_ws_all, size_t ws_bytes) {
    if (nranks != NRANKS || (unsigned)rank >= (unsigned)NRANKS || ws_bytes < (size_t)WS_NEED || n_in != 2 ||
        in_sizes[0] != 512 * 4096 || in_sizes[1] != 4096 * 512 || out_size != 4096 * 512) {
        fprintf(stderr, "kernel_launch: unexpected problem shape\n");
        return;
    }
    Params p;
    p.in[0] = (const char*)d_in[1];
    p.in[1] = (const char*)d_in[0];
    p.out = (char*)d_out;
    for (int r = 0; r < NRANKS; ++r) p.ws[r] = (char*)d_ws_all[r];
    p.myws = p.ws[rank];
    p.rank = (u32)rank;
    p.pad = 0;
    {
        unsigned char ord[128];
        int n = 0, nx[4] = {0, 0, 0, 0};
        const int lag[4] = {0, LAG_R, 0, LAG_B};
        for (int t = 1; t <= STEPS; ++t)
            for (int st = 0; st < 4; ++st)
                while (nx[st] < (t * (STEPS - lag[st]) + STEPS - 1) / STEPS) { ord[n++] = (unsigned char)(st * 32 + nx[st]); ++nx[st]; }
        while (n < 128)
            for (int st = 1; st < 4; st += 2)
                if (nx[st] < STEPS) { ord[n++] = (unsigned char)(st * 32 + nx[st]); ++nx[st]; }
        unsigned char rel[32] = {0};
        for (int k = 0; k < N_RELAY; ++k) {
            int pos = RELAY_SHIFT + (RELAY_LAST - RELAY_SHIFT) * k / (N_RELAY > 1 ? N_RELAY - 1 : 1);
            while (pos < 128 && (ord[pos] < 64 || ord[pos] >= 128)) ++pos;
            if (pos >= 128) { fprintf(stderr, "kernel_launch: bad relay plan\n"); return; }
            rel[k] = ord[pos] |= 128;
        }
        int top[32];
        for (int g = 0; g < 128; ++g) if ((ord[g] & 96) == 64) top[ord[g] & 31] = g;
        const int ownq = ((rank & 1) << 1) | (rank >> 2);
        int key[512];
        unsigned short quad[512];
        for (int g = 0; g < 128; ++g)
            for (int c = 0; c < 4; ++c) {
                const bool own = (ord[g] & 64) && c == ownq;
                int k = own ? top[ord[g] & 31] * ((ord[g] & 32) ? OWN_RATE : 100) / 100 : (ord[g] & 128) ? g - RELAY_SHIFT : g;
                int q = 4 * g + c;
                const unsigned short code = (unsigned short)(ord[g] | (c << 8));
                for (; q > 0 && key[q - 1] > k; --q) { key[q] = key[q - 1]; quad[q] = quad[q - 1]; }
                key[q] = k; quad[q] = code;
            }
        unsigned char role[256];
        int np = 0, nr = 0;
        for (int b = 0; b < 256; ++b) role[b] = 255;
        for (int pass = 0; pass < 3; ++pass)
            for (int b = 0; b < 256; ++b) {
                const int xcd = b & 7, idx = b >> 3, tm = (xcd >> 1) * 4 + (idx & 3), po = idx >> 2;
                const bool local = (tm >> 1) == po || po == rank || po == (rank ^ 2);
                if (role[b] != 255 || (pass == 0 && !local) || (pass == 1 && (!(tm & 1) || ((tm >> 1) | 2) == (rank | 2)))) continue;
                if (np < PUSH_W) role[b] = (unsigned char)np++;
                else if (nr < RELAY_W) role[b] = (unsigned char)(160 + nr++);
            }
        auto word = [](const unsigned char* c) { return c[0] | (c[1] << 8) | (c[2] << 16) | ((u32)c[3] << 24); };
        for (int i = 0; i < 32; ++i) p.fill[i] = 0;
        for (int i = 0; i < 256; ++i) p.order[i] = quad[2 * i] | ((u32)quad[2 * i + 1] << 16);
        for (int i = 0; i < 8; ++i) p.relay[i] = word(rel + 4 * i);
        for (int i = 0; i < 64; ++i) p.role[i] = word(role + 4 * i);
    }
    hipLaunchKernelGGL(ag_gemm_silu, dim3(N_WG), dim3(N_WAVES * 64), 0, stream, p);
    HIP_CHECK(hipGetLastError());
}
```
